```python
import numpy as np

kernel_name = "gpu_gemm_ar_m4096_k4096_n16384_e4m3e4m3_dqsilu"


ABSMAX_RTOL = 0.03

COMPUTE_DTYPE = "float32"
EPILOGUE = "dequant_silu"
IN_QUANT = None
OUT_QUANT = None
IN_FP8 = "e4m3e4m3"
IN_DTYPES = {"x": "e4m3", "w_mat": "e4m3",
             "scale_x": "float32", "scale_w": "float32"}
WORLD_SIZE = 8

def _snap_fp8(x, mant_bits, max_abs, min_normal):
    a = np.abs(x.astype(np.float64))
    _m, _e = np.frexp(a)
    q = np.where(a >= min_normal, np.ldexp(1.0, _e - 1 - mant_bits),
                 min_normal * 2.0 ** -mant_bits)
    snapped = np.minimum(np.round(a / q) * q, max_abs)
    return (np.sign(x) * snapped).astype(np.float32)

def setup_inputs():

    x = _snap_fp8(np.random.randn(4096, 4096).astype(np.float32), 3, 448.0, 2.0 ** -6)
    w_mat = _snap_fp8(np.random.randn(4096, 16384).astype(np.float32), 3, 448.0, 2.0 ** -6)
    scale_x = (0.02 * (1.0 + np.random.rand(1))).astype(np.float32)
    scale_w = (0.02 * (1.0 + np.random.rand(1))).astype(np.float32)
    return {"x": x, "w_mat": w_mat, "scale_x": scale_x, "scale_w": scale_w}

def reference(x, w_mat, scale_x, scale_w):
    acc = (x @ w_mat).astype(np.float32)
    y = (acc * (scale_x * scale_w)).astype(np.float32)
    return (y / (1.0 + np.exp(-np.clip(y, -60.0, 60.0)))).astype(np.float32)
```

```cpp
#include <hip/hip_runtime.h>
#include <rccl/rccl.h>
#include <cstdio>

namespace gar {
#define LAS __attribute__((address_space(3)))
typedef int i32x4 __attribute__((ext_vector_type(4)));
typedef int i32x8 __attribute__((ext_vector_type(8)));
typedef unsigned u32x4 __attribute__((ext_vector_type(4)));
typedef float f32x4 __attribute__((ext_vector_type(4)));
typedef unsigned char u8;
typedef __amdgpu_buffer_rsrc_t rsrc_t;

constexpr int M = 4096, N = 16384, NR = 8, KS = 512;
constexpr int NT = 32;
constexpr int HTB = 16384;
constexpr int LDS_BYTES = 8 * HTB;
constexpr int GRID = 256, NCOPY = 16, NSLOT = 15, NROUND = 4;
constexpr unsigned OFF_CNT = 0, OFF_TMO = 256;
constexpr unsigned OFF_FLAGS = 65536;
constexpr unsigned FLAG_BYTES = 8 * 16 * 8 * 128;
constexpr unsigned OFF_DATA = 1u << 20;
constexpr unsigned XREG = (unsigned)M * KS;
constexpr unsigned WREG = (unsigned)N * KS;
constexpr unsigned XALL = NR * XREG, WALL = NR * WREG;
constexpr unsigned PAR_BYTES = XALL + WALL;
constexpr size_t WS_NEED = (size_t)OFF_DATA + 2 * (size_t)PAR_BYTES;
constexpr unsigned long long DEADLINE = 30000000ull;
constexpr int RSRC3 = 0x00020000;

struct Params {
  const u8* x; const u8* w; const float* sx; const float* sw; float* out;
  u8* ws[NR];
  int rank; int pad;
};

__device__ __forceinline__ i32x8 cat(i32x4 a, i32x4 b) { return __builtin_shufflevector(a, b, 0, 1, 2, 3, 4, 5, 6, 7); }
__device__ __forceinline__ unsigned rfl(unsigned v) { return __builtin_amdgcn_readfirstlane(v); }
__device__ __forceinline__ rsrc_t mk_rsrc(const void* p, unsigned bytes) {
  const unsigned long long a = (unsigned long long)p;
  const unsigned lo = rfl((unsigned)a), hi = rfl((unsigned)(a >> 32));
  return __builtin_amdgcn_make_buffer_rsrc((void*)(((unsigned long long)hi << 32) | lo), (short)0, (int)bytes, RSRC3);
}
#define WAIT_V(n) asm volatile("s_waitcnt vmcnt(" #n ")" ::: "memory")
#define WAIT_L(n) asm volatile("s_waitcnt lgkmcnt(" #n ")" ::: "memory")
#define BAR __builtin_amdgcn_s_barrier()
#define SCHED __builtin_amdgcn_sched_barrier(0)

__device__ __forceinline__ void copy_role(const Params& p, unsigned call, int j) {
  const int tid = threadIdx.x, v = rfl(tid >> 6), lane = tid & 63, quad = v >> 2, g = v & 3, ks = lane >> 4, fr = lane & 15;
  const unsigned rank = (unsigned)p.rank;
  const unsigned par = OFF_DATA + (call & 1u) * PAR_BYTES;
  rsrc_t dst[NR];
#pragma unroll
  for (int d = 0; d < NR; ++d) dst[d] = mk_rsrc(p.ws[(rank + 1 + d) & 7], (unsigned)WS_NEED);
  const rsrc_t srcx = mk_rsrc(p.x, (unsigned)M * KS), srcw = mk_rsrc(p.w, (unsigned)N * KS);
  unsigned* flagp = (unsigned*)(p.ws[lane & 7] + OFF_FLAGS + ((rank * 16 + j) * 8 + v) * 128);
  const unsigned base = call * 32u;
  u32x4 raw[4];
  auto issue_loads = [&](int e) {
    if (e < 16 && quad == 0) {
      const int c = e >> 1;
#pragma unroll
      for (int t = 0; t < 4; ++t)
        raw[t] = __builtin_bit_cast(u32x4, __builtin_amdgcn_raw_buffer_load_b128(srcx, (g * 64 + t * 16 + fr) * KS + ks * 16, j * 256 * KS + c * 64, 0));
    } else {
      const int r = e < 16 ? 0 : 1 + ((e - 16) >> 3), c = e < 16 ? (e >> 1) : ((e - 16) & 7);
      const int pn = j < NSLOT ? NSLOT * r + j : NSLOT * NROUND + r;
#pragma unroll
      for (int i = 0; i < 16; ++i)
        raw[i >> 2][i & 3] = __builtin_amdgcn_raw_buffer_load_b32(srcw, (ks * 16 + i) * N + g * 64 + fr * 4, c * 64 * N + pn * 256, 0);
    }
  };
  issue_loads(quad);
  for (int e = quad; e < 40; e += 2) {
    u32x4 o[4]; unsigned voff, step;
    if (e < 16 && quad == 0) {
      const int c = e >> 1;
#pragma unroll
      for (int t = 0; t < 4; ++t) o[t] = raw[t];
      voff = par + rank * XREG + ((j * NT + 4 * c + ks) * 2 + (g >> 1)) * 2048 + (g & 1) * 1024 + fr * 16;
      step = 256;
    } else {
      const int r = e < 16 ? 0 : 1 + ((e - 16) >> 3), c = e < 16 ? (e >> 1) : ((e - 16) & 7);
      const int pn = j < NSLOT ? NSLOT * r + j : NSLOT * NROUND + r;
#pragma unroll
      for (int q = 0; q < 4; ++q) {
        const unsigned t0 = __builtin_amdgcn_perm(raw[q][1], raw[q][0], 0x05010400u), t1 = __builtin_amdgcn_perm(raw[q][1], raw[q][0], 0x07030602u);
        const unsigned t2 = __builtin_amdgcn_perm(raw[q][3], raw[q][2], 0x05010400u), t3 = __builtin_amdgcn_perm(raw[q][3], raw[q][2], 0x07030602u);
        o[0][q] = __builtin_amdgcn_perm(t2, t0, 0x05040100u); o[1][q] = __builtin_amdgcn_perm(t2, t0, 0x07060302u);
        o[2][q] = __builtin_amdgcn_perm(t3, t1, 0x05040100u); o[3][q] = __builtin_amdgcn_perm(t3, t1, 0x07060302u);
      }
      voff = par + XALL + rank * WREG + ((pn * NT + 4 * c + ks) * 2 + (g >> 1)) * 2048 + (g & 1) * 256 + fr * 16;
      step = 512;
    }
#pragma unroll
    for (int t = 0; t < 4; ++t) asm volatile("" : "+v"(o[t]));
    if (e + 2 < 40) issue_loads(e + 2);
#pragma unroll
    for (int t = 0; t < 4; ++t)
#pragma unroll
      for (int d = 0; d < NR; ++d) __builtin_amdgcn_raw_buffer_store_b128(o[t], dst[d], voff + t * step, 0, 17);
    WAIT_V(0);
    if (lane < NR) __hip_atomic_store(flagp, base + (unsigned)(e >> 1) + 1u, __ATOMIC_RELAXED, __HIP_MEMORY_SCOPE_SYSTEM);
    BAR;
  }
}

struct FlagQ { unsigned off; unsigned thr; };
__device__ __forceinline__ FlagQ flag_query(int src, int pm, int jw, int r, int c, unsigned base, int lane) {
  const int quadw = r == 0 ? 1 : (c & 1);
  const int nw = r == 0 ? c + 1 : 8 + 4 * (r - 1) + (c >> 1) + 1;
  const int nx = r == 0 ? c + 1 : 8;
  FlagQ q;
  const bool isx = (lane & 4) == 0;
  const int jj = isx ? pm : jw, vv = isx ? (lane & 3) : quadw * 4 + (lane & 3);
  q.off = lane < 8 ? OFF_FLAGS + ((src * 16 + jj) * 8 + vv) * 128 : OFF_TMO;
  q.thr = lane < 8 ? base + (unsigned)(isx ? nx : nw) : 0u;
  return q;
}

typedef __attribute__((address_space(1))) unsigned gu32;
__device__ __forceinline__ gu32* gptr(u8* p, unsigned off) { return (gu32*)(p + off); }
__device__ __noinline__ bool spin_flags(u8* ws, unsigned off, unsigned thr, unsigned call) {
  gu32* tmo = gptr(ws, OFF_TMO);
  gu32* fp = gptr(ws, off);
  const unsigned long long t0 = __builtin_amdgcn_s_memrealtime();
  for (;;) {
    const unsigned v = __hip_atomic_load(fp, __ATOMIC_RELAXED, __HIP_MEMORY_SCOPE_SYSTEM);
    if (__all((int)(v - thr) >= 0)) return false;
    if (rfl(__hip_atomic_load(tmo, __ATOMIC_RELAXED, __HIP_MEMORY_SCOPE_AGENT)) == call || __builtin_amdgcn_s_memrealtime() - t0 > DEADLINE) {
      __hip_atomic_store(tmo, call, __ATOMIC_RELAXED, __HIP_MEMORY_SCOPE_AGENT);
      WAIT_V(0);
      return true;
    }
    __builtin_amdgcn_s_sleep(8);
  }
}

__device__ __forceinline__ void store_out(f32x4 v, rsrc_t r, unsigned voff, unsigned soff) {
  asm volatile("buffer_store_dwordx4 %0, %1, %2, %3 offen nt\n\ts_nop 1" :: "v"(v), "v"(voff), "s"(r), "s"(soff) : "memory");
}
__device__ __forceinline__ float silu1(float a, float cz) {
  const float z = a * cz;
  const float t = __builtin_amdgcn_fmed3f(-z, -86.56170245333781f, 86.56170245333781f);
  const float e = __builtin_amdgcn_exp2f(t);
  const float d = __builtin_fmaf(e, 1.4426950408889634f, 1.4426950408889634f);
  return z * __builtin_amdgcn_rcpf(d);
}

__global__ void __launch_bounds__(512, 2) gemm_ag_kernel(Params p) {
  extern __shared__ __attribute__((aligned(16))) unsigned char shm[];
  LAS unsigned char* lds = (LAS unsigned char*)shm;
  u8* myws = p.ws[p.rank];
  const unsigned call = rfl(__hip_atomic_load((unsigned*)(myws + OFF_CNT), __ATOMIC_RELAXED, __HIP_MEMORY_SCOPE_AGENT)) / GRID + 1u;
  const int bid = blockIdx.x;
#define STAMP(i)
  STAMP(0);
  if (bid < NCOPY) { copy_role(p, call, bid); STAMP(1); }

  const int tid = threadIdx.x, wid = rfl(tid >> 6), lane = tid & 63, wr = wid >> 2, wc = wid & 3, fr = lane & 15, fq = lane >> 4;
  const unsigned base = call * 32u;
  const unsigned par = OFF_DATA + (call & 1u) * PAR_BYTES;
  const rsrc_t rs = mk_rsrc(myws + par, PAR_BYTES);
  const rsrc_t ro = mk_rsrc(p.out, (unsigned)M * N * 4u);
  const unsigned voff = lane * 16;
  const unsigned ldsw = (unsigned)wid * 2048u;
  const unsigned xw = (unsigned)wid * XREG, ww = XALL + (unsigned)wid * WREG;
  const float scale = p.sx[0] * p.sw[0];
  const float cz = scale * 1.4426950408889634f;
  bool dead = false;
#define STAGE(bufoff, soff) do { \
    __builtin_amdgcn_raw_ptr_buffer_load_lds(rs, (LAS void*)(lds + (bufoff) + ldsw), 16, voff, (soff), 0, 17); \
    __builtin_amdgcn_raw_ptr_buffer_load_lds(rs, (LAS void*)(lds + (bufoff) + ldsw), 16, voff, (soff), 1024, 17); } while (0)

  if (bid >= NCOPY) {
    const int gidx = bid - NCOPY, xcd = gidx & 7, idx = gidx >> 3, tl = (xcd >> 1) * 30 + idx;
    const int pm = (xcd & 1) * 8 + (tl & 7), slot = tl >> 3;
    const int aoff = fq * 2048 + wr * 256 + fr * 16, boff = fq * 2048 + wc * 512 + fr * 16;
#define SA(b, h) (((b) * 2 + (h)) * HTB)
#define SB(b, h) ((4 + (b) * 2 + (h)) * HTB)
#define LDA(dst, b, h) do { _Pragma("unroll") for (int m = 0; m < 4; ++m) _Pragma("unroll") for (int k = 0; k < 2; ++k) dst[m][k] = *(const LAS i32x4*)(lds + SA(b, h) + aoff + m * 512 + k * 8192); } while (0)
#define LDB(dst, b, h) do { _Pragma("unroll") for (int n = 0; n < 2; ++n) _Pragma("unroll") for (int k = 0; k < 2; ++k) dst[n][k] = *(const LAS i32x4*)(lds + SB(b, h) + boff + n * 256 + k * 8192); } while (0)
#define MMA(ai, bj, At, Bt) do { __builtin_amdgcn_s_setprio(1); _Pragma("unroll") for (int m = 0; m < 4; ++m) _Pragma("unroll") for (int n = 0; n < 2; ++n) \
        acc[ai][bj][m][n] = __builtin_amdgcn_mfma_scale_f32_16x16x128_f8f6f4(cat(Bt[n][0], Bt[n][1]), cat(At[m][0], At[m][1]), acc[ai][bj][m][n], 0, 0, 0, 0, 0, 0); __builtin_amdgcn_s_setprio(0); } while (0)
    f32x4 acc[2][2][4][2];
#pragma unroll
    for (int a = 0; a < 2; ++a)
#pragma unroll
      for (int b = 0; b < 2; ++b)
#pragma unroll
        for (int m = 0; m < 4; ++m)
#pragma unroll
          for (int n = 0; n < 2; ++n) acc[a][b][m][n] = (f32x4){0.f, 0.f, 0.f, 0.f};
    i32x4 At[4][2], B0[2][2], B1[2][2];
    const unsigned kstep = 4096, hstep = 2048, tstep = NT * 4096;
    const unsigned cB = xw + (unsigned)pm * tstep;
    unsigned cA = ww + (unsigned)slot * tstep;
    { const FlagQ q = flag_query(wid, pm, slot, 0, 0, base, lane);
      dead |= spin_flags(myws, q.off, q.thr, call); }
    STAMP(1);
    STAGE(SB(0, 0), cB); STAGE(SA(0, 0), cA); STAGE(SB(0, 1), cB + hstep); STAGE(SA(0, 1), cA + hstep);
    if (wr == 1) BAR;
    WAIT_V(4); BAR;
    STAGE(SB(1, 0), cB + kstep); STAGE(SA(1, 0), cA + kstep); STAGE(SB(1, 1), cB + hstep + kstep);
    WAIT_V(6); BAR;
    unsigned fval = 0;
    { const FlagQ q = flag_query(wid, pm, slot, 0, 0, base, lane); fval = q.thr; }
    for (int r = 0; r < NROUND; ++r) {
      const bool has_next = r + 1 < NROUND;
      const unsigned nA = has_next ? cA + NSLOT * tstep : cA;
      for (int t = 0; t < NT; t += 2) {
        const bool last = (t == NT - 2);
        const unsigned a1 = cA + (unsigned)(t + 1) * kstep;
        const unsigned a2 = last ? nA : cA + (unsigned)(t + 2) * kstep, b2 = last ? cB : cB + (unsigned)(t + 2) * kstep;
        const unsigned b1 = cB + (unsigned)(t + 1) * kstep;
        const unsigned a3 = a2 + kstep, b3 = b2 + kstep;
        const int rq = last ? (has_next ? r + 1 : r) : r, cq = last ? 0 : ((t + 2) >> 2);
        const FlagQ q = flag_query(wid, pm, slot, rq, cq, base, lane);
        const int tn = t + 4; const int rn = tn >= NT ? (has_next ? r + 1 : r) : r, cn = tn >= NT ? ((tn - NT) >> 2) : (tn >> 2);
        const FlagQ qn = flag_query(wid, pm, slot, rn, cn, base, lane);
        LDB(B0, 0, 0); SCHED; LDA(At, 0, 0); STAGE(SA(1, 1), a1 + hstep);
        WAIT_L(8); BAR; WAIT_L(0); MMA(0, 0, At, B0); BAR; SCHED;
        if (!__all((int)(fval - q.thr) >= 0)) dead |= spin_flags(myws, q.off, q.thr, call);
        LDB(B1, 0, 1); STAGE(SB(0, 0), b2);
        BAR; WAIT_L(0); MMA(0, 1, At, B1); BAR;
        LDA(At, 0, 1); STAGE(SA(0, 0), a2);
        BAR; WAIT_L(0); MMA(1, 0, At, B0); BAR; SCHED;
        STAGE(SB(0, 1), b2 + hstep);
        WAIT_V(6); BAR; MMA(1, 1, At, B1); BAR;
        asm volatile("global_load_dword %0, %1, %2 sc0 sc1" : "=v"(fval) : "v"(qn.off), "s"(myws) : "memory");
        LDB(B0, 1, 0); SCHED; LDA(At, 1, 0); STAGE(SA(0, 1), a2 + hstep);
        WAIT_L(8); BAR; WAIT_L(0); MMA(0, 0, At, B0); BAR; SCHED;
        LDB(B1, 1, 1); STAGE(SB(1, 0), b3);
        BAR; WAIT_L(0); MMA(0, 1, At, B1); BAR;
        LDA(At, 1, 1); STAGE(SA(1, 0), a3);
        BAR; WAIT_L(0); MMA(1, 0, At, B0); BAR; SCHED;
        STAGE(SB(1, 1), b3 + hstep);
        WAIT_V(6); asm volatile("" : "+v"(fval)); BAR; MMA(1, 1, At, B1); BAR;
        (void)b1;
      }
      STAMP(2 + 2 * r);
      {
        int tz = threadIdx.x; asm volatile("" : "+v"(tz));
        const int lane2 = tz & 63, fr2 = lane2 & 15, fq2 = lane2 >> 4;
        const int pn = NSLOT * r + slot;
        const unsigned ovoff = (unsigned)(wc * 32 + fq2 * 4) * (N * 4) + (unsigned)(wr * 64 + fr2 * 4) * 4;
        const unsigned osoff = (unsigned)(pm * 256) * (N * 4) + (unsigned)pn * 1024;
        const bool bad = dead || __any(lane2 >= 8 && fval == call);
        const float czz = bad ? __builtin_nanf("") : cz;
#pragma unroll
        for (int ai = 0; ai < 2; ++ai)
#pragma unroll
          for (int bj = 0; bj < 2; ++bj)
#pragma unroll
            for (int n = 0; n < 2; ++n)
#pragma unroll
              for (int j = 0; j < 4; ++j) {
                f32x4 o;
#pragma unroll
                for (int m = 0; m < 4; ++m) o[m] = silu1(acc[ai][bj][m][n][j], czz);
                store_out(o, ro, ovoff, osoff + (unsigned)(bj * 128 + n * 16 + j) * (N * 4) + ai * 512);
              }
      }
      STAMP(3 + 2 * r);
      if (!has_next) break;
#pragma unroll
      for (int a = 0; a < 2; ++a)
#pragma unroll
        for (int b = 0; b < 2; ++b)
#pragma unroll
          for (int m = 0; m < 4; ++m)
#pragma unroll
            for (int n = 0; n < 2; ++n) acc[a][b][m][n] = (f32x4){0.f, 0.f, 0.f, 0.f};
      cA = nA;
    }
    WAIT_V(0);
    if (wr == 0) BAR;
    BAR;
  }
  {
    const int xcd = bid & 7, idx = bid >> 3;
    const int tp = xcd >> 1, rb = (xcd & 1) * 32 + idx;
    const int pn = NSLOT * NROUND + tp, pm = rb >> 2;
    const int cg = wid & 3, rg = wid >> 2;
    constexpr int STG = 40960;
    const unsigned cA = ww + (unsigned)pn * (NT * 4096);
    const unsigned cB = xw + (unsigned)pm * (NT * 4096) + ((rb >> 1) & 1) * 2048 + (rb & 1) * 1024;
    const int aoff = (cg >> 1) * 16384 + fq * 2048 + (cg & 1) * 256 + fr * 16;
    const int boff = 32768 + fq * 1024 + rg * 512 + fr * 16;
#define TSTAGE(buf, s) do { \
      STAGE((buf) * STG, cA + (s) * 4096); STAGE((buf) * STG + 16384, cA + (s) * 4096 + 2048); \
      __builtin_amdgcn_raw_ptr_buffer_load_lds(rs, (LAS void*)(lds + (buf) * STG + 32768 + wid * 1024), 16, voff, cB + (s) * 4096, 0, 17); } while (0)
    f32x4 acc[4][2];
#pragma unroll
    for (int m = 0; m < 4; ++m)
#pragma unroll
      for (int n = 0; n < 2; ++n) acc[m][n] = (f32x4){0.f, 0.f, 0.f, 0.f};
    {
      const int grp = lane >> 2;
      const unsigned off = grp > 2 ? OFF_TMO : OFF_FLAGS + ((wid * 16 + (grp == 0 ? pm : NSLOT)) * 8 + (grp == 2 ? 4 : 0) + (lane & 3)) * 128;
      const unsigned thr = grp > 2 ? 0u : base + (grp == 0 ? 8u : (grp == 1 && tp == 0) ? 0u : 8u + 4u * tp);
      dead |= spin_flags(myws, off, thr, call);
    }
    STAMP(10);
    TSTAGE(0, 0); TSTAGE(1, 1);
    int buf = 0;
    for (int s = 0; s < NT; ++s) {
      if (s + 1 < NT) WAIT_V(5); else WAIT_V(0);
      BAR;
      const int nb = buf == 0 ? 2 : buf - 1;
      if (s + 2 < NT) TSTAGE(nb, s + 2);
      i32x4 At[4][2], Bt[2][2];
#pragma unroll
      for (int n = 0; n < 2; ++n)
#pragma unroll
        for (int k = 0; k < 2; ++k) Bt[n][k] = *(const LAS i32x4*)(lds + buf * STG + boff + n * 256 + k * 4096);
#pragma unroll
      for (int m = 0; m < 4; ++m)
#pragma unroll
        for (int k = 0; k < 2; ++k) At[m][k] = *(const LAS i32x4*)(lds + buf * STG + aoff + m * 512 + k * 8192);
#pragma unroll
      for (int m = 0; m < 4; ++m)
#pragma unroll
        for (int n = 0; n < 2; ++n)
          acc[m][n] = __builtin_amdgcn_mfma_scale_f32_16x16x128_f8f6f4(cat(Bt[n][0], Bt[n][1]), cat(At[m][0], At[m][1]), acc[m][n], 0, 0, 0, 0, 0, 0);
      buf = buf == 2 ? 0 : buf + 1;
    }
    BAR;
    const bool bad = dead || rfl(__hip_atomic_load((unsigned*)(myws + OFF_TMO), __ATOMIC_RELAXED, __HIP_MEMORY_SCOPE_AGENT)) == call;
    const float czz = bad ? __builtin_nanf("") : cz;
    const unsigned ovoff = (unsigned)(rg * 32 + fq * 4) * (N * 4) + (unsigned)(cg * 64 + fr * 4) * 4;
    const unsigned osoff = (unsigned)(rb * 64) * (N * 4) + (unsigned)pn * 1024;
#pragma unroll
    for (int n = 0; n < 2; ++n)
#pragma unroll
      for (int j = 0; j < 4; ++j) {
        f32x4 o;
#pragma unroll
        for (int m = 0; m < 4; ++m) o[m] = silu1(acc[m][n][j], czz);
        store_out(o, ro, ovoff, osoff + (unsigned)(n * 16 + j) * (N * 4));
      }
  }
  STAMP(11);
  if (threadIdx.x == 0) __hip_atomic_fetch_add((unsigned*)(myws + OFF_CNT), 1u, __ATOMIC_RELAXED, __HIP_MEMORY_SCOPE_AGENT);
}
}
#define HIP_CHECK(expr) do { hipError_t _e = (expr); if (_e != hipSuccess) { \
    fprintf(stderr, "HIP error %s at %s:%d\n", hipGetErrorString(_e), __FILE__, __LINE__); \
    return; } } while (0)

extern "C" void kernel_launch(int rank, int nranks, ncclComm_t comm, hipStream_t stream,
                              void* const* d_in, const int* in_sizes, int n_in,
                              void* d_out, int out_size, const int* in_dtypes, int out_dtype,
                              void* const* d_out_all, void* const* d_ws_all, size_t ws_bytes) {
  (void)comm; (void)in_dtypes; (void)out_dtype; (void)d_out_all;
  if (nranks != gar::NR || n_in != 4 || (unsigned)rank >= (unsigned)gar::NR || ws_bytes < gar::WS_NEED ||
      in_sizes[0] != gar::M * gar::KS || in_sizes[1] != gar::N * gar::KS || out_size != gar::M * gar::N) {
    fprintf(stderr, "kernel_launch: unexpected shapes\n");
    return;
  }
  static bool attr_set = false;
  if (!attr_set) {
    HIP_CHECK(hipFuncSetAttribute((const void*)gar::gemm_ag_kernel, hipFuncAttributeMaxDynamicSharedMemorySize, gar::LDS_BYTES));
    attr_set = true;
  }
  gar::Params p;
  p.x = (const gar::u8*)d_in[0]; p.w = (const gar::u8*)d_in[1];
  p.sx = (const float*)d_in[2]; p.sw = (const float*)d_in[3];
  p.out = (float*)d_out;
  for (int i = 0; i < gar::NR; ++i) p.ws[i] = (gar::u8*)d_ws_all[i];
  p.rank = rank; p.pad = 0;
  gar::gemm_ag_kernel<<<gar::GRID, 512, gar::LDS_BYTES, stream>>>(p);
  HIP_CHECK(hipGetLastError());
}
```

```cpp
#include <hip/hip_runtime.h>
#include <rccl/rccl.h>
#include <cstdio>

extern "C" __device__ float gar_fmul_legacy(float, float) __asm("llvm.amdgcn.fmul.legacy");
namespace gar {
__device__ __forceinline__ float fmul_single(float a, float b) { return gar_fmul_legacy(a, b); }
#define LAS __attribute__((address_space(3)))
typedef int i32x4 __attribute__((ext_vector_type(4)));
typedef int i32x8 __attribute__((ext_vector_type(8)));
typedef unsigned u32x4 __attribute__((ext_vector_type(4)));
typedef float f32x4 __attribute__((ext_vector_type(4)));
typedef float f32x2 __attribute__((ext_vector_type(2)));
typedef unsigned char u8;
typedef __amdgpu_buffer_rsrc_t rsrc_t;
typedef __attribute__((address_space(1))) unsigned gu32;

constexpr int M = 4096, N = 16384, NR = 8, KS = 512;
constexpr int NT = 32;
constexpr int HTB = 16384;
constexpr int LDS_BYTES = 8 * HTB;
constexpr int GRID = 256, NCOPY = 16, NSLOT = 15, NROUND = 4, NSTEP = 40;
constexpr unsigned OFF_CNT = 0, OFF_TMO = 256;
constexpr unsigned OFF_FLAGS = 65536;
constexpr unsigned OFF_DATA = 1u << 20;
constexpr unsigned XREG = (unsigned)M * KS;
constexpr unsigned WREG = (unsigned)N * KS;
constexpr unsigned XALL = NR * XREG, WALL = NR * WREG;
constexpr unsigned PAR_BYTES = XALL + WALL;
constexpr size_t WS_NEED = (size_t)OFF_DATA + 2 * (size_t)PAR_BYTES;
constexpr unsigned long long DEADLINE = 30000000ull;
constexpr int RSRC3 = 0x00020000;
#define SLEEPV 8
#define KPRE 8

struct Params {
  const u8* x; const u8* w; const float* sx; const float* sw; float* out;
  u8* ws[NR];
  int rank; int pad;
};

__device__ __forceinline__ i32x8 cat(i32x4 a, i32x4 b) { return __builtin_shufflevector(a, b, 0, 1, 2, 3, 4, 5, 6, 7); }
__device__ __forceinline__ unsigned rfl(unsigned v) { return __builtin_amdgcn_readfirstlane(v); }
__device__ __forceinline__ rsrc_t mk_rsrc(const void* p, unsigned bytes) {
  const unsigned long long a = (unsigned long long)p;
  const unsigned lo = rfl((unsigned)a), hi = rfl((unsigned)(a >> 32));
  return __builtin_amdgcn_make_buffer_rsrc((void*)(((unsigned long long)hi << 32) | lo), (short)0, (int)bytes, RSRC3);
}
#define STR2(x) #x
#define STR(x) STR2(x)
#define WAIT_V(n) asm volatile("s_waitcnt vmcnt(" STR(n) ")" ::: "memory")
#define WAIT_L(n) asm volatile("s_waitcnt lgkmcnt(" STR(n) ")" ::: "memory")
#define BAR __builtin_amdgcn_s_barrier()
#define SCHED __builtin_amdgcn_sched_barrier(0)
#define STAMP(i)
#define CSTAMP(e)

#define PIN16(d) asm volatile("" : "+v"(d[0]), "+v"(d[1]), "+v"(d[2]), "+v"(d[3]), "+v"(d[4]), "+v"(d[5]), "+v"(d[6]), "+v"(d[7]), \
                                   "+v"(d[8]), "+v"(d[9]), "+v"(d[10]), "+v"(d[11]), "+v"(d[12]), "+v"(d[13]), "+v"(d[14]), "+v"(d[15]))
#define PIN4(x) asm volatile("" : "+v"(x[0]), "+v"(x[1]), "+v"(x[2]), "+v"(x[3]))
__device__ __forceinline__ void copy_role(const Params& p, unsigned call, int j) {
  const int tid = threadIdx.x, g = rfl(tid >> 6), lane = tid & 63, ks = lane >> 4, fr = lane & 15;
  if (g >= 4) { for (int e = 0; e < NSTEP; ++e) BAR; return; }
  const unsigned rank = (unsigned)p.rank;
  const unsigned par = OFF_DATA + (call & 1u) * PAR_BYTES;
  u8* dst[NR];
#pragma unroll
  for (int d = 0; d < NR; ++d) { const unsigned long long a = (unsigned long long)p.ws[(rank + 1 + d) & 7]; dst[d] = (u8*)(((unsigned long long)rfl((unsigned)(a >> 32)) << 32) | rfl((unsigned)a)); }
  const rsrc_t srcx = mk_rsrc(p.x, (unsigned)M * KS), srcw = mk_rsrc(p.w, (unsigned)N * KS);
  gu32* flagp = (gu32*)(p.ws[lane & 7] + OFF_FLAGS + ((rank * 16 + j) * 4 + g) * 128);
  const unsigned base = call * 64u;
  const unsigned xvo = (g * 64 + fr) * KS + ks * 16, wvo = ks * 16 * N + g * 64 + fr * 4;
  const unsigned xdo = par + rank * XREG + (j * NT + ks) * 4096 + (g >> 1) * 2048 + (g & 1) * 1024 + fr * 16;
  const unsigned wdo = par + XALL + rank * WREG + ks * 4096 + (g >> 1) * 2048 + (g & 1) * 256 + fr * 16;
  unsigned d[16]; u32x4 xr[4], o[4];
#define XLOAD(c) do { _Pragma("unroll") for (int t = 0; t < 4; ++t) \
    asm volatile("buffer_load_dwordx4 %0, %1, %2, %3 offen" : "=v"(xr[t]) : "v"(xvo), "s"(srcx), "s"(j * 256 * KS + (c) * 64 + t * 16 * KS) : "memory"); } while (0)
#define WLOAD(pn, c) do { _Pragma("unroll") for (int i = 0; i < 16; ++i) \
    asm volatile("buffer_load_dword %0, %1, %2, %3 offen" : "=v"(d[i]) : "v"(wvo), "s"(srcw), "s"((c) * 64 * N + (pn) * 256 + i * N) : "memory"); } while (0)
#define WTRANS() do { _Pragma("unroll") for (int q = 0; q < 4; ++q) { \
    const unsigned t0 = __builtin_amdgcn_perm(d[4 * q + 1], d[4 * q], 0x05010400u), t1 = __builtin_amdgcn_perm(d[4 * q + 1], d[4 * q], 0x07030602u); \
    const unsigned t2 = __builtin_amdgcn_perm(d[4 * q + 3], d[4 * q + 2], 0x05010400u), t3 = __builtin_amdgcn_perm(d[4 * q + 3], d[4 * q + 2], 0x07030602u); \
    o[0][q] = __builtin_amdgcn_perm(t2, t0, 0x05040100u); o[1][q] = __builtin_amdgcn_perm(t2, t0, 0x07060302u); \
    o[2][q] = __builtin_amdgcn_perm(t3, t1, 0x05040100u); o[3][q] = __builtin_amdgcn_perm(t3, t1, 0x07060302u); } PIN4(o); } while (0)
#define PUSH1(q, voff, step) asm volatile("global_store_dwordx4 %0, %1, %2 sc0 sc1\n\ts_nop 1" :: "v"((voff) + ((q) >> 3) * (step)), "v"(o[(q) >> 3]), "s"(dst[(q) & 7]) : "memory")
#define STEP(e, voff, step, LOADS) do { \
    _Pragma("unroll") for (int q = 0; q < KPRE; ++q) PUSH1(q, voff, step); \
    WAIT_V(KPRE); \
    if ((e) >= 1) { if (lane < NR) __hip_atomic_store(flagp, base + (unsigned)(e), __ATOMIC_RELAXED, __HIP_MEMORY_SCOPE_SYSTEM); CSTAMP((e) - 1); } \
    LOADS; \
    _Pragma("unroll") for (int q = KPRE; q < 32; ++q) PUSH1(q, voff, step); \
    BAR; } while (0)
#define WAIT_LOADS() WAIT_V(32 - KPRE)
  const int pn0 = j < NSLOT ? j : NSLOT * NROUND;
  XLOAD(0);
  WAIT_V(0);
  for (int c = 0; c < 8; ++c) {
    WAIT_LOADS(); PIN4(xr);
#pragma unroll
    for (int t = 0; t < 4; ++t) o[t] = xr[t];
    PIN4(o);
    { const unsigned voff = xdo + c * 16384; STEP(2 * c, voff, 256, WLOAD(pn0, c)); }
    WAIT_LOADS(); PIN16(d);
    WTRANS();
    { const unsigned voff = wdo + pn0 * 131072 + c * 16384; STEP(2 * c + 1, voff, 512, XLOAD(c < 7 ? c + 1 : 7)); }
  }
  WAIT_V(0); PIN4(xr);
  if (lane < NR) __hip_atomic_store(flagp, base + 16u, __ATOMIC_RELAXED, __HIP_MEMORY_SCOPE_SYSTEM);
  WLOAD(j < NSLOT ? NSLOT + j : NSLOT * NROUND + 1, 0);
  WAIT_V(0);
  for (int e = 16; e < NSTEP; ++e) {
    const int r = 1 + ((e - 16) >> 3), c = (e - 16) & 7;
    const int pn = j < NSLOT ? NSLOT * r + j : NSLOT * NROUND + r;
    const int en = e + 1 < NSTEP ? e + 1 : e, rn = 1 + ((en - 16) >> 3), cn = (en - 16) & 7;
    const int pnn = j < NSLOT ? NSLOT * rn + j : NSLOT * NROUND + rn;
    WAIT_LOADS(); PIN16(d);
    WTRANS();
    const unsigned voff = wdo + pn * 131072 + c * 16384;
    STEP(e, voff, 512, WLOAD(pnn, cn));
  }
  WAIT_V(0); PIN16(d);
  if (lane < NR) __hip_atomic_store(flagp, base + (unsigned)NSTEP, __ATOMIC_RELAXED, __HIP_MEMORY_SCOPE_SYSTEM);
  CSTAMP(NSTEP - 1);
}

__device__ __forceinline__ unsigned flag_off(int src, int pm, int jw, int lane) {
  return lane < 8 ? OFF_FLAGS + ((src * 16 + ((lane & 4) == 0 ? pm : jw)) * 4 + (lane & 3)) * 128 : OFF_TMO;
}
__device__ __forceinline__ unsigned flag_thr(int r, int c, unsigned base, int lane) {
  const int nw = r == 0 ? 2 * c + 2 : 9 + 8 * r + c;
  const int nx = r == 0 ? 2 * c + 1 : 15;
  return lane < 8 ? base + (unsigned)((lane & 4) == 0 ? nx : nw) : 0u;
}
__device__ __noinline__ unsigned spin_flags(u8* ws, unsigned off, unsigned thr, unsigned call) {
  gu32* tmo = (gu32*)(ws + OFF_TMO);
  gu32* fp = (gu32*)(ws + off);
  const unsigned long long t0 = __builtin_amdgcn_s_memrealtime();
  for (;;) {
    const unsigned v = __hip_atomic_load(fp, __ATOMIC_RELAXED, __HIP_MEMORY_SCOPE_SYSTEM);
    if (__all((int)(v - thr) >= 0)) return v;
    if (rfl(__hip_atomic_load(tmo, __ATOMIC_RELAXED, __HIP_MEMORY_SCOPE_AGENT)) == call || __builtin_amdgcn_s_memrealtime() - t0 > DEADLINE) {
      __hip_atomic_store(tmo, call, __ATOMIC_RELAXED, __HIP_MEMORY_SCOPE_AGENT);
      WAIT_V(0);
      return (threadIdx.x & 63u) >= 8u ? call : thr;
    }
    __builtin_amdgcn_s_sleep(SLEEPV);
  }
}

__device__ __forceinline__ void store_out(f32x4 v, rsrc_t r, unsigned voff, unsigned soff) {
  asm volatile("buffer_store_dwordx4 %0, %1, %2, %3 offen nt\n\ts_nop 1" :: "v"(v), "v"(voff), "s"(r), "s"(soff) : "memory");
}
constexpr float LOG2E = 1.4426950408889634f, CLIP2 = 86.56170245333781f;
__device__ __forceinline__ void silu_rows(const f32x4 (&a)[4], float cz, f32x4 (&o)[4]) {
#pragma unroll
  for (int h = 0; h < 2; ++h) {
    f32x2 z[4], r[4];
#pragma unroll
    for (int m = 0; m < 4; ++m) {
      z[m] = (f32x2){a[m][2 * h], a[m][2 * h + 1]} * cz;
      f32x2 e;
      e.x = __builtin_amdgcn_exp2f(__builtin_amdgcn_fmed3f(-z[m].x, -CLIP2, CLIP2));
      e.y = __builtin_amdgcn_exp2f(__builtin_amdgcn_fmed3f(-z[m].y, -CLIP2, CLIP2));
      const f32x2 dd = __builtin_elementwise_fma(e, (f32x2){LOG2E, LOG2E}, (f32x2){LOG2E, LOG2E});
      r[m].x = __builtin_amdgcn_rcpf(dd.x); r[m].y = __builtin_amdgcn_rcpf(dd.y);
    }
#pragma unroll
    for (int m = 0; m < 4; ++m) {
      o[2 * h][m] = fmul_single(z[m].x, r[m].x); o[2 * h + 1][m] = fmul_single(z[m].y, r[m].y); }
  }
}

__global__ void __launch_bounds__(512, 2) gemm_ag_kernel(Params p) {
  extern __shared__ __attribute__((aligned(16))) unsigned char shm[];
  LAS unsigned char* lds = (LAS unsigned char*)shm;
  u8* myws = p.ws[p.rank];
  const unsigned call = rfl(__hip_atomic_load((gu32*)(myws + OFF_CNT), __ATOMIC_RELAXED, __HIP_MEMORY_SCOPE_AGENT)) / GRID + 1u;
  const int bid = blockIdx.x;
  STAMP(0);
  if (bid < NCOPY) { copy_role(p, call, bid); STAMP(1); }

  const int tid = threadIdx.x, wid = rfl(tid >> 6), lane = tid & 63, wr = wid >> 2, wc = wid & 3, fr = lane & 15, fq = lane >> 4;
  const unsigned base = call * 64u;
  const unsigned par = OFF_DATA + (call & 1u) * PAR_BYTES;
  const rsrc_t rs = mk_rsrc(myws + par, PAR_BYTES);
  const rsrc_t ro = mk_rsrc(p.out, (unsigned)M * N * 4u);
  const unsigned voff = lane * 16;
  const unsigned ldsw = (unsigned)wid * 2048u;
  const unsigned xw = (unsigned)wid * XREG, ww = XALL + (unsigned)wid * WREG;
  const float cz = p.sx[0] * p.sw[0] * LOG2E;
#define STAGE(bufoff, soff) do { \
    __builtin_amdgcn_raw_ptr_buffer_load_lds(rs, (LAS void*)(lds + (bufoff) + ldsw), 16, voff, (soff), 0, 17); \
    __builtin_amdgcn_raw_ptr_buffer_load_lds(rs, (LAS void*)(lds + (bufoff) + ldsw), 16, voff, (soff), 1024, 17); } while (0)

  if (bid >= NCOPY) {
    const int gidx = bid - NCOPY, xcd = gidx & 7, idx = gidx >> 3, tl = (xcd >> 1) * 30 + idx;
    const int pm = (xcd & 1) * 8 + (tl & 7), slot = tl >> 3;
    const int aoff = fq * 2048 + wr * 256 + fr * 16, boff = fq * 2048 + wc * 512 + fr * 16;
#define SA(b, h) (((b) * 2 + (h)) * HTB)
#define SB(b, h) ((4 + (b) * 2 + (h)) * HTB)
#define LDA(dst, b, h) do { _Pragma("unroll") for (int m = 0; m < 4; ++m) _Pragma("unroll") for (int k = 0; k < 2; ++k) dst[m][k] = *(const LAS i32x4*)(lds + SA(b, h) + aoff + m * 512 + k * 8192); } while (0)
#define LDB(dst, b, h) do { _Pragma("unroll") for (int n = 0; n < 2; ++n) _Pragma("unroll") for (int k = 0; k < 2; ++k) dst[n][k] = *(const LAS i32x4*)(lds + SB(b, h) + boff + n * 256 + k * 8192); } while (0)
#define MMA(ai, bj, At, Bt) do { __builtin_amdgcn_s_setprio(1); _Pragma("unroll") for (int m = 0; m < 4; ++m) _Pragma("unroll") for (int n = 0; n < 2; ++n) \
        acc[ai][bj][m][n] = __builtin_amdgcn_mfma_scale_f32_16x16x128_f8f6f4(cat(Bt[n][0], Bt[n][1]), cat(At[m][0], At[m][1]), acc[ai][bj][m][n], 0, 0, 0, 0, 0, 0); __builtin_amdgcn_s_setprio(0); } while (0)
    f32x4 acc[2][2][4][2];
#pragma unroll
    for (int a = 0; a < 2; ++a)
#pragma unroll
      for (int b = 0; b < 2; ++b)
#pragma unroll
        for (int m = 0; m < 4; ++m)
#pragma unroll
          for (int n = 0; n < 2; ++n) acc[a][b][m][n] = (f32x4){0.f, 0.f, 0.f, 0.f};
    i32x4 At[4][2], B0[2][2], B1[2][2];
    const unsigned kstep = 4096, hstep = 2048, tstep = NT * 4096;
    const unsigned cB = xw + (unsigned)pm * tstep;
    unsigned cA = ww + (unsigned)slot * tstep;
    const unsigned foff = flag_off(wid, pm, slot, lane);
    unsigned fval = spin_flags(myws, foff, flag_thr(0, 0, base, lane), call), fnew = fval;
    STAMP(1);
    STAGE(SB(0, 0), cB); STAGE(SA(0, 0), cA); STAGE(SB(0, 1), cB + hstep); STAGE(SA(0, 1), cA + hstep);
    if (wr == 1) BAR;
    WAIT_V(4); BAR;
    STAGE(SB(1, 0), cB + kstep); STAGE(SA(1, 0), cA + kstep); STAGE(SB(1, 1), cB + hstep + kstep);
    WAIT_V(6); BAR;
    for (int r = 0; r < NROUND; ++r) {
      const bool has_next = r + 1 < NROUND;
      const unsigned nA = has_next ? cA + NSLOT * tstep : cA;
      for (int t = 0; t < NT; t += 2) {
        const bool last = (t == NT - 2);
        const unsigned a1 = cA + (unsigned)(t + 1) * kstep;
        const unsigned a2 = last ? nA : cA + (unsigned)(t + 2) * kstep, b2 = last ? cB : cB + (unsigned)(t + 2) * kstep;
        const unsigned a3 = a2 + kstep, b3 = b2 + kstep;
        const int rq = last ? (has_next ? r + 1 : r) : r, cq = last ? 0 : ((t + 2) >> 2);
        const unsigned thr = flag_thr(rq, cq, base, lane);
        LDB(B0, 0, 0); SCHED; LDA(At, 0, 0); STAGE(SA(1, 1), a1 + hstep);
        WAIT_L(8); BAR; WAIT_L(0); MMA(0, 0, At, B0); BAR; SCHED;
        if (!__all((int)(fval - thr) >= 0)) fval = spin_flags(myws, foff, thr, call);
        LDB(B1, 0, 1); STAGE(SB(0, 0), b2);
        BAR; WAIT_L(0); MMA(0, 1, At, B1); BAR;
        LDA(At, 0, 1); STAGE(SA(0, 0), a2);
        BAR; WAIT_L(0); MMA(1, 0, At, B0); BAR; SCHED;
        STAGE(SB(0, 1), b2 + hstep);
        WAIT_V(6); asm volatile("" : "+v"(fnew));
        if ((int)(fnew - fval) > 0 || lane >= 8) fval = fnew;
        BAR; MMA(1, 1, At, B1); BAR;
        LDB(B0, 1, 0); SCHED; LDA(At, 1, 0); STAGE(SA(0, 1), a2 + hstep);
        WAIT_L(8); BAR; WAIT_L(0); MMA(0, 0, At, B0); BAR; SCHED;
        LDB(B1, 1, 1); STAGE(SB(1, 0), b3);
        asm volatile("global_load_dword %0, %1, %2 sc0 sc1" : "=v"(fnew) : "v"(foff), "s"(myws) : "memory");
        BAR; WAIT_L(0); MMA(0, 1, At, B1); BAR;
        LDA(At, 1, 1); STAGE(SA(1, 0), a3);
        BAR; WAIT_L(0); MMA(1, 0, At, B0); BAR; SCHED;
        STAGE(SB(1, 1), b3 + hstep);
        WAIT_V(7); BAR; MMA(1, 1, At, B1); BAR;
      }
      STAMP(2 + 2 * r);
      {
        int tz = threadIdx.x; asm volatile("" : "+v"(tz));
        const int lane2 = tz & 63, fr2 = lane2 & 15, fq2 = lane2 >> 4;
        const int pn = NSLOT * r + slot;
        const unsigned ovoff = (unsigned)(wc * 32 + fq2 * 4) * (N * 4) + (unsigned)(wr * 64 + fr2 * 4) * 4;
        const unsigned osoff = (unsigned)(pm * 256) * (N * 4) + (unsigned)pn * 1024;
        const bool bad = __any(lane2 >= 8 && fval == call);
        const float czz = bad ? __builtin_nanf("") : cz;
#pragma unroll
        for (int ai = 0; ai < 2; ++ai)
#pragma unroll
          for (int bj = 0; bj < 2; ++bj)
#pragma unroll
            for (int n = 0; n < 2; ++n) {
              const f32x4 a4[4] = {acc[ai][bj][0][n], acc[ai][bj][1][n], acc[ai][bj][2][n], acc[ai][bj][3][n]};
              f32x4 o4[4];
              silu_rows(a4, czz, o4);
#pragma unroll
              for (int j = 0; j < 4; ++j) store_out(o4[j], ro, ovoff, osoff + (unsigned)(bj * 128 + n * 16 + j) * (N * 4) + ai * 512);
            }
      }
      STAMP(3 + 2 * r);
      if (!has_next) break;
#pragma unroll
      for (int a = 0; a < 2; ++a)
#pragma unroll
        for (int b = 0; b < 2; ++b)
#pragma unroll
          for (int m = 0; m < 4; ++m)
#pragma unroll
            for (int n = 0; n < 2; ++n) acc[a][b][m][n] = (f32x4){0.f, 0.f, 0.f, 0.f};
      cA = nA;
    }
    WAIT_V(0); asm volatile("" : "+v"(fnew));
    if (wr == 0) BAR;
    BAR;
  }
  {
    const int xcd = bid & 7, idx = bid >> 3;
    const int tp = xcd >> 1, rb = (xcd & 1) * 32 + idx;
    const int pn = NSLOT * NROUND + tp, pm = rb >> 2;
    const int cg = wid & 3, rg = wid >> 2;
    constexpr int STG = 40960;
    const unsigned cA = ww + (unsigned)pn * (NT * 4096);
    const unsigned cB = xw + (unsigned)pm * (NT * 4096) + ((rb >> 1) & 1) * 2048 + (rb & 1) * 1024;
    const int aoff = (cg >> 1) * 16384 + fq * 2048 + (cg & 1) * 256 + fr * 16;
    const int boff = 32768 + fq * 1024 + rg * 512 + fr * 16;
#define TSTAGE(buf, s) do { \
      STAGE((buf) * STG, cA + (s) * 4096); STAGE((buf) * STG + 16384, cA + (s) * 4096 + 2048); \
      __builtin_amdgcn_raw_ptr_buffer_load_lds(rs, (LAS void*)(lds + (buf) * STG + 32768 + wid * 1024), 16, voff, cB + (s) * 4096, 0, 17); } while (0)
    f32x4 acc[4][2];
#pragma unroll
    for (int m = 0; m < 4; ++m)
#pragma unroll
      for (int n = 0; n < 2; ++n) acc[m][n] = (f32x4){0.f, 0.f, 0.f, 0.f};
    (void)spin_flags(myws, flag_off(wid, pm, NSLOT, lane), lane < 8 ? base + ((lane & 4) == 0 ? 15u : 16u + 8u * tp) : 0u, call);
    STAMP(10);
    TSTAGE(0, 0); TSTAGE(1, 1);
    int buf = 0;
    for (int s = 0; s < NT; ++s) {
      if (s + 1 < NT) WAIT_V(5); else WAIT_V(0);
      BAR;
      const int nb = buf == 0 ? 2 : buf - 1;
      if (s + 2 < NT) TSTAGE(nb, s + 2);
      i32x4 At[4][2], Bt[2][2];
#pragma unroll
      for (int n = 0; n < 2; ++n)
#pragma unroll
        for (int k = 0; k < 2; ++k) Bt[n][k] = *(const LAS i32x4*)(lds + buf * STG + boff + n * 256 + k * 4096);
#pragma unroll
      for (int m = 0; m < 4; ++m)
#pragma unroll
        for (int k = 0; k < 2; ++k) At[m][k] = *(const LAS i32x4*)(lds + buf * STG + aoff + m * 512 + k * 8192);
#pragma unroll
      for (int m = 0; m < 4; ++m)
#pragma unroll
        for (int n = 0; n < 2; ++n)
          acc[m][n] = __builtin_amdgcn_mfma_scale_f32_16x16x128_f8f6f4(cat(Bt[n][0], Bt[n][1]), cat(At[m][0], At[m][1]), acc[m][n], 0, 0, 0, 0, 0, 0);
      buf = buf == 2 ? 0 : buf + 1;
    }
    const bool bad = rfl(__hip_atomic_load((gu32*)(myws + OFF_TMO), __ATOMIC_RELAXED, __HIP_MEMORY_SCOPE_AGENT)) == call;
    const float czz = bad ? __builtin_nanf("") : cz;
    const unsigned ovoff = (unsigned)(rg * 32 + fq * 4) * (N * 4) + (unsigned)(cg * 64 + fr * 4) * 4;
    const unsigned osoff = (unsigned)(rb * 64) * (N * 4) + (unsigned)pn * 1024;
#pragma unroll
    for (int n = 0; n < 2; ++n) {
      const f32x4 a4[4] = {acc[0][n], acc[1][n], acc[2][n], acc[3][n]};
      f32x4 o4[4];
      silu_rows(a4, czz, o4);
#pragma unroll
      for (int j = 0; j < 4; ++j) store_out(o4[j], ro, ovoff, osoff + (unsigned)(n * 16 + j) * (N * 4));
    }
  }
  STAMP(11);
  if (threadIdx.x == 0) __hip_atomic_fetch_add((gu32*)(myws + OFF_CNT), 1u, __ATOMIC_RELAXED, __HIP_MEMORY_SCOPE_AGENT);
}
}
#define HIP_CHECK(expr) do { hipError_t _e = (expr); if (_e != hipSuccess) { \
    fprintf(stderr, "HIP error %s at %s:%d\n", hipGetErrorString(_e), __FILE__, __LINE__); \
    return; } } while (0)

extern "C" void kernel_launch(int rank, int nranks, ncclComm_t comm, hipStream_t stream,
                              void* const* d_in, const int* in_sizes, int n_in,
                              void* d_out, int out_size, const int* in_dtypes, int out_dtype,
                              void* const* d_out_all, void* const* d_ws_all, size_t ws_bytes) {
  (void)comm; (void)in_dtypes; (void)out_dtype; (void)d_out_all;
  if (nranks != gar::NR || n_in != 4 || (unsigned)rank >= (unsigned)gar::NR || ws_bytes < gar::WS_NEED ||
      in_sizes[0] != gar::M * gar::KS || in_sizes[1] != gar::N * gar::KS || out_size != gar::M * gar::N) {
    fprintf(stderr, "kernel_launch: unexpected shapes\n");
    return;
  }
  static bool attr_set = false;
  if (!attr_set) {
    HIP_CHECK(hipFuncSetAttribute((const void*)gar::gemm_ag_kernel, hipFuncAttributeMaxDynamicSharedMemorySize, gar::LDS_BYTES));
    attr_set = true;
  }
  gar::Params p;
  p.x = (const gar::u8*)d_in[0]; p.w = (const gar::u8*)d_in[1];
  p.sx = (const float*)d_in[2]; p.sw = (const float*)d_in[3];
  p.out = (float*)d_out;
  for (int i = 0; i < gar::NR; ++i) p.ws[i] = (gar::u8*)d_ws_all[i];
  p.rank = rank; p.pad = 0;
  gar::gemm_ag_kernel<<<gar::GRID, 512, gar::LDS_BYTES, stream>>>(p);
  HIP_CHECK(hipGetLastError());
}
```

```cpp
#include <hip/hip_runtime.h>
#include <rccl/rccl.h>
#include <cstdio>

extern "C" __device__ float gar_fmul_legacy(float, float) __asm("llvm.amdgcn.fmul.legacy");
namespace gar {
__device__ __forceinline__ float fmul_single(float a, float b) { return gar_fmul_legacy(a, b); }
#define LAS __attribute__((address_space(3)))
typedef int i32x4 __attribute__((ext_vector_type(4)));
typedef int i32x8 __attribute__((ext_vector_type(8)));
typedef unsigned u32x4 __attribute__((ext_vector_type(4)));
typedef float f32x4 __attribute__((ext_vector_type(4)));
typedef float f32x2 __attribute__((ext_vector_type(2)));
typedef unsigned char u8;
typedef __amdgpu_buffer_rsrc_t rsrc_t;
typedef __attribute__((address_space(1))) unsigned gu32;

constexpr int M = 4096, N = 16384, NR = 8, KS = 512;
constexpr int NT = 32;
constexpr int HTB = 16384;
constexpr int LDS_BYTES = 8 * HTB;
constexpr int GRID = 256, NCOPY = 16, NSLOT = 15, NROUND = 4, NSTEP = 42, NSTEP_A = 10;
constexpr unsigned OFF_CNT = 0, OFF_TMO = 256;
constexpr unsigned OFF_FLAGS = 65536;
constexpr unsigned OFF_DATA = 1u << 20;
constexpr unsigned XREG = (unsigned)M * KS;
constexpr unsigned WREG = (unsigned)N * KS;
constexpr unsigned XALL = NR * XREG, WALL = NR * WREG;
constexpr unsigned PAR_BYTES = XALL + WALL;
constexpr size_t WS_NEED = (size_t)OFF_DATA + 2 * (size_t)PAR_BYTES;
constexpr unsigned long long DEADLINE = 30000000ull;
constexpr int RSRC3 = 0x00020000;
#define SLEEPV 8
#define KPRE 8

struct Params {
  const u8* x; const u8* w; const float* sx; const float* sw; float* out;
  u8* ws[NR];
  int rank; int pad;
};

__device__ __forceinline__ i32x8 cat(i32x4 a, i32x4 b) { return __builtin_shufflevector(a, b, 0, 1, 2, 3, 4, 5, 6, 7); }
__device__ __forceinline__ unsigned rfl(unsigned v) { return __builtin_amdgcn_readfirstlane(v); }
__device__ __forceinline__ rsrc_t mk_rsrc(const void* p, unsigned bytes) {
  const unsigned long long a = (unsigned long long)p;
  const unsigned lo = rfl((unsigned)a), hi = rfl((unsigned)(a >> 32));
  return __builtin_amdgcn_make_buffer_rsrc((void*)(((unsigned long long)hi << 32) | lo), (short)0, (int)bytes, RSRC3);
}
#define STR2(x) #x
#define STR(x) STR2(x)
#define WAIT_V(n) asm volatile("s_waitcnt vmcnt(" STR(n) ")" ::: "memory")
#define WAIT_L(n) asm volatile("s_waitcnt lgkmcnt(" STR(n) ")" ::: "memory")
#define BAR __builtin_amdgcn_s_barrier()
#define SCHED __builtin_amdgcn_sched_barrier(0)
#define STAMP(i)
#define CSTAMP(e)

#define PIN16(d) asm volatile("" : "+v"(d[0]), "+v"(d[1]), "+v"(d[2]), "+v"(d[3]), "+v"(d[4]), "+v"(d[5]), "+v"(d[6]), "+v"(d[7]), \
                                   "+v"(d[8]), "+v"(d[9]), "+v"(d[10]), "+v"(d[11]), "+v"(d[12]), "+v"(d[13]), "+v"(d[14]), "+v"(d[15]))
#define PIN4(x) asm volatile("" : "+v"(x[0]), "+v"(x[1]), "+v"(x[2]), "+v"(x[3]))
__device__ __forceinline__ void copy_role(const Params& p, unsigned call, int j) {
  const int tid = threadIdx.x, g = rfl(tid >> 6), lane = tid & 63, ks = lane >> 4, fr = lane & 15;
  const int nstep = j < NSLOT ? NSTEP : NSTEP_A;
  if (g >= 4) { for (int e = 0; e < nstep; ++e) BAR; return; }
  const unsigned rank = (unsigned)p.rank;
  const unsigned par = OFF_DATA + (call & 1u) * PAR_BYTES;
  u8* dst[NR];
#pragma unroll
  for (int d = 0; d < NR; ++d) { const unsigned long long a = (unsigned long long)p.ws[(rank + 1 + d) & 7]; dst[d] = (u8*)(((unsigned long long)rfl((unsigned)(a >> 32)) << 32) | rfl((unsigned)a)); }
  const rsrc_t srcx = mk_rsrc(p.x, (unsigned)M * KS), srcw = mk_rsrc(p.w, (unsigned)N * KS);
  gu32* flagp = (gu32*)(p.ws[lane & 7] + OFF_FLAGS + ((rank * 16 + j) * 4 + g) * 128);
  const unsigned base = call * 64u;
  const unsigned xvo = (g * 64 + fr) * KS + ks * 16, wvo = ks * 16 * N + g * 64 + fr * 4;
  const unsigned xdo = par + rank * XREG + (j * NT + ks) * 4096 + (g >> 1) * 2048 + (g & 1) * 1024 + fr * 16;
  const unsigned wdo = par + XALL + rank * WREG + ks * 4096 + (g >> 1) * 2048 + (g & 1) * 256 + fr * 16;
  unsigned d[16]; u32x4 xr[4], o[4];
#define XLOAD(c) do { _Pragma("unroll") for (int t = 0; t < 4; ++t) \
    asm volatile("buffer_load_dwordx4 %0, %1, %2, %3 offen" : "=v"(xr[t]) : "v"(xvo), "s"(srcx), "s"(j * 256 * KS + (c) * 64 + t * 16 * KS) : "memory"); } while (0)
#define WLOAD(pn, c) do { _Pragma("unroll") for (int i = 0; i < 16; ++i) \
    asm volatile("buffer_load_dword %0, %1, %2, %3 offen" : "=v"(d[i]) : "v"(wvo), "s"(srcw), "s"((c) * 64 * N + (pn) * 256 + i * N) : "memory"); } while (0)
#define WTRANS() do { _Pragma("unroll") for (int q = 0; q < 4; ++q) { \
    const unsigned t0 = __builtin_amdgcn_perm(d[4 * q + 1], d[4 * q], 0x05010400u), t1 = __builtin_amdgcn_perm(d[4 * q + 1], d[4 * q], 0x07030602u); \
    const unsigned t2 = __builtin_amdgcn_perm(d[4 * q + 3], d[4 * q + 2], 0x05010400u), t3 = __builtin_amdgcn_perm(d[4 * q + 3], d[4 * q + 2], 0x07030602u); \
    o[0][q] = __builtin_amdgcn_perm(t2, t0, 0x05040100u); o[1][q] = __builtin_amdgcn_perm(t2, t0, 0x07060302u); \
    o[2][q] = __builtin_amdgcn_perm(t3, t1, 0x05040100u); o[3][q] = __builtin_amdgcn_perm(t3, t1, 0x07060302u); } PIN4(o); } while (0)
#define PUSH1(q, voff, step) asm volatile("global_store_dwordx4 %0, %1, %2 sc0 sc1\n\ts_nop 1" :: "v"((voff) + ((q) >> 3) * (step)), "v"(o[(q) >> 3]), "s"(dst[(q) & 7]) : "memory")
#define STEP(e, voff, step, LOADS) do { \
    _Pragma("unroll") for (int q = 0; q < KPRE; ++q) PUSH1(q, voff, step); \
    WAIT_V(KPRE); \
    if ((e) >= 1) { if (lane < NR) __hip_atomic_store(flagp, base + (unsigned)(e), __ATOMIC_RELAXED, __HIP_MEMORY_SCOPE_SYSTEM); CSTAMP((e) - 1); } \
    LOADS; \
    _Pragma("unroll") for (int q = KPRE; q < 32; ++q) PUSH1(q, voff, step); \
    BAR; } while (0)
#define WAIT_LOADS() WAIT_V(32 - KPRE)
  const int tpn = NSLOT * NROUND + (j & 3), tc = j >> 2;
  WLOAD(tpn, tc);
  WAIT_V(0);
  for (int h = 0; h < 2; ++h) {
    WAIT_LOADS(); PIN16(d);
    WTRANS();
    { const unsigned voff = wdo + tpn * 131072 + (tc + 4 * h) * 16384; STEP(5 * h, voff, 512, XLOAD(4 * h)); }
    for (int c = 4 * h; c < 4 * h + 3; ++c) {
      WAIT_LOADS(); PIN4(xr);
#pragma unroll
      for (int t = 0; t < 4; ++t) o[t] = xr[t];
      PIN4(o);
      const unsigned voff = xdo + c * 16384; STEP(c + h + 1, voff, 256, XLOAD(c + 1));
    }
    WAIT_LOADS(); PIN4(xr);
#pragma unroll
    for (int t = 0; t < 4; ++t) o[t] = xr[t];
    PIN4(o);
    const int pnn = h == 0 ? tpn : (j < NSLOT ? j : tpn), cn = h == 0 ? tc + 4 : 0;
    const unsigned voff = xdo + (4 * h + 3) * 16384; STEP(5 * h + 4, voff, 256, WLOAD(pnn, cn));
  }
  for (int e = NSTEP_A; e < nstep; ++e) {
    const int r = (e - NSTEP_A) >> 3, c = (e - NSTEP_A) & 7;
    const int pn = NSLOT * r + j;
    const int en = e + 1 < nstep ? e + 1 : e, rn = (en - NSTEP_A) >> 3, cn = (en - NSTEP_A) & 7;
    WAIT_LOADS(); PIN16(d);
    WTRANS();
    const unsigned voff = wdo + pn * 131072 + c * 16384;
    STEP(e, voff, 512, WLOAD(NSLOT * rn + j, cn));
  }
  WAIT_V(0); PIN16(d);
  if (lane < NR) __hip_atomic_store(flagp, base + (unsigned)nstep, __ATOMIC_RELAXED, __HIP_MEMORY_SCOPE_SYSTEM);
  CSTAMP(nstep - 1);
}

__device__ __forceinline__ unsigned flag_off(int src, int pm, int jw, int lane) {
  return lane < 8 ? OFF_FLAGS + ((src * 16 + ((lane & 4) == 0 ? pm : jw)) * 4 + (lane & 3)) * 128 : OFF_TMO;
}
__device__ __noinline__ unsigned spin_flags(u8* ws, unsigned off, unsigned thr, unsigned call) {
  gu32* tmo = (gu32*)(ws + OFF_TMO);
  gu32* fp = (gu32*)(ws + off);
  const unsigned long long t0 = __builtin_amdgcn_s_memrealtime();
  for (;;) {
    const unsigned v = __hip_atomic_load(fp, __ATOMIC_RELAXED, __HIP_MEMORY_SCOPE_SYSTEM);
    if (__all((int)(v - thr) >= 0)) return v;
    if (rfl(__hip_atomic_load(tmo, __ATOMIC_RELAXED, __HIP_MEMORY_SCOPE_AGENT)) == call || __builtin_amdgcn_s_memrealtime() - t0 > DEADLINE) {
      __hip_atomic_store(tmo, call, __ATOMIC_RELAXED, __HIP_MEMORY_SCOPE_AGENT);
      WAIT_V(0);
      return (threadIdx.x & 63u) >= 8u ? call : thr;
    }
    __builtin_amdgcn_s_sleep(SLEEPV);
  }
}

__device__ __forceinline__ void store_out(f32x4 v, rsrc_t r, unsigned voff, unsigned soff) {
#define OUTPOL "nt"
  asm volatile("buffer_store_dwordx4 %0, %1, %2, %3 offen " OUTPOL "\n\ts_nop 1" :: "v"(v), "v"(voff), "s"(r), "s"(soff) : "memory");
}
constexpr float LOG2E = 1.4426950408889634f, CLIP2 = 86.56170245333781f;
__device__ __forceinline__ void silu_rows(const f32x4 (&a)[4], float cz, f32x4 (&o)[4]) {
#pragma unroll
  for (int h = 0; h < 2; ++h) {
    f32x2 z[4], r[4];
#pragma unroll
    for (int m = 0; m < 4; ++m) {
      z[m] = (f32x2){a[m][2 * h], a[m][2 * h + 1]} * cz;
      f32x2 e;
      e.x = __builtin_amdgcn_exp2f(__builtin_amdgcn_fmed3f(-z[m].x, -CLIP2, CLIP2));
      e.y = __builtin_amdgcn_exp2f(__builtin_amdgcn_fmed3f(-z[m].y, -CLIP2, CLIP2));
      const f32x2 dd = __builtin_elementwise_fma(e, (f32x2){LOG2E, LOG2E}, (f32x2){LOG2E, LOG2E});
      r[m].x = __builtin_amdgcn_rcpf(dd.x); r[m].y = __builtin_amdgcn_rcpf(dd.y);
    }
#pragma unroll
    for (int m = 0; m < 4; ++m) {
      o[2 * h][m] = fmul_single(z[m].x, r[m].x); o[2 * h + 1][m] = fmul_single(z[m].y, r[m].y); }
  }
}

__global__ void __launch_bounds__(512, 2) gemm_ag_kernel(Params p) {
  extern __shared__ __attribute__((aligned(16))) unsigned char shm[];
  LAS unsigned char* lds = (LAS unsigned char*)shm;
  u8* myws = p.ws[p.rank];
  const unsigned call = rfl(__hip_atomic_load((gu32*)(myws + OFF_CNT), __ATOMIC_RELAXED, __HIP_MEMORY_SCOPE_AGENT)) / GRID + 1u;
  const int bid = blockIdx.x;
  STAMP(0);
  if (bid < NCOPY) { copy_role(p, call, bid); STAMP(1); }

  const int tid = threadIdx.x, wid = rfl(tid >> 6), lane = tid & 63, wr = wid >> 2, wc = wid & 3, fr = lane & 15, fq = lane >> 4;
  const unsigned base = call * 64u;
  const unsigned par = OFF_DATA + (call & 1u) * PAR_BYTES;
  const rsrc_t rs = mk_rsrc(myws + par, PAR_BYTES);
  const rsrc_t ro = mk_rsrc(p.out, (unsigned)M * N * 4u);
  const unsigned voff = lane * 16;
  const unsigned ldsw = (unsigned)wid * 2048u;
  const unsigned xw = (unsigned)wid * XREG, ww = XALL + (unsigned)wid * WREG;
  const float cz = p.sx[0] * p.sw[0] * LOG2E;
#define STAGE(bufoff, soff) do { \
    __builtin_amdgcn_raw_ptr_buffer_load_lds(rs, (LAS void*)(lds + (bufoff) + ldsw), 16, voff, (soff), 0, 17); \
    __builtin_amdgcn_raw_ptr_buffer_load_lds(rs, (LAS void*)(lds + (bufoff) + ldsw), 16, voff, (soff), 1024, 17); } while (0)

  {
    const int xcd = bid & 7, idx = bid >> 3;
    const int tp = xcd >> 1, rb = (xcd & 1) * 32 + idx;
    const int pn = NSLOT * NROUND + tp, pm = rb >> 2;
    const int cg = wid & 3, rg = wid >> 2;
    constexpr int STG = 40960;
    const unsigned cA = ww + (unsigned)pn * (NT * 4096);
    const unsigned cB = xw + (unsigned)pm * (NT * 4096) + ((rb >> 1) & 1) * 2048 + (rb & 1) * 1024;
    const int aoff = (cg >> 1) * 16384 + fq * 2048 + (cg & 1) * 256 + fr * 16;
    const int boff = 32768 + fq * 1024 + rg * 512 + fr * 16;
#define TSTAGE(buf, s) do { \
      STAGE((buf) * STG, cA + (s) * 4096); STAGE((buf) * STG + 16384, cA + (s) * 4096 + 2048); \
      __builtin_amdgcn_raw_ptr_buffer_load_lds(rs, (LAS void*)(lds + (buf) * STG + 32768 + wid * 1024), 16, voff, cB + (s) * 4096, 0, 17); } while (0)
    f32x4 acc[4][2];
#pragma unroll
    for (int m = 0; m < 4; ++m)
#pragma unroll
      for (int n = 0; n < 2; ++n) acc[m][n] = (f32x4){0.f, 0.f, 0.f, 0.f};
    const unsigned toff = lane < 16 ? OFF_FLAGS + ((wid * 16 + (((lane >> 2) << 2) | tp)) * 4 + (lane & 3)) * 128
                        : lane == 16 ? OFF_FLAGS + ((wid * 16 + pm) * 4 + (rb & 3)) * 128 : OFF_TMO;
    const unsigned tall = lane < 16 ? base + 6u : lane == 16 ? base + 10u : 0u;
    const bool ready = __all((int)(__hip_atomic_load((gu32*)(myws + toff), __ATOMIC_RELAXED, __HIP_MEMORY_SCOPE_SYSTEM) - tall) >= 0);
#define TCHECK(c) do { if (!ready) (void)spin_flags(myws, toff, lane < 16 ? base + ((c) < 4 ? 1u : 6u) : lane == 16 ? base + (unsigned)((c) < 4 ? (c) + 2 : (c) + 3) : 0u, call); } while (0)
    TCHECK(0);
    TSTAGE(0, 0); TSTAGE(1, 1);
    int buf = 0;
    for (int s = 0; s < NT; ++s) {
      if (s + 1 < NT) WAIT_V(5); else WAIT_V(0);
      BAR;
      const int nb = buf == 0 ? 2 : buf - 1;
      if (s + 2 < NT) { if (((s + 2) & 3) == 0) TCHECK((s + 2) >> 2); TSTAGE(nb, s + 2); }
      i32x4 At[4][2], Bt[2][2];
#pragma unroll
      for (int n = 0; n < 2; ++n)
#pragma unroll
        for (int k = 0; k < 2; ++k) Bt[n][k] = *(const LAS i32x4*)(lds + buf * STG + boff + n * 256 + k * 4096);
#pragma unroll
      for (int m = 0; m < 4; ++m)
#pragma unroll
        for (int k = 0; k < 2; ++k) At[m][k] = *(const LAS i32x4*)(lds + buf * STG + aoff + m * 512 + k * 8192);
#pragma unroll
      for (int m = 0; m < 4; ++m)
#pragma unroll
        for (int n = 0; n < 2; ++n)
          acc[m][n] = __builtin_amdgcn_mfma_scale_f32_16x16x128_f8f6f4(cat(Bt[n][0], Bt[n][1]), cat(At[m][0], At[m][1]), acc[m][n], 0, 0, 0, 0, 0, 0);
      buf = buf == 2 ? 0 : buf + 1;
    }
    const bool bad = rfl(__hip_atomic_load((gu32*)(myws + OFF_TMO), __ATOMIC_RELAXED, __HIP_MEMORY_SCOPE_AGENT)) == call;
    const float czz = bad ? __builtin_nanf("") : cz;
    const unsigned ovoff = (unsigned)(rg * 32 + fq * 4) * (N * 4) + (unsigned)(cg * 64 + fr * 4) * 4;
    const unsigned osoff = (unsigned)(rb * 64) * (N * 4) + (unsigned)pn * 1024;
#pragma unroll
    for (int n = 0; n < 2; ++n) {
      const f32x4 a4[4] = {acc[0][n], acc[1][n], acc[2][n], acc[3][n]};
      f32x4 o4[4];
      silu_rows(a4, czz, o4);
#pragma unroll
      for (int j = 0; j < 4; ++j) store_out(o4[j], ro, ovoff, osoff + (unsigned)(n * 16 + j) * (N * 4));
    }
      STAMP(10);
    BAR;
  }
  if (bid >= NCOPY) {
    const int gidx = bid - NCOPY, xcd = gidx & 7, idx = gidx >> 3, tl = (xcd >> 1) * 30 + idx;
    const int pm = (xcd & 1) * 8 + (tl & 7), slot = tl >> 3;
    const int aoff = fq * 2048 + wr * 256 + fr * 16, boff = fq * 2048 + wc * 512 + fr * 16;
#define SA(b, h) (((b) * 2 + (h)) * HTB)
#define SB(b, h) ((4 + (b) * 2 + (h)) * HTB)
#define LDA(dst, b, h) do { _Pragma("unroll") for (int m = 0; m < 4; ++m) _Pragma("unroll") for (int k = 0; k < 2; ++k) dst[m][k] = *(const LAS i32x4*)(lds + SA(b, h) + aoff + m * 512 + k * 8192); } while (0)
#define LDB(dst, b, h) do { _Pragma("unroll") for (int n = 0; n < 2; ++n) _Pragma("unroll") for (int k = 0; k < 2; ++k) dst[n][k] = *(const LAS i32x4*)(lds + SB(b, h) + boff + n * 256 + k * 8192); } while (0)
#define SNAKE 1
#define MIDX(q) (SNAKE == 1 ? (((q) >> 2) ? 3 - ((q) & 3) : ((q) & 3)) : SNAKE == 2 ? ((q) >> 1) : ((q) >> 1))
#define NIDX(q) (SNAKE == 1 ? ((q) >> 2) : SNAKE == 2 ? ((((q) >> 1) & 1) ? 1 - ((q) & 1) : ((q) & 1)) : ((q) & 1))
#define MMA(ai, bj, At, Bt) do { __builtin_amdgcn_s_setprio(1); _Pragma("unroll") for (int q = 0; q < 8; ++q) { const int m = MIDX(q), n = NIDX(q); \
        acc[ai][bj][m][n] = __builtin_amdgcn_mfma_scale_f32_16x16x128_f8f6f4(cat(Bt[n][0], Bt[n][1]), cat(At[m][0], At[m][1]), acc[ai][bj][m][n], 0, 0, 0, 0, 0, 0); } __builtin_amdgcn_s_setprio(0); } while (0)
    f32x4 acc[2][2][4][2];
    i32x4 At[4][2], B0[2][2], B1[2][2];
    const unsigned kstep = 4096, hstep = 2048, tstep = NT * 4096;
    const unsigned cB = xw + (unsigned)pm * tstep;
    unsigned cA = ww + (unsigned)slot * tstep;
    const unsigned foff = flag_off(wid, pm, slot, lane);
    unsigned fval = spin_flags(myws, foff, lane < 4 ? base + 10u : lane < 8 ? base + 11u : 0u, call), fnew = fval;
    const unsigned fadd = lane >= 4 && lane < 8 ? 0u : 0x40000000u;
    unsigned need = base + 11u;
    STAMP(1);
    STAGE(SB(0, 0), cB); STAGE(SA(0, 0), cA); STAGE(SB(0, 1), cB + hstep); STAGE(SA(0, 1), cA + hstep);
    if (wr == 1) BAR;
    WAIT_V(4); BAR;
    STAGE(SB(1, 0), cB + kstep); STAGE(SA(1, 0), cA + kstep); STAGE(SB(1, 1), cB + hstep + kstep); STAGE(SA(1, 1), cA + hstep + kstep);
    WAIT_V(8); BAR;
#define MMA0(ai, bj, At, Bt) do { __builtin_amdgcn_s_setprio(1); _Pragma("unroll") for (int q = 0; q < 8; ++q) { const int m = MIDX(q), n = NIDX(q); \
        acc[ai][bj][m][n] = __builtin_amdgcn_mfma_scale_f32_16x16x128_f8f6f4(cat(Bt[n][0], Bt[n][1]), cat(At[m][0], At[m][1]), (f32x4){0.f, 0.f, 0.f, 0.f}, 0, 0, 0, 0, 0, 0); } __builtin_amdgcn_s_setprio(0); } while (0)
#define FCHECK() do { if (!__all(fval + fadd >= need)) fval = spin_flags(myws, foff, lane < 8 ? need - fadd : 0u, call); } while (0)
#define ITER(FIRST, MM, a1h, a2, b2, NEXT_NEED) do { \
        const unsigned a3 = (a2) + kstep, b3 = (b2) + kstep; \
          \
        LDB(B0, 0, 0); SCHED; LDA(At, 0, 0); if (!(FIRST)) STAGE(SA(1, 1), a1h); \
        WAIT_L(8); BAR; WAIT_L(0); MM(0, 0, At, B0); BAR; SCHED; \
          \
        FCHECK(); \
        LDB(B1, 0, 1); STAGE(SB(0, 0), b2); \
        BAR; WAIT_L(0); MM(0, 1, At, B1); BAR; \
          \
        LDA(At, 0, 1); STAGE(SA(0, 0), a2); \
        BAR; WAIT_L(0); MM(1, 0, At, B0); BAR; SCHED; \
          \
        STAGE(SB(0, 1), (b2) + hstep); \
        if ((FIRST) && r > 0) WAIT_V(38); else WAIT_V(6); \
        asm volatile("" : "+v"(fnew));         \
        fval = lane >= 8 ? fnew : (fnew > fval ? fnew : fval); \
        BAR; MM(1, 1, At, B1); BAR; \
          \
        LDB(B0, 1, 0); SCHED; LDA(At, 1, 0); STAGE(SA(0, 1), (a2) + hstep); \
        WAIT_L(8); BAR; WAIT_L(0); MMA(0, 0, At, B0); BAR; SCHED; \
          \
        LDB(B1, 1, 1); STAGE(SB(1, 0), b3); \
        asm volatile("global_load_dword %0, %1, %2 sc0 sc1" : "=v"(fnew) : "v"(foff), "s"(myws) : "memory"); \
        BAR; WAIT_L(0); MMA(0, 1, At, B1); BAR; \
          \
        LDA(At, 1, 1); STAGE(SA(1, 0), a3); \
        BAR; WAIT_L(0); MMA(1, 0, At, B0); BAR; SCHED; \
          \
        STAGE(SB(1, 1), b3 + hstep); \
        NEXT_NEED; \
        WAIT_V(7); BAR; MMA(1, 1, At, B1); BAR; } while (0)
    for (int r = 0; r < NROUND; ++r) {
      const bool has_next = r + 1 < NROUND;
      const unsigned nA = has_next ? cA + NSLOT * tstep : cA;
      ITER(true, MMA0, 0u, cA + 2 * kstep, cB + 2 * kstep, need += 1u);
      for (int t = 2; t < NT - 2; t += 2) {
        ITER(false, MMA, cA + (unsigned)(t + 1) * kstep + hstep, cA + (unsigned)(t + 2) * kstep, cB + (unsigned)(t + 2) * kstep, need += (t & 2) ? 0u : 1u);
      }
      need = has_next ? base + 11u + 8u * (unsigned)(r + 1) : 0u;
      ITER(false, MMA, cA + (unsigned)(NT - 1) * kstep + hstep, nA, cB, (void)0);
      STAGE(SA(1, 1), nA + kstep + hstep);
      STAMP(2 + 2 * r);
      {
        int tz = threadIdx.x; asm volatile("" : "+v"(tz));
        const int lane2 = tz & 63, fr2 = lane2 & 15, fq2 = lane2 >> 4;
        const int pn = NSLOT * r + slot;
        const unsigned ovoff = (unsigned)(wc * 32 + fq2 * 4) * (N * 4) + (unsigned)(wr * 64 + fr2 * 4) * 4;
        const unsigned osoff = (unsigned)(pm * 256) * (N * 4) + (unsigned)pn * 1024;
        const bool bad = __any(lane2 >= 8 && fval == call);
        const float czz = bad ? __builtin_nanf("") : cz;
#pragma unroll
        for (int ai = 0; ai < 2; ++ai)
#pragma unroll
          for (int bj = 0; bj < 2; ++bj)
#pragma unroll
            for (int n = 0; n < 2; ++n) {
              const f32x4 a4[4] = {acc[ai][bj][0][n], acc[ai][bj][1][n], acc[ai][bj][2][n], acc[ai][bj][3][n]};
              f32x4 o4[4];
              silu_rows(a4, czz, o4);
#pragma unroll
              for (int j = 0; j < 4; ++j) store_out(o4[j], ro, ovoff, osoff + (unsigned)(bj * 128 + n * 16 + j) * (N * 4) + ai * 512);
            }
      }
      STAMP(3 + 2 * r);
      cA = nA;
    }
    WAIT_V(0); asm volatile("" : "+v"(fnew));
    if (wr == 0) BAR;
  }
  STAMP(11);
  if (threadIdx.x == 0) __hip_atomic_fetch_add((gu32*)(myws + OFF_CNT), 1u, __ATOMIC_RELAXED, __HIP_MEMORY_SCOPE_AGENT);
}
}
#define HIP_CHECK(expr) do { hipError_t _e = (expr); if (_e != hipSuccess) { \
    fprintf(stderr, "HIP error %s at %s:%d\n", hipGetErrorString(_e), __FILE__, __LINE__); \
    return; } } while (0)

extern "C" void kernel_launch(int rank, int nranks, ncclComm_t comm, hipStream_t stream,
                              void* const* d_in, const int* in_sizes, int n_in,
                              void* d_out, int out_size, const int* in_dtypes, int out_dtype,
                              void* const* d_out_all, void* const* d_ws_all, size_t ws_bytes) {
  (void)comm; (void)in_dtypes; (void)out_dtype; (void)d_out_all;
  if (nranks != gar::NR || n_in != 4 || (unsigned)rank >= (unsigned)gar::NR || ws_bytes < gar::WS_NEED ||
      in_sizes[0] != gar::M * gar::KS || in_sizes[1] != gar::N * gar::KS || out_size != gar::M * gar::N) {
    fprintf(stderr, "kernel_launch: unexpected shapes\n");
    return;
  }
  static bool attr_set = false;
  if (!attr_set) {
    HIP_CHECK(hipFuncSetAttribute((const void*)gar::gemm_ag_kernel, hipFuncAttributeMaxDynamicSharedMemorySize, gar::LDS_BYTES));
    attr_set = true;
  }
  gar::Params p;
  p.x = (const gar::u8*)d_in[0]; p.w = (const gar::u8*)d_in[1];
  p.sx = (const float*)d_in[2]; p.sw = (const float*)d_in[3];
  p.out = (float*)d_out;
  for (int i = 0; i < gar::NR; ++i) p.ws[i] = (gar::u8*)d_ws_all[i];
  p.rank = rank; p.pad = 0;
  gar::gemm_ag_kernel<<<gar::GRID, 512, gar::LDS_BYTES, stream>>>(p);
  HIP_CHECK(hipGetLastError());
}
```

```cpp
#include <hip/hip_runtime.h>
#include <rccl/rccl.h>
#include <cstdio>

extern "C" __device__ float gar_fmul_legacy(float, float) __asm("llvm.amdgcn.fmul.legacy");
namespace gar {
__device__ __forceinline__ float fmul_single(float a, float b) { return gar_fmul_legacy(a, b); }
#define LAS __attribute__((address_space(3)))
typedef int i32x4 __attribute__((ext_vector_type(4)));
typedef int i32x8 __attribute__((ext_vector_type(8)));
typedef unsigned u32x4 __attribute__((ext_vector_type(4)));
typedef float f32x4 __attribute__((ext_vector_type(4)));
typedef float f32x2 __attribute__((ext_vector_type(2)));
typedef unsigned char u8;
typedef __amdgpu_buffer_rsrc_t rsrc_t;
typedef __attribute__((address_space(1))) unsigned gu32;

constexpr int M = 4096, N = 16384, NR = 8, KS = 512;
constexpr int NT = 32;
constexpr int HTB = 16384;
constexpr int LDS_E = 8 * HTB;
constexpr int LDS_BYTES = 8 * HTB + 16384;
constexpr int GRID = 256, NCOPY = 16, NSLOT = 15, NROUND = 4;
#define NCW 4
constexpr unsigned NCWAVE = NCOPY * NCW;
#define GAR_OUTS 1
constexpr unsigned NOPER = 2432, NUNIT0 = 992, NUNIT = NOPER + 512;
constexpr unsigned NLOCAL = NOPER + 128;
constexpr unsigned OFF_CNT = 0, OFF_TMO = 256, OFF_QUEUE = 512;
constexpr unsigned OFF_OFLAG = 4096, OFF_DONE = 8192;
constexpr unsigned OFF_FLAGS = 65536;
constexpr unsigned FLAG_SRC = 80 * 128, FLAG_ROUND = NSLOT * 128;
constexpr unsigned OFF_DATA = 1u << 20;
constexpr unsigned XREG = (unsigned)M * KS;
constexpr unsigned WREG = (unsigned)N * KS;
constexpr unsigned XALL = NR * XREG, WALL = NR * WREG;
constexpr unsigned PAR_BYTES = XALL + WALL;
constexpr size_t WS_NEED = (size_t)OFF_DATA + 2 * (size_t)PAR_BYTES;
constexpr unsigned long long DEADLINE = 30000000ull;
constexpr int RSRC3 = 0x00020000;
#define SLEEPV 8
#define OUTP 1
#define STAGN 2
#define KPRE 7

struct Params {
  const u8* x; const u8* w; const float* sx; const float* sw; float* out;
  u8* ws[NR];
  float* outs[NR];
  int rank; int pad;
};

__device__ __forceinline__ i32x8 cat(i32x4 a, i32x4 b) { return __builtin_shufflevector(a, b, 0, 1, 2, 3, 4, 5, 6, 7); }
__device__ __forceinline__ unsigned rfl(unsigned v) { return __builtin_amdgcn_readfirstlane(v); }
__device__ __forceinline__ rsrc_t mk_rsrc(const void* p, unsigned bytes) {
  const unsigned long long a = (unsigned long long)p;
  const unsigned lo = rfl((unsigned)a), hi = rfl((unsigned)(a >> 32));
  return __builtin_amdgcn_make_buffer_rsrc((void*)(((unsigned long long)hi << 32) | lo), (short)0, (int)bytes, RSRC3);
}
#define STR2(x) #x
#define STR(x) STR2(x)
#define WAIT_V(n) asm volatile("s_waitcnt vmcnt(" STR(n) ")" ::: "memory")
#define WAIT_L(n) asm volatile("s_waitcnt lgkmcnt(" STR(n) ")" ::: "memory")
#define BAR __builtin_amdgcn_s_barrier()
#define SCHED __builtin_amdgcn_sched_barrier(0)
#define STAMP(i)
#define CSTAMP(e, v)

__device__ __noinline__ void spin_flags(u8* ws, unsigned off, unsigned long long ign, unsigned call) {
  gu32* tmo = (gu32*)(ws + OFF_TMO);
  gu32* fp = (gu32*)(ws + off);
  const unsigned long long t0 = __builtin_amdgcn_s_memrealtime();
  for (;;) {
    const unsigned v = __hip_atomic_load(fp, __ATOMIC_RELAXED, __HIP_MEMORY_SCOPE_SYSTEM);
    if ((__ballot((int)(v - call) >= 0) | ign) == ~0ull) return;
    if (rfl(__hip_atomic_load(tmo, __ATOMIC_RELAXED, __HIP_MEMORY_SCOPE_AGENT)) == call || __builtin_amdgcn_s_memrealtime() - t0 > DEADLINE) {
      __hip_atomic_store(tmo, call, __ATOMIC_RELAXED, __HIP_MEMORY_SCOPE_AGENT);
      WAIT_V(0);
      return;
    }
    __builtin_amdgcn_s_sleep(SLEEPV);
  }
}

struct Unit { unsigned src, dst, flag; bool isx; };
__device__ __forceinline__ Unit decode_unit(unsigned u) {
  unsigned panel, h, lev; bool isx = false;
  if (u < NUNIT0) {
    lev = u / 62u; const unsigned jj = u - lev * 62u;
    h = jj & 1u; isx = jj < 32u;
    panel = isx ? jj >> 1 : (jj - 32u) >> 1;
  } else if (u < NOPER) {
    const unsigned v = u - NUNIT0, r1 = v / 480u, w = v - r1 * 480u;
    lev = w / 30u; const unsigned jj = w - lev * 30u;
    h = jj & 1u; panel = NSLOT * (r1 + 1u) + (jj >> 1);
  } else {
    const unsigned v = u - NOPER;
    lev = v & 15u; h = (v >> 4) & 1u; panel = NSLOT * NROUND + (v >> 5);
  }
  Unit o; o.isx = isx;
  o.src = isx ? (panel * 256u + h * 128u) * KS + lev * 32u : lev * 32u * N + panel * 256u + h * 128u;
  o.dst = (isx ? 0u : XALL) + panel * (NT * 4096u) + lev * 8192u + h * 2048u;
  o.flag = (((isx ? 0u : 16u) + panel) * 2u + h) * 64u + lev * 4u;
  return o;
}
__device__ __forceinline__ void unit_load(const Unit& u, rsrc_t srcx, rsrc_t srcw, int lane, unsigned (&d)[16]) {
  if (u.isx) {
#pragma unroll
    for (int t = 0; t < 4; ++t) { const u32x4 v = __builtin_amdgcn_raw_buffer_load_b128(srcx, (lane & 31) * KS + (lane >> 5) * 16, u.src + t * 32 * KS, 0);
      d[4 * t] = v[0]; d[4 * t + 1] = v[1]; d[4 * t + 2] = v[2]; d[4 * t + 3] = v[3]; }
  } else {
#pragma unroll
    for (int i = 0; i < 16; ++i) d[i] = __builtin_amdgcn_raw_buffer_load_b32(srcw, (lane >> 5) * 16 * N + (lane & 31) * 4, u.src + i * N, 0);
  }
}
__device__ __forceinline__ void unit_retile(const Unit& u, const unsigned (&d)[16], u32x4 (&o)[4]) {
  if (u.isx) {
#pragma unroll
    for (int t = 0; t < 4; ++t) o[t] = (u32x4){d[4 * t], d[4 * t + 1], d[4 * t + 2], d[4 * t + 3]};
  } else {
#pragma unroll
    for (int q = 0; q < 4; ++q) {
      const unsigned t0 = __builtin_amdgcn_perm(d[4 * q + 1], d[4 * q], 0x05010400u), t1 = __builtin_amdgcn_perm(d[4 * q + 1], d[4 * q], 0x07030602u);
      const unsigned t2 = __builtin_amdgcn_perm(d[4 * q + 3], d[4 * q + 2], 0x05010400u), t3 = __builtin_amdgcn_perm(d[4 * q + 3], d[4 * q + 2], 0x07030602u);
      o[0][q] = __builtin_amdgcn_perm(t2, t0, 0x05040100u); o[1][q] = __builtin_amdgcn_perm(t2, t0, 0x07060302u);
      o[2][q] = __builtin_amdgcn_perm(t3, t1, 0x05040100u); o[3][q] = __builtin_amdgcn_perm(t3, t1, 0x07060302u);
    }
  }
}
__device__ __forceinline__ void local_role(const Params& p, unsigned call, unsigned gw) {
  const int lane = threadIdx.x & 63;
  const unsigned rank = (unsigned)p.rank;
  const unsigned par = OFF_DATA + (call & 1u) * PAR_BYTES;
  const rsrc_t srcx = mk_rsrc(p.x, (unsigned)M * KS), srcw = mk_rsrc(p.w, (unsigned)N * KS);
  const unsigned dvo = par + (lane >> 5) * 4096 + (lane & 31) * 16;
  constexpr unsigned NGW = (GRID - NCOPY) * 8;
  const unsigned ua = gw < 128u ? NOPER + gw : gw - 128u, ub = gw + NGW - 128u;
  const bool two = ub < NOPER;
  const Unit a = decode_unit(ua), b = decode_unit(two ? ub : 0u);
  unsigned da[16], db[16]; u32x4 o[4];
  unit_load(a, srcx, srcw, lane, da);
  if (two) unit_load(b, srcx, srcw, lane, db);
  u8* wa = p.ws[gw < 128u ? gw >> 4 : rank];
  {
    const rsrc_t dst = mk_rsrc(wa, (unsigned)WS_NEED);
    unit_retile(a, da, o);
    const unsigned voff = dvo + a.dst + rank * (a.isx ? XREG : WREG);
#pragma unroll
    for (int q = 0; q < 4; ++q) __builtin_amdgcn_raw_buffer_store_b128(o[q], dst, voff + q * 512, 0, 17);
  }
  if (two) {
    const rsrc_t dst = mk_rsrc(p.ws[rank], (unsigned)WS_NEED);
    unit_retile(b, db, o);
    const unsigned voff = dvo + b.dst + rank * WREG;
#pragma unroll
    for (int q = 0; q < 4; ++q) __builtin_amdgcn_raw_buffer_store_b128(o[q], dst, voff + q * 512, 0, 17);
  }
  WAIT_V(0);
  if (lane == 0) {
    __hip_atomic_store((gu32*)(wa + OFF_FLAGS + rank * FLAG_SRC + a.flag), call, __ATOMIC_RELAXED, __HIP_MEMORY_SCOPE_SYSTEM);
    if (two) __hip_atomic_store((gu32*)(p.ws[rank] + OFF_FLAGS + rank * FLAG_SRC + b.flag), call, __ATOMIC_RELAXED, __HIP_MEMORY_SCOPE_SYSTEM);
  }
}
__device__ __forceinline__ void copy_role(const Params& p, unsigned call, int j) {
  const int tid = threadIdx.x, g = rfl(tid >> 6), lane = tid & 63;
  if (g >= NCW) return;
  const unsigned rank = (unsigned)p.rank;
  const unsigned par = OFF_DATA + (call & 1u) * PAR_BYTES;
  const rsrc_t rq = mk_rsrc(p.ws[rank], OFF_FLAGS);
  unsigned qvo = lane == 0 ? OFF_QUEUE : 0x80000000u;
  asm volatile("" : "+v"(qvo));
  const unsigned qbase = (call - 1u) * NUNIT - 2u * NCWAVE;
  unsigned cur = (unsigned)j * NCW + (unsigned)g, nxt = cur + NCWAVE, ticket;
  const unsigned wave = cur;
  {
    rsrc_t dst[NR - 1];
#pragma unroll
    for (int d = 0; d < NR - 1; ++d) dst[d] = mk_rsrc(p.ws[(rank + 1 + d) & 7], (unsigned)WS_NEED);
    const rsrc_t srcx = mk_rsrc(p.x, (unsigned)M * KS), srcw = mk_rsrc(p.w, (unsigned)N * KS);
    u8* flagp = p.ws[(rank + 1 + lane) & 7] + OFF_FLAGS + rank * FLAG_SRC;
    const unsigned dvo = par + (lane >> 5) * 4096 + (lane & 31) * 16;
    unsigned d[16]; u32x4 o[4];
#define PUSH1(q) __builtin_amdgcn_raw_buffer_store_b128(o[(q) / 7], dst[(q) % 7], voff + ((q) / 7) * 512, 0, 17)
#define PUBLISH(fl) do { if (lane < NR - 1) __hip_atomic_store((gu32*)(flagp + (fl)), call, __ATOMIC_RELAXED, __HIP_MEMORY_SCOPE_SYSTEM); } while (0)
    Unit uc = decode_unit(cur);
    unit_load(uc, srcx, srcw, lane, d);
    unsigned pflag = ~0u;
    for (;;) {
      unit_retile(uc, d, o);
      const unsigned voff = dvo + uc.dst + rank * (uc.isx ? XREG : WREG);
      SCHED;
#pragma unroll
      for (int q = 0; q < KPRE; ++q) PUSH1(q);
      WAIT_V(KPRE);
      if (pflag != ~0u) PUBLISH(pflag);
      pflag = uc.flag;
      SCHED;
      const Unit un = decode_unit(nxt < NOPER ? nxt : NOPER - 1u);
      unit_load(un, srcx, srcw, lane, d);
      ticket = (unsigned)__builtin_amdgcn_raw_ptr_buffer_atomic_add_i32(1, rq, qvo, 0, 16);
      SCHED;
#pragma unroll
      for (int q = KPRE; q < 28; ++q) PUSH1(q);
      SCHED;
      cur = nxt; uc = un;
      if (cur >= NOPER) break;
      nxt = rfl(ticket) - qbase;
    }
    WAIT_V(0);
    PUBLISH(pflag);
    CSTAMP(8 + g, __builtin_amdgcn_s_memrealtime());
  }
  nxt = rfl(ticket) - qbase;
  if (cur < NUNIT) {
    u8* myws = p.ws[rank];
    for (int q = 0; q < 4; ++q) spin_flags(myws, OFF_OFLAG + (q * 64 + lane) * 4, 0ull, call);
    rsrc_t dst[NR - 1];
#pragma unroll
    for (int d = 0; d < NR - 1; ++d) dst[d] = mk_rsrc(p.outs[(rank + 1 + d) & 7], (unsigned)M * N * 4u);
    const rsrc_t src = mk_rsrc(p.out, (unsigned)M * N * 4u);
    const unsigned lvo = (unsigned)(lane >> 4) * (N * 4) + (unsigned)(lane & 15) * 16 + (NSLOT * NROUND * 256 + rank * 128) * 4;
#define OUNIT(u_) ({ const unsigned v_ = ((u_) < NUNIT ? (u_) : NUNIT - 1u) - NOPER; ((v_ >> 5) * 256u + ((v_ >> 1) & 7u) * 32u + (v_ & 1u) * 16u) * (N * 4u) + ((v_ >> 4) & 1u) * 256u; })
    u32x4 d[4], o[4];
    {
      const unsigned so = OUNIT(cur);
#pragma unroll
      for (int q = 0; q < 4; ++q) d[q] = __builtin_amdgcn_raw_buffer_load_b128(src, lvo, so + q * 4 * (N * 4), 17);
    }
    for (;;) {
      const unsigned voff = lvo + OUNIT(cur);
#pragma unroll
      for (int q = 0; q < 4; ++q) o[q] = d[q];
      SCHED;
      const unsigned so = OUNIT(nxt);
#pragma unroll
      for (int q = 0; q < 4; ++q) d[q] = __builtin_amdgcn_raw_buffer_load_b128(src, lvo, so + q * 4 * (N * 4), 17);
      ticket = (unsigned)__builtin_amdgcn_raw_ptr_buffer_atomic_add_i32(1, rq, qvo, 0, 16);
      SCHED;
#pragma unroll
      for (int q = 0; q < 4; ++q)
#pragma unroll
        for (int e = 0; e < NR - 1; ++e) __builtin_amdgcn_raw_buffer_store_b128(o[q], dst[e], voff + q * 4 * (N * 4), 0, 17);
      SCHED;
      cur = nxt;
      if (cur >= NUNIT) break;
      nxt = rfl(ticket) - qbase;
    }
  }
  WAIT_V(0);
  if (lane < NR) __hip_atomic_store((gu32*)(p.ws[lane & 7] + OFF_DONE + (rank * 64u + wave) * 4u), call, __ATOMIC_RELAXED, __HIP_MEMORY_SCOPE_SYSTEM);
  CSTAMP(24 + g, __builtin_amdgcn_s_memrealtime());
}

template <int AUX> __device__ __forceinline__ void store_rows(const f32x4 (&o)[4], rsrc_t r, unsigned voff, unsigned soff) {
  SCHED;
#pragma unroll
  for (int j = 0; j < 4; ++j) {
    __builtin_amdgcn_raw_buffer_store_b128(__builtin_bit_cast(u32x4, o[j]), r, voff, soff + (unsigned)j * (N * 4), AUX);
  }
  asm volatile("s_nop 1" ::: "memory");
  SCHED;
}
constexpr float LOG2E = 1.4426950408889634f, CLIP2 = 86.56170245333781f;
__device__ __forceinline__ void silu_rows(const f32x4 (&a)[4], float cz, f32x4 (&o)[4]) {
#pragma unroll
  for (int h = 0; h < 2; ++h) {
    f32x2 z[4], r[4];
#pragma unroll
    for (int m = 0; m < 4; ++m) {
      z[m] = (f32x2){a[m][2 * h], a[m][2 * h + 1]} * cz;
      f32x2 e;
      e.x = __builtin_amdgcn_exp2f(__builtin_amdgcn_fmed3f(-z[m].x, -CLIP2, CLIP2));
      e.y = __builtin_amdgcn_exp2f(__builtin_amdgcn_fmed3f(-z[m].y, -CLIP2, CLIP2));
      const f32x2 dd = __builtin_elementwise_fma(e, (f32x2){LOG2E, LOG2E}, (f32x2){LOG2E, LOG2E});
      r[m].x = __builtin_amdgcn_rcpf(dd.x); r[m].y = __builtin_amdgcn_rcpf(dd.y);
    }
#pragma unroll
    for (int m = 0; m < 4; ++m) {
      o[2 * h][m] = fmul_single(z[m].x, r[m].x); o[2 * h + 1][m] = fmul_single(z[m].y, r[m].y); }
  }
}

__global__ void __launch_bounds__(512, 2) gemm_ag_kernel(Params p) {
  extern __shared__ __attribute__((aligned(16))) unsigned char shm[];
  LAS unsigned char* lds = (LAS unsigned char*)shm;
  u8* myws = p.ws[p.rank];
  const unsigned call = rfl(__hip_atomic_load((gu32*)(myws + OFF_CNT), __ATOMIC_RELAXED, __HIP_MEMORY_SCOPE_AGENT)) / GRID + 1u;
  const int bid = blockIdx.x;
  STAMP(0);
  if (bid < NCOPY) {
    copy_role(p, call, bid);
    if (bid == 0) {
      BAR;
      spin_flags(myws, OFF_DONE + (threadIdx.x >> 6) * 256u + (threadIdx.x & 63u) * 4u, 0ull, call);
      if (rfl(__hip_atomic_load((gu32*)(myws + OFF_TMO), __ATOMIC_RELAXED, __HIP_MEMORY_SCOPE_AGENT)) == call && (threadIdx.x & 63u) == 0u) p.out[0] = __builtin_nanf("");
    }
    STAMP(1);
  }

  const int tid = threadIdx.x, wid = rfl(tid >> 6), lane = tid & 63, wr = wid >> 2, wc = wid & 3, fr = lane & 15, fq = lane >> 4;
  const unsigned par = OFF_DATA + (call & 1u) * PAR_BYTES;
  const rsrc_t rs = mk_rsrc(myws + par, PAR_BYTES);
  const rsrc_t ro = mk_rsrc(p.out, (unsigned)M * N * 4u), rsf = mk_rsrc(myws, OFF_DATA);
  const unsigned voff = lane * 16;
  const unsigned ldsw = (unsigned)wid * 2048u;
  const unsigned xw = (unsigned)wid * XREG, ww = XALL + (unsigned)wid * WREG;
  const float cz = p.sx[0] * p.sw[0] * LOG2E;
#define STAGE(bufoff, soff) do { \
    __builtin_amdgcn_raw_ptr_buffer_load_lds(rs, (LAS void*)(lds + (bufoff) + ldsw), 16, voff, (soff), 0, 17); \
    __builtin_amdgcn_raw_ptr_buffer_load_lds(rs, (LAS void*)(lds + (bufoff) + ldsw), 16, voff, (soff), 1024, 17); } while (0)

  const unsigned voffE = (unsigned)(lane >> 4) * 512u + (unsigned)(lane & 15) * 16u;
#define STAGEE(buf, soff) __builtin_amdgcn_raw_ptr_buffer_load_lds(rs, (LAS void*)(lds + (buf) + wid * 1024), 16, voffE, (soff), 0, 17)
#define EBASE(q) (ww + (unsigned)(NSLOT * NROUND + (p.rank >> 1)) * (NT * 4096) + (unsigned)((p.rank & 1) * 2048) + (unsigned)((q) & 1) * 256)
  if (bid >= NCOPY) {
    local_role(p, call, (unsigned)(bid - NCOPY) * 8u + (unsigned)wid);
    const int gidx = bid - NCOPY, xcd = gidx & 7, idx = gidx >> 3, tl = (xcd >> 1) * 30 + idx;
    const int pm = (xcd & 1) * 8 + (tl & 7), slot = tl >> 3;
    const LAS unsigned char* pA = lds + fq * 2048 + wr * 256 + fr * 16;
    const LAS unsigned char* pB = lds + 4 * HTB + fq * 2048 + wc * 512 + fr * 16;
    const LAS unsigned char* pE = lds + LDS_E + fq * 1024 + fr * 16;
    asm volatile("" : "+v"(pA), "+v"(pB), "+v"(pE));
#define SA(b, h) (((b) * 2 + (h)) * HTB)
#define SB(b, h) ((4 + (b) * 2 + (h)) * HTB)
#define LDA(dst, b, h) do { _Pragma("unroll") for (int m = 0; m < 4; ++m) _Pragma("unroll") for (int k = 0; k < 2; ++k) dst[m][k] = *(const LAS i32x4*)(pA + SA(b, h) + m * 512 + k * 8192); } while (0)
#define LDB(dst, b, h) do { _Pragma("unroll") for (int n = 0; n < 2; ++n) _Pragma("unroll") for (int k = 0; k < 2; ++k) dst[n][k] = *(const LAS i32x4*)(pB + SA(b, h) + n * 256 + k * 8192); } while (0)
#define LDE(dst, b) do { _Pragma("unroll") for (int m = 0; m < 4; ++m) _Pragma("unroll") for (int k = 0; k < 2; ++k) dst[m][k] = *(const LAS i32x4*)(pE + (b) * 8192 + m * 256 + k * 4096); } while (0)
#define MIDX(q) (((q) >> 2) ? 3 - ((q) & 3) : ((q) & 3))
#define NIDX(q) ((q) >> 2)
#define MM8(C, At, Bt) do { __builtin_amdgcn_s_setprio(1); _Pragma("unroll") for (int q = 0; q < 8; ++q) { const int m = MIDX(q), n = NIDX(q); \
        asm volatile("v_mfma_f32_16x16x128_f8f6f4 %0, %1, %2, %0" : "+v"(C[m][n]) : "v"(cat(Bt[n][0], Bt[n][1])), "v"(cat(At[m][0], At[m][1]))); } __builtin_amdgcn_s_setprio(0); } while (0)
#define MM8Z(C, At, Bt) do { __builtin_amdgcn_s_setprio(1); _Pragma("unroll") for (int q = 0; q < 8; ++q) { const int m = MIDX(q), n = NIDX(q); \
        asm volatile("v_mfma_f32_16x16x128_f8f6f4 %0, %1, %2, 0" : "=&v"(C[m][n]) : "v"(cat(Bt[n][0], Bt[n][1])), "v"(cat(At[m][0], At[m][1]))); } __builtin_amdgcn_s_setprio(0); } while (0)
#define SELB() do { _Pragma("unroll") for (int n = 0; n < 2; ++n) _Pragma("unroll") for (int k = 0; k < 2; ++k) { B0[n][k] = wr ? B1[n][k] : B0[n][k]; asm volatile("" : "+v"(B0[n][k])); } } while (0)
#define MME(At) MM8(accE, At, B0)
#define MMEZ(At) MM8Z(accE, At, B0)
#define MMA(ai, bj, At, Bt) MM8(acc[ai][bj], At, Bt)
#define MMA0(ai, bj, At, Bt) MM8Z(acc[ai][bj], At, Bt)
    f32x4 acc[2][2][4][2], accE[4][2];
    i32x4 At[4][2], B0[2][2], B1[2][2];
    const unsigned kstep = 4096, hstep = 2048, tstep = NT * 4096;
    const unsigned cB = xw + (unsigned)pm * tstep;
    unsigned cA = ww + (unsigned)slot * tstep;
    const bool ex = slot < 2;
    const unsigned cE = EBASE(slot);
    const int l8 = lane & 7;
    const unsigned foff = OFF_FLAGS + (unsigned)wid * FLAG_SRC + (l8 < 2 ? pm * 2 + l8 : l8 == 4 ? (16 + NSLOT * NROUND) * 2 + p.rank : (16 + slot) * 2 + (l8 == 3)) * 64;
    constexpr unsigned long long IGN_XE = 0x1313131313131313ull, IGN_E = 0x1010101010101010ull;
    spin_flags(myws, foff, ex ? 0ull : IGN_E, call);
    int fval = -1; unsigned fnew = call - 1u, tm = 0u;
    unsigned sA = 4u, sB = 8u, sC = 12u;
    STAMP(1);
    if (ex) STAGEE(LDS_E, cE);
    STAGE(SB(0, 0), cB); STAGE(SA(0, 0), cA); STAGE(SB(0, 1), cB + hstep); STAGE(SA(0, 1), cA + hstep);
    if (wr == 1) BAR;
    WAIT_V(4); BAR;
    STAGE(SB(1, 0), cB + kstep); STAGE(SA(1, 0), cA + kstep); STAGE(SB(1, 1), cB + hstep + kstep); STAGE(SA(1, 1), cA + hstep + kstep);
    WAIT_V(8); BAR;
#define FCHECK(ign) do { if ((__ballot(fval >= 0) | (ign)) != ~0ull) spin_flags(myws, foff + sA, (ign), call); } while (0)
#define NEXTLEV() do { sA = sB; sB = sC; sC += (sC & 63u) == 60u ? FLAG_ROUND - 60u : 4u; } while (0)
#define ITER(EX, FIRST, MM, MMX, W4, W8, FCHK, a1h, a2, b2, e1, e2, LAST) do { \
        const unsigned a3 = (a2) + kstep, b3 = (b2) + kstep; \
          \
        LDB(B0, 0, 0); SCHED; LDA(At, 0, 0); if (!(FIRST)) STAGE(SA(1, 1), a1h); \
        WAIT_L(8); BAR; WAIT_L(0); MM(0, 0, At, B0); BAR; SCHED; \
          \
        FCHK; \
        LDB(B1, 0, 1); STAGE(SB(0, 0), b2); if (EX) STAGEE(LDS_E + 8192, e1); \
        BAR; WAIT_L(0); MM(0, 1, At, B1); BAR; \
          \
        LDA(At, 0, 1); STAGE(SA(0, 0), a2); \
        BAR; WAIT_L(0); MM(1, 0, At, B0); BAR; SCHED; \
          \
        STAGE(SB(0, 1), (b2) + hstep); \
        W4; \
        fval = (int)(fnew - call); asm volatile("" : "+v"(fval)); \
        BAR; MM(1, 1, At, B1); BAR; \
        if (EX) { SCHED; LDE(At, 0); SELB(); BAR; WAIT_L(0); MMX(At); BAR; SCHED; } \
          \
        LDB(B0, 1, 0); SCHED; LDA(At, 1, 0); STAGE(SA(0, 1), (a2) + hstep); \
        WAIT_L(8); BAR; WAIT_L(0); MMA(0, 0, At, B0); BAR; SCHED; \
          \
        LDB(B1, 1, 1); STAGE(SB(1, 0), b3); if (EX) STAGEE(LDS_E, e2); \
        fnew = __builtin_amdgcn_raw_buffer_load_b32(rsf, foff, sC, 17); \
        if (LAST) tm = __builtin_amdgcn_raw_buffer_load_b32(rsf, OFF_TMO, 0, 17);        \
        BAR; WAIT_L(0); MMA(0, 1, At, B1); BAR; \
          \
        LDA(At, 1, 1); STAGE(SA(1, 0), a3); \
        BAR; WAIT_L(0); MMA(1, 0, At, B0); BAR; SCHED; \
          \
        STAGE(SB(1, 1), b3 + hstep); \
        NEXTLEV(); \
        W8; BAR; MMA(1, 1, At, B1); BAR; \
        if (EX) { SCHED; LDE(At, 1); SELB(); BAR; WAIT_L(0); MME(At); BAR; SCHED; } } while (0)
#define EPILOGUE(r) do { \
        int tz = threadIdx.x; asm volatile("" : "+v"(tz)); \
        const int lane2 = tz & 63, fr2 = lane2 & 15, fq2 = lane2 >> 4; \
        const int pn = NSLOT * (r) + slot; \
        const unsigned ovoff = (unsigned)(wc * 32 + fq2 * 4) * (N * 4) + (unsigned)(wr * 64 + fr2 * 4) * 4; \
        const unsigned osoff = (unsigned)(pm * 256) * (N * 4) + (unsigned)pn * 1024; \
        const bool bad = rfl(tm) == call; \
        const float czz = bad ? __builtin_nanf("") : cz; \
        _Pragma("unroll") for (int ai = 0; ai < 2; ++ai) _Pragma("unroll") for (int bj = 0; bj < 2; ++bj) _Pragma("unroll") for (int n = 0; n < 2; ++n) { \
              const f32x4 a4[4] = {acc[ai][bj][0][n], acc[ai][bj][1][n], acc[ai][bj][2][n], acc[ai][bj][3][n]}; \
              f32x4 o4[4]; \
              silu_rows(a4, czz, o4); \
              store_rows<2>(o4, ro, ovoff, osoff + (unsigned)(bj * 128 + n * 16) * (N * 4) + ai * 512); } } while (0)
    if (ex) {
      const unsigned nA = cA + NSLOT * tstep;
      ITER(true, true, MMA0, MMEZ, WAIT_V(7), WAIT_V(8), FCHECK(0ull), 0u, cA + 2 * kstep, cB + 2 * kstep, cE + kstep, cE + 2 * kstep, false);
      for (int t = 2; t < NT - 2; t += 2) {
        ITER(true, false, MMA, MME, WAIT_V(7), WAIT_V(8), FCHECK(0ull), cA + (unsigned)(t + 1) * kstep + hstep, cA + (unsigned)(t + 2) * kstep, cB + (unsigned)(t + 2) * kstep,
             cE + (unsigned)(t + 1) * kstep, cE + (unsigned)(t + 2) * kstep, false);
      }
      ITER(true, false, MMA, MME, WAIT_V(7), WAIT_V(8), FCHECK(IGN_XE), cA + (unsigned)(NT - 1) * kstep + hstep, nA, cB, cE + (unsigned)(NT - 1) * kstep, cE, true);
      asm volatile("s_nop 15" ::: "memory");
      STAMP(2);
      {
        int tz = threadIdx.x; asm volatile("" : "+v"(tz));
        const int lane2 = tz & 63, fr2 = lane2 & 15, fq2 = lane2 >> 4;
        const unsigned ovoff = (unsigned)(wr * 128 + wc * 32 + fq2 * 4) * (N * 4) + (unsigned)(fr2 * 4) * 4;
        const unsigned osoff = (unsigned)(pm * 256) * (N * 4) + (unsigned)(NSLOT * NROUND * 256 + p.rank * 128 + slot * 64) * 4;
        const float czz = rfl(tm) == call ? __builtin_nanf("") : cz;
#pragma unroll
        for (int n = 0; n < 2; ++n) {
          const f32x4 a4[4] = {accE[0][n], accE[1][n], accE[2][n], accE[3][n]};
          f32x4 o4[4];
          silu_rows(a4, czz, o4);
          store_rows<17>(o4, ro, ovoff, osoff + (unsigned)(n * 16) * (N * 4));
        }
      }
      STAGE(SA(1, 1), nA + kstep + hstep);
      EPILOGUE(0);
      STAMP(3);
      cA = nA;
    }
    for (int r = ex ? 1 : 0; r < NROUND; ++r) {
      const bool has_next = r + 1 < NROUND;
      const unsigned long long ign = r == 0 ? IGN_E : IGN_XE;
      const unsigned nA = has_next ? cA + NSLOT * tstep : cA;
      ITER(false, true, MMA0, MMEZ, if (r > 0) { WAIT_V(38); if (ex && r == 1 && lane == 0) __hip_atomic_store((gu32*)(myws + OFF_OFLAG + ((pm * 2 + slot) * 8 + wid) * 4), call, __ATOMIC_RELAXED, __HIP_MEMORY_SCOPE_SYSTEM); } else WAIT_V(6),
           WAIT_V(7), FCHECK(ign), 0u, cA + 2 * kstep, cB + 2 * kstep, 0u, 0u, false);
      for (int t = 2; t < NT - 2; t += 2) {
        ITER(false, false, MMA, MME, WAIT_V(6), WAIT_V(7), FCHECK(ign), cA + (unsigned)(t + 1) * kstep + hstep, cA + (unsigned)(t + 2) * kstep, cB + (unsigned)(t + 2) * kstep, 0u, 0u, false);
      }
      const unsigned long long ignl = has_next ? IGN_XE : ~0ull;
      ITER(false, false, MMA, MME, WAIT_V(6), WAIT_V(7), FCHECK(ignl), cA + (unsigned)(NT - 1) * kstep + hstep, nA, cB, 0u, 0u, true);
      STAGE(SA(1, 1), nA + kstep + hstep);
      asm volatile("s_nop 15" ::: "memory");
      STAMP(2 + 2 * r);
      EPILOGUE(r);
      STAMP(3 + 2 * r);
      cA = nA;
    }
    WAIT_V(0);
    if (wr == 0) BAR;
  }
  STAMP(11);
  if (threadIdx.x == 0) __hip_atomic_fetch_add((gu32*)(myws + OFF_CNT), 1u, __ATOMIC_RELAXED, __HIP_MEMORY_SCOPE_AGENT);
}
}
#define HIP_CHECK(expr) do { hipError_t _e = (expr); if (_e != hipSuccess) { \
    fprintf(stderr, "HIP error %s at %s:%d\n", hipGetErrorString(_e), __FILE__, __LINE__); \
    return; } } while (0)

extern "C" void kernel_launch(int rank, int nranks, ncclComm_t comm, hipStream_t stream,
                              void* const* d_in, const int* in_sizes, int n_in,
                              void* d_out, int out_size, const int* in_dtypes, int out_dtype,
                              void* const* d_out_all, void* const* d_ws_all, size_t ws_bytes) {
  (void)comm; (void)in_dtypes; (void)out_dtype; (void)d_out_all;
  if (nranks != gar::NR || n_in != 4 || (unsigned)rank >= (unsigned)gar::NR || ws_bytes < gar::WS_NEED ||
      in_sizes[0] != gar::M * gar::KS || in_sizes[1] != gar::N * gar::KS || out_size != gar::M * gar::N) {
    fprintf(stderr, "kernel_launch: unexpected shapes\n");
    return;
  }
  static bool attr_set = false;
  if (!attr_set) {
    HIP_CHECK(hipFuncSetAttribute((const void*)gar::gemm_ag_kernel, hipFuncAttributeMaxDynamicSharedMemorySize, gar::LDS_BYTES));
    attr_set = true;
  }
  gar::Params p;
  p.x = (const gar::u8*)d_in[0]; p.w = (const gar::u8*)d_in[1];
  p.sx = (const float*)d_in[2]; p.sw = (const float*)d_in[3];
  p.out = (float*)d_out;
  for (int i = 0; i < gar::NR; ++i) p.ws[i] = (gar::u8*)d_ws_all[i];
  for (int i = 0; i < gar::NR; ++i) p.outs[i] = (float*)d_out_all[i];
  p.rank = rank; p.pad = 0;
  gar::gemm_ag_kernel<<<gar::GRID, 512, gar::LDS_BYTES, stream>>>(p);
  HIP_CHECK(hipGetLastError());
}
```

```cpp
#include <hip/hip_runtime.h>
#include <rccl/rccl.h>
#include <cstdio>

extern "C" __device__ float gar_fmul_legacy(float, float) __asm("llvm.amdgcn.fmul.legacy");
namespace gar {
__device__ __forceinline__ float fmul_single(float a, float b) { return gar_fmul_legacy(a, b); }
#define LAS __attribute__((address_space(3)))
typedef int i32x4 __attribute__((ext_vector_type(4)));
typedef int i32x8 __attribute__((ext_vector_type(8)));
typedef unsigned u32x4 __attribute__((ext_vector_type(4)));
typedef float f32x4 __attribute__((ext_vector_type(4)));
typedef float f32x2 __attribute__((ext_vector_type(2)));
typedef unsigned char u8;
typedef __amdgpu_buffer_rsrc_t rsrc_t;
typedef __attribute__((address_space(1))) unsigned gu32;

constexpr int M = 4096, N = 16384, NR = 8, KS = 512;
constexpr int NT = 32;
constexpr int HTB = 16384;
constexpr int LDS_E = 8 * HTB;
constexpr int LDS_BYTES = 8 * HTB + 16384;
constexpr int GRID = 256, NCOPY = 16, NSLOT = 15, NROUND = 4;
#define NCW 4
constexpr unsigned NCWAVE = NCOPY * NCW;
#define GAR_OUTS 1
constexpr unsigned NOPER = 2432, NUNIT0 = 992, NUNIT = NOPER + 512;
constexpr unsigned NLOCAL = NOPER + 128;
constexpr unsigned OFF_CNT = 0, OFF_TMO = 256, OFF_QUEUE = 512;
constexpr unsigned OFF_OFLAG = 4096, OFF_DONE = 8192;
constexpr unsigned OFF_FLAGS = 65536;
constexpr unsigned FLAG_SRC = 80 * 128, FLAG_ROUND = NSLOT * 128;
constexpr unsigned OFF_DATA = 1u << 20;
constexpr unsigned XREG = (unsigned)M * KS;
constexpr unsigned WREG = (unsigned)N * KS;
constexpr unsigned XALL = NR * XREG, WALL = NR * WREG;
constexpr unsigned PAR_BYTES = XALL + WALL;
constexpr size_t WS_NEED = (size_t)OFF_DATA + 2 * (size_t)PAR_BYTES;
constexpr unsigned long long DEADLINE = 30000000ull;
constexpr int RSRC3 = 0x00020000;
#define SLEEPV 8
#define OUTP 1
#define STAGN 2
#define KPRE 7

struct Params {
  const u8* x; const u8* w; const float* sx; const float* sw; float* out;
  u8* ws[NR];
  float* outs[NR];
  int rank; int pad;
};

__device__ __forceinline__ i32x8 cat(i32x4 a, i32x4 b) { return __builtin_shufflevector(a, b, 0, 1, 2, 3, 4, 5, 6, 7); }
__device__ __forceinline__ unsigned rfl(unsigned v) { return __builtin_amdgcn_readfirstlane(v); }
__device__ __forceinline__ rsrc_t mk_rsrc(const void* p, unsigned bytes) {
  const unsigned long long a = (unsigned long long)p;
  const unsigned lo = rfl((unsigned)a), hi = rfl((unsigned)(a >> 32));
  return __builtin_amdgcn_make_buffer_rsrc((void*)(((unsigned long long)hi << 32) | lo), (short)0, (int)bytes, RSRC3);
}
#define STR2(x) #x
#define STR(x) STR2(x)
#define WAIT_V(n) asm volatile("s_waitcnt vmcnt(" STR(n) ")" ::: "memory")
#define WAIT_L(n) asm volatile("s_waitcnt lgkmcnt(" STR(n) ")" ::: "memory")
#define BAR __builtin_amdgcn_s_barrier()
#define SCHED __builtin_amdgcn_sched_barrier(0)
#define STAMP(i)
#define CSTAMP(e, v)

__device__ __noinline__ void spin_flags(u8* ws, unsigned off, unsigned long long ign, unsigned call) {
  gu32* tmo = (gu32*)(ws + OFF_TMO);
  gu32* fp = (gu32*)(ws + off);
  const unsigned long long t0 = __builtin_amdgcn_s_memrealtime();
  for (;;) {
    const unsigned v = __hip_atomic_load(fp, __ATOMIC_RELAXED, __HIP_MEMORY_SCOPE_SYSTEM);
    if ((__ballot((int)(v - call) >= 0) | ign) == ~0ull) return;
    if (rfl(__hip_atomic_load(tmo, __ATOMIC_RELAXED, __HIP_MEMORY_SCOPE_AGENT)) == call || __builtin_amdgcn_s_memrealtime() - t0 > DEADLINE) {
      __hip_atomic_store(tmo, call, __ATOMIC_RELAXED, __HIP_MEMORY_SCOPE_AGENT);
      WAIT_V(0);
      return;
    }
    __builtin_amdgcn_s_sleep(SLEEPV);
  }
}

#define XDL 5, 3, 5, 3, 5, 3, 5, 3, 2, 0, 2, 0, 2, 0, 2, 0
struct XSched { unsigned long long perm, del, cnt; unsigned last; };
constexpr XSched make_xsched() {
  const unsigned d[16] = {XDL};
  XSched s{0ull, 0ull, 0ull, 0u};
  unsigned n = 0;
  for (unsigned v = 0; v < 12; ++v) {
    for (unsigned i = 0; i < 16; ++i) if (d[i] == v) { s.perm |= (unsigned long long)i << (4 * n); s.del |= (unsigned long long)v << (4 * n); s.last = v; ++n; }
    s.cnt |= (unsigned long long)n << (5 * v);
  }
  return s;
}
constexpr XSched XS = make_xsched();
static_assert(((XS.cnt >> 55) & 31) == 16 && (XS.del & 15) == 0, "delays 0..11, the first 0");
struct Unit { unsigned src, dst, flag; int kind; };
__device__ __forceinline__ Unit decode_unit(unsigned u) {
  unsigned panel, h, lev; bool isx = false;
  if (u < NOPER) {
    unsigned v = 0u, jj = u, g0 = 0u, nx = 0u;
    constexpr unsigned VA = 16u + XS.last, NA = 512u + VA * 30u;
#define XCNT(t) ((t) >= 11u ? 16u : (unsigned)(XS.cnt >> (5u * (t))) & 31u)
    if (u < NA) {
      for (;; ++v) {
        g0 = v < 16u ? 0u : XCNT(v - 16u);
        nx = 2u * (XCNT(v) - g0);
        if (jj < nx + 30u) break;
        jj -= nx + 30u;
      }
    } else { const unsigned t = u - NA; v = t / 30u; jj = t - v * 30u; v += VA; }
    isx = jj < nx;
    const unsigned si = 4u * ((g0 + (jj >> 1)) & 15u), jw = jj - nx;
    h = jj & 1u;
    panel = isx ? (unsigned)(XS.perm >> si) & 15u : NSLOT * (v >> 4) + (jw >> 1);
    lev = isx ? v - ((unsigned)(XS.del >> si) & 15u) : v & 15u;
  } else {
    const unsigned v = u - NOPER;
    lev = v & 15u; h = (v >> 4) & 1u; panel = NSLOT * NROUND + (v >> 5);
  }
  Unit o; o.kind = isx ? 0 : 1;
  o.src = isx ? (panel * 256u + h * 128u) * KS + lev * 32u : lev * 32u * N + panel * 256u + h * 128u;
  o.dst = (isx ? 0u : XALL) + panel * (NT * 4096u) + lev * 8192u + h * 2048u;
  o.flag = (((isx ? 0u : 16u) + panel) * 2u + h) * 64u + lev * 4u;
  return o;
}
__device__ __forceinline__ Unit decode_queue(unsigned u) {
  if (u < NOPER) return decode_unit(u);
  const unsigned v = (u < NUNIT ? u : NUNIT - 1u) - NOPER;
  Unit o; o.kind = 2; o.flag = ~0u;
  o.src = o.dst = ((v >> 5) * 256u + ((v >> 1) & 7u) * 32u + (v & 1u) * 16u) * (N * 4u) + ((v >> 4) & 1u) * 256u;
  return o;
}
__device__ __forceinline__ void unit_load(const Unit& u, rsrc_t srcx, rsrc_t srcw, int lane, unsigned (&d)[16]) {
  if (u.kind == 0) {
#pragma unroll
    for (int t = 0; t < 4; ++t) { const u32x4 v = __builtin_amdgcn_raw_buffer_load_b128(srcx, (lane & 31) * KS + (lane >> 5) * 16, u.src + t * 32 * KS, 0);
      d[4 * t] = v[0]; d[4 * t + 1] = v[1]; d[4 * t + 2] = v[2]; d[4 * t + 3] = v[3]; }
  } else {
#pragma unroll
    for (int i = 0; i < 16; ++i) d[i] = __builtin_amdgcn_raw_buffer_load_b32(srcw, (lane >> 5) * 16 * N + (lane & 31) * 4, u.src + i * N, 0);
  }
}
__device__ __forceinline__ void unit_retile(const Unit& u, const unsigned (&d)[16], u32x4 (&o)[4]) {
  if (u.kind != 1) {
#pragma unroll
    for (int t = 0; t < 4; ++t) o[t] = (u32x4){d[4 * t], d[4 * t + 1], d[4 * t + 2], d[4 * t + 3]};
  } else {
#pragma unroll
    for (int q = 0; q < 4; ++q) {
      const unsigned t0 = __builtin_amdgcn_perm(d[4 * q + 1], d[4 * q], 0x05010400u), t1 = __builtin_amdgcn_perm(d[4 * q + 1], d[4 * q], 0x07030602u);
      const unsigned t2 = __builtin_amdgcn_perm(d[4 * q + 3], d[4 * q + 2], 0x05010400u), t3 = __builtin_amdgcn_perm(d[4 * q + 3], d[4 * q + 2], 0x07030602u);
      o[0][q] = __builtin_amdgcn_perm(t2, t0, 0x05040100u); o[1][q] = __builtin_amdgcn_perm(t2, t0, 0x07060302u);
      o[2][q] = __builtin_amdgcn_perm(t3, t1, 0x05040100u); o[3][q] = __builtin_amdgcn_perm(t3, t1, 0x07060302u);
    }
  }
}
__device__ __forceinline__ void local_role(const Params& p, unsigned call, unsigned gw) {
  const int lane = threadIdx.x & 63;
  const unsigned rank = (unsigned)p.rank;
  const unsigned par = OFF_DATA + (call & 1u) * PAR_BYTES;
  const rsrc_t srcx = mk_rsrc(p.x, (unsigned)M * KS), srcw = mk_rsrc(p.w, (unsigned)N * KS);
  const unsigned dvo = par + (lane >> 5) * 4096 + (lane & 31) * 16;
  constexpr unsigned NGW = (GRID - NCOPY) * 8;
  const unsigned ua = gw < 128u ? NOPER + gw : gw - 128u, ub = gw + NGW - 128u;
  const bool two = ub < NOPER;
  const Unit a = decode_unit(ua), b = decode_unit(two ? ub : 0u);
  unsigned da[16], db[16]; u32x4 o[4];
  unit_load(a, srcx, srcw, lane, da);
  if (two) unit_load(b, srcx, srcw, lane, db);
  u8* wa = p.ws[gw < 128u ? gw >> 4 : rank];
  {
    const rsrc_t dst = mk_rsrc(wa, (unsigned)WS_NEED);
    unit_retile(a, da, o);
    const unsigned voff = dvo + a.dst + rank * (a.kind == 0 ? XREG : WREG);
#pragma unroll
    for (int q = 0; q < 4; ++q) __builtin_amdgcn_raw_buffer_store_b128(o[q], dst, voff + q * 512, 0, 17);
  }
  if (two) {
    const rsrc_t dst = mk_rsrc(p.ws[rank], (unsigned)WS_NEED);
    unit_retile(b, db, o);
    const unsigned voff = dvo + b.dst + rank * WREG;
#pragma unroll
    for (int q = 0; q < 4; ++q) __builtin_amdgcn_raw_buffer_store_b128(o[q], dst, voff + q * 512, 0, 17);
  }
  WAIT_V(0);
  if (lane == 0) {
    __hip_atomic_store((gu32*)(wa + OFF_FLAGS + rank * FLAG_SRC + a.flag), call, __ATOMIC_RELAXED, __HIP_MEMORY_SCOPE_SYSTEM);
    if (two) __hip_atomic_store((gu32*)(p.ws[rank] + OFF_FLAGS + rank * FLAG_SRC + b.flag), call, __ATOMIC_RELAXED, __HIP_MEMORY_SCOPE_SYSTEM);
  }
}
__device__ __forceinline__ void copy_role(const Params& p, unsigned call, int j) {
  const int tid = threadIdx.x, g = rfl(tid >> 6), lane = tid & 63;
  if (g >= NCW) return;
  const unsigned rank = (unsigned)p.rank;
  const unsigned par = OFF_DATA + (call & 1u) * PAR_BYTES;
  u8* myws = p.ws[rank];
  const rsrc_t rq = mk_rsrc(myws, OFF_FLAGS);
  unsigned qvo = lane == 0 ? OFF_QUEUE : 0x80000000u;
  asm volatile("" : "+v"(qvo));
  const unsigned qbase = (call - 1u) * NUNIT - 2u * NCWAVE;
  unsigned cur = (unsigned)j * NCW + (unsigned)g, nxt = cur + NCWAVE;
  const unsigned wave = cur;
  rsrc_t dst[NR - 1], dsto[NR - 1];
#pragma unroll
  for (int d = 0; d < NR - 1; ++d) { dst[d] = mk_rsrc(p.ws[(rank + 1 + d) & 7], (unsigned)WS_NEED); dsto[d] = mk_rsrc(p.outs[(rank + 1 + d) & 7], (unsigned)M * N * 4u); }
  const rsrc_t srcx = mk_rsrc(p.x, (unsigned)M * KS), srcw = mk_rsrc(p.w, (unsigned)N * KS), srco = mk_rsrc(p.out, (unsigned)M * N * 4u);
  u8* flagp = p.ws[(rank + 1 + lane) & 7] + OFF_FLAGS + rank * FLAG_SRC;
  const unsigned dvo = par + (lane >> 5) * 4096 + (lane & 31) * 16;
  const unsigned ovo = (unsigned)(lane >> 4) * (N * 4) + (unsigned)(lane & 15) * 16 + (NSLOT * NROUND * 256 + rank * 128) * 4;
  unsigned d[16]; u32x4 o[4];
#define PUSH(q0, q1) do { if (uc.kind == 2) { _Pragma("unroll") for (int q = q0; q < q1; ++q) __builtin_amdgcn_raw_buffer_store_b128(o[q / 7], dsto[q % 7], voff + (q / 7) * 4 * (N * 4), 0, 17); } \
    else { _Pragma("unroll") for (int q = q0; q < q1; ++q) __builtin_amdgcn_raw_buffer_store_b128(o[q / 7], dst[q % 7], voff + (q / 7) * 512, 0, 17); } } while (0)
#define PUBLISH(fl) do { if (lane < NR - 1) __hip_atomic_store((gu32*)(flagp + (fl)), call, __ATOMIC_RELAXED, __HIP_MEMORY_SCOPE_SYSTEM); } while (0)
  Unit uc = decode_unit(cur);
  unit_load(uc, srcx, srcw, lane, d);
  unsigned pflag = ~0u;
  bool oready = false;
  for (;;) {
    const Unit un = decode_queue(nxt);
    SCHED;
    unit_retile(uc, d, o);
    const unsigned voff = uc.kind == 2 ? ovo + uc.dst : dvo + uc.dst + rank * (uc.kind == 0 ? XREG : WREG);
    SCHED;
    PUSH(0, KPRE);
    WAIT_V(KPRE);
    if (pflag != ~0u) PUBLISH(pflag);
    pflag = uc.flag;
    SCHED;
    if (!oready && nxt >= NOPER - 480u + wave * 6u) {
      const u32x4 of = __builtin_amdgcn_raw_buffer_load_b128(rq, OFF_OFLAG + lane * 16, 0, 17);
      if (!__all((int)(of[0] - call) >= 0 && (int)(of[1] - call) >= 0 && (int)(of[2] - call) >= 0 && (int)(of[3] - call) >= 0))
        for (int q = 0; q < 4; ++q) spin_flags(myws, OFF_OFLAG + (q * 64 + lane) * 4, 0ull, call);
      oready = true;
    }
    if (un.kind == 2) {
#pragma unroll
      for (int t = 0; t < 4; ++t) { const u32x4 v = __builtin_amdgcn_raw_buffer_load_b128(srco, ovo, un.src + t * 4 * (N * 4), 17);
        d[4 * t] = v[0]; d[4 * t + 1] = v[1]; d[4 * t + 2] = v[2]; d[4 * t + 3] = v[3]; }
    } else unit_load(un, srcx, srcw, lane, d);
    const unsigned ticket = (unsigned)__builtin_amdgcn_raw_ptr_buffer_atomic_add_i32(1, rq, qvo, 0, 16);
    SCHED;
    PUSH(KPRE, 28);
    SCHED;
    cur = nxt; uc = un;
    if (cur >= NUNIT) break;
    nxt = rfl(ticket) - qbase;
  }
  WAIT_V(0);
  if (pflag != ~0u) PUBLISH(pflag);
  if (lane < NR) __hip_atomic_store((gu32*)(p.ws[lane & 7] + OFF_DONE + (rank * 64u + wave) * 4u), call, __ATOMIC_RELAXED, __HIP_MEMORY_SCOPE_SYSTEM);
  CSTAMP(24 + g, __builtin_amdgcn_s_memrealtime());
}

template <int AUX> __device__ __forceinline__ void store_rows(const f32x4 (&o)[4], rsrc_t r, unsigned voff, unsigned soff) {
  SCHED;
#pragma unroll
  for (int j = 0; j < 4; ++j) {
    __builtin_amdgcn_raw_buffer_store_b128(__builtin_bit_cast(u32x4, o[j]), r, voff, soff + (unsigned)j * (N * 4), AUX);
  }
  asm volatile("s_nop 1" ::: "memory");
  SCHED;
}
constexpr float LOG2E = 1.4426950408889634f, CLIP2 = 86.56170245333781f;
__device__ __forceinline__ void silu_rows(const f32x4 (&a)[4], float cz, f32x4 (&o)[4]) {
#pragma unroll
  for (int h = 0; h < 2; ++h) {
    f32x2 z[4], r[4];
#pragma unroll
    for (int m = 0; m < 4; ++m) {
      z[m] = (f32x2){a[m][2 * h], a[m][2 * h + 1]} * cz;
      f32x2 e;
      e.x = __builtin_amdgcn_exp2f(__builtin_amdgcn_fmed3f(-z[m].x, -CLIP2, CLIP2));
      e.y = __builtin_amdgcn_exp2f(__builtin_amdgcn_fmed3f(-z[m].y, -CLIP2, CLIP2));
      const f32x2 dd = __builtin_elementwise_fma(e, (f32x2){LOG2E, LOG2E}, (f32x2){LOG2E, LOG2E});
      r[m].x = __builtin_amdgcn_rcpf(dd.x); r[m].y = __builtin_amdgcn_rcpf(dd.y);
    }
#pragma unroll
    for (int m = 0; m < 4; ++m) {
      o[2 * h][m] = fmul_single(z[m].x, r[m].x); o[2 * h + 1][m] = fmul_single(z[m].y, r[m].y); }
  }
}

__global__ void __launch_bounds__(512, 2) gemm_ag_kernel(Params p) {
  extern __shared__ __attribute__((aligned(16))) unsigned char shm[];
  LAS unsigned char* lds = (LAS unsigned char*)shm;
  u8* myws = p.ws[p.rank];
  const unsigned call = rfl(__hip_atomic_load((gu32*)(myws + OFF_CNT), __ATOMIC_RELAXED, __HIP_MEMORY_SCOPE_AGENT)) / GRID + 1u;
  const int bid = blockIdx.x;
#define XMAP 0
  const bool is_copy = bid < NCOPY;
  const int cj = bid;
  STAMP(0);
  if (is_copy) {
    copy_role(p, call, cj);
    if (cj == 0) {
      BAR;
      spin_flags(myws, OFF_DONE + (threadIdx.x >> 6) * 256u + (threadIdx.x & 63u) * 4u, 0ull, call);
      if (rfl(__hip_atomic_load((gu32*)(myws + OFF_TMO), __ATOMIC_RELAXED, __HIP_MEMORY_SCOPE_AGENT)) == call && (threadIdx.x & 63u) == 0u) p.out[0] = __builtin_nanf("");
    }
    STAMP(1);
  }

  const int tid = threadIdx.x, wid = rfl(tid >> 6), lane = tid & 63, wr = wid >> 2, wc = wid & 3, fr = lane & 15, fq = lane >> 4;
  const unsigned par = OFF_DATA + (call & 1u) * PAR_BYTES;
  const rsrc_t rs = mk_rsrc(myws + par, PAR_BYTES);
  const rsrc_t ro = mk_rsrc(p.out, (unsigned)M * N * 4u), rsf = mk_rsrc(myws, OFF_DATA);
  const unsigned voff = lane * 16;
  const unsigned ldsw = (unsigned)wid * 2048u;
  const unsigned xw = (unsigned)wid * XREG, ww = XALL + (unsigned)wid * WREG;
  const float cz = p.sx[0] * p.sw[0] * LOG2E;
#define STAGE(bufoff, soff) do { \
    __builtin_amdgcn_raw_ptr_buffer_load_lds(rs, (LAS void*)(lds + (bufoff) + ldsw), 16, voff, (soff), 0, 17); \
    __builtin_amdgcn_raw_ptr_buffer_load_lds(rs, (LAS void*)(lds + (bufoff) + ldsw), 16, voff, (soff), 1024, 17); } while (0)

  const unsigned voffE = (unsigned)(lane >> 4) * 512u + (unsigned)(lane & 15) * 16u;
#define STAGEE(buf, soff) __builtin_amdgcn_raw_ptr_buffer_load_lds(rs, (LAS void*)(lds + (buf) + wid * 1024), 16, voffE, (soff), 0, 17)
#define EBASE(q) (ww + (unsigned)(NSLOT * NROUND + (p.rank >> 1)) * (NT * 4096) + (unsigned)((p.rank & 1) * 2048) + (unsigned)((q) & 1) * 256)
  if (!is_copy) {
    const int gidx = bid - NCOPY, xcd = gidx & 7, idx = gidx >> 3, tl = (xcd >> 1) * 30 + idx;
    const int pm = (xcd & 1) * 8 + (tl & 7), slot = tl >> 3;
    local_role(p, call, (unsigned)gidx * 8u + (unsigned)wid);
    const LAS unsigned char* pA = lds + fq * 2048 + wr * 256 + fr * 16;
    const LAS unsigned char* pB = lds + 4 * HTB + fq * 2048 + wc * 512 + fr * 16;
    const LAS unsigned char* pE = lds + LDS_E + fq * 1024 + fr * 16;
    asm volatile("" : "+v"(pA), "+v"(pB), "+v"(pE));
#define SA(b, h) (((b) * 2 + (h)) * HTB)
#define SB(b, h) ((4 + (b) * 2 + (h)) * HTB)
#define LDA(dst, b, h) do { _Pragma("unroll") for (int m = 0; m < 4; ++m) _Pragma("unroll") for (int k = 0; k < 2; ++k) dst[m][k] = *(const LAS i32x4*)(pA + SA(b, h) + m * 512 + k * 8192); } while (0)
#define LDB(dst, b, h) do { _Pragma("unroll") for (int n = 0; n < 2; ++n) _Pragma("unroll") for (int k = 0; k < 2; ++k) dst[n][k] = *(const LAS i32x4*)(pB + SA(b, h) + n * 256 + k * 8192); } while (0)
#define LDE(dst, b) do { _Pragma("unroll") for (int m = 0; m < 4; ++m) _Pragma("unroll") for (int k = 0; k < 2; ++k) dst[m][k] = *(const LAS i32x4*)(pE + (b) * 8192 + m * 256 + k * 4096); } while (0)
#define MIDX(q) (((q) >> 2) ? 3 - ((q) & 3) : ((q) & 3))
#define NIDX(q) ((q) >> 2)
#define MM8(C, At, Bt) do { __builtin_amdgcn_s_setprio(1); _Pragma("unroll") for (int q = 0; q < 8; ++q) { const int m = MIDX(q), n = NIDX(q); \
        asm volatile("v_mfma_f32_16x16x128_f8f6f4 %0, %1, %2, %0" : "+v"(C[m][n]) : "v"(cat(Bt[n][0], Bt[n][1])), "v"(cat(At[m][0], At[m][1]))); } __builtin_amdgcn_s_setprio(0); } while (0)
#define MM8Z(C, At, Bt) do { __builtin_amdgcn_s_setprio(1); _Pragma("unroll") for (int q = 0; q < 8; ++q) { const int m = MIDX(q), n = NIDX(q); \
        asm volatile("v_mfma_f32_16x16x128_f8f6f4 %0, %1, %2, 0" : "=&v"(C[m][n]) : "v"(cat(Bt[n][0], Bt[n][1])), "v"(cat(At[m][0], At[m][1]))); } __builtin_amdgcn_s_setprio(0); } while (0)
#define SELB() do { _Pragma("unroll") for (int n = 0; n < 2; ++n) _Pragma("unroll") for (int k = 0; k < 2; ++k) { B0[n][k] = wr ? B1[n][k] : B0[n][k]; asm volatile("" : "+v"(B0[n][k])); } } while (0)
#define MME(At) MM8(accE, At, B0)
#define MMEZ(At) MM8Z(accE, At, B0)
#define MMA(ai, bj, At, Bt) MM8(acc[ai][bj], At, Bt)
#define MMA0(ai, bj, At, Bt) MM8Z(acc[ai][bj], At, Bt)
    f32x4 acc[2][2][4][2], accE[4][2];
    i32x4 At[4][2], B0[2][2], B1[2][2];
    const unsigned kstep = 4096, hstep = 2048, tstep = NT * 4096;
    const unsigned cB = xw + (unsigned)pm * tstep;
    unsigned cA = ww + (unsigned)slot * tstep;
    const bool ex = slot < 2;
    const unsigned cE = EBASE(slot);
    const int l8 = lane & 7;
    const unsigned foff = OFF_FLAGS + (unsigned)wid * FLAG_SRC + (l8 < 2 ? pm * 2 + l8 : l8 == 4 ? (16 + NSLOT * NROUND) * 2 + p.rank : (16 + slot) * 2 + (l8 == 3)) * 64;
    constexpr unsigned long long IGN_XE = 0x1313131313131313ull, IGN_E = 0x1010101010101010ull;
    spin_flags(myws, foff, ex ? 0ull : IGN_E, call);
    unsigned fnew = call - 1u, tm = 0u;
    unsigned sA = 4u, sB = 8u;
    STAMP(1);
    if (ex) STAGEE(LDS_E, cE);
    STAGE(SB(0, 0), cB); STAGE(SA(0, 0), cA); STAGE(SB(0, 1), cB + hstep); STAGE(SA(0, 1), cA + hstep);
    if (wr == 1) BAR;
    WAIT_V(4); BAR;
    STAGE(SB(1, 0), cB + kstep); STAGE(SA(1, 0), cA + kstep); STAGE(SB(1, 1), cB + hstep + kstep); STAGE(SA(1, 1), cA + hstep + kstep);
    WAIT_V(8); BAR;
#define FCHECK(ign) do { if ((__ballot((int)(fnew - call) >= 0) | (ign)) != ~0ull) spin_flags(myws, foff + sA, (ign), call); } while (0)
#define ITSTAMP()
#define NEXTLEV() do { sA = sB; sB += (sB & 63u) == 60u ? FLAG_ROUND - 60u : 4u; } while (0)
#define ITER(EX, FIRST, MM, MMX, W4, W8, FCHK, a1h, a2, b2, e1, e2, LAST) do { \
        const unsigned a3 = (a2) + kstep, b3 = (b2) + kstep; \
          \
        LDB(B0, 0, 0); SCHED; LDA(At, 0, 0); if (!(FIRST)) STAGE(SA(1, 1), a1h); \
        WAIT_L(8); BAR; WAIT_L(0); MM(0, 0, At, B0); BAR; SCHED; \
          \
        FCHK; \
        LDB(B1, 0, 1); STAGE(SB(0, 0), b2); if (EX) STAGEE(LDS_E + 8192, e1); \
        BAR; WAIT_L(0); MM(0, 1, At, B1); BAR; \
          \
        LDA(At, 0, 1); STAGE(SA(0, 0), a2); \
        BAR; WAIT_L(0); MM(1, 0, At, B0); BAR; SCHED; \
          \
        STAGE(SB(0, 1), (b2) + hstep); \
        W4; \
        BAR; MM(1, 1, At, B1); BAR; \
        if (EX) { SCHED; LDE(At, 0); SELB(); BAR; WAIT_L(0); MMX(At); BAR; SCHED; } \
          \
        LDB(B0, 1, 0); SCHED; LDA(At, 1, 0); STAGE(SA(0, 1), (a2) + hstep); \
        WAIT_L(8); BAR; WAIT_L(0); MMA(0, 0, At, B0); BAR; SCHED; \
          \
        LDB(B1, 1, 1); STAGE(SB(1, 0), b3); if (EX) STAGEE(LDS_E, e2); \
        fnew = __builtin_amdgcn_raw_buffer_load_b32(rsf, foff, sB, 17); \
        if (LAST) tm = __builtin_amdgcn_raw_buffer_load_b32(rsf, OFF_TMO, 0, 17);        \
        BAR; WAIT_L(0); MMA(0, 1, At, B1); BAR; \
          \
        LDA(At, 1, 1); STAGE(SA(1, 0), a3); \
        BAR; WAIT_L(0); MMA(1, 0, At, B0); BAR; SCHED; \
          \
        STAGE(SB(1, 1), b3 + hstep); \
        NEXTLEV(); ITSTAMP(); \
        W8; BAR; MMA(1, 1, At, B1); BAR; \
        if (EX) { SCHED; LDE(At, 1); SELB(); BAR; WAIT_L(0); MME(At); BAR; SCHED; } } while (0)
#define EPILOGUE(r, AUX) do { \
        int tz = threadIdx.x; asm volatile("" : "+v"(tz)); \
        const int lane2 = tz & 63, fr2 = lane2 & 15, fq2 = lane2 >> 4; \
        const int pn = NSLOT * (r) + slot; \
        const unsigned ovoff = (unsigned)(wc * 32 + fq2 * 4) * (N * 4) + (unsigned)(wr * 64 + fr2 * 4) * 4; \
        const unsigned osoff = (unsigned)(pm * 256) * (N * 4) + (unsigned)pn * 1024; \
        const bool bad = rfl(tm) == call; \
        const float czz = bad ? __builtin_nanf("") : cz; \
        _Pragma("unroll") for (int ai = 0; ai < 2; ++ai) _Pragma("unroll") for (int bj = 0; bj < 2; ++bj) _Pragma("unroll") for (int n = 0; n < 2; ++n) { \
              const f32x4 a4[4] = {acc[ai][bj][0][n], acc[ai][bj][1][n], acc[ai][bj][2][n], acc[ai][bj][3][n]}; \
              f32x4 o4[4]; \
              silu_rows(a4, czz, o4); \
              store_rows<AUX>(o4, ro, ovoff, osoff + (unsigned)(bj * 128 + n * 16) * (N * 4) + ai * 512); } } while (0)
    if (ex) {
      const unsigned nA = cA + NSLOT * tstep;
      ITER(true, true, MMA0, MMEZ, WAIT_V(7), WAIT_V(8), FCHECK(0ull), 0u, cA + 2 * kstep, cB + 2 * kstep, cE + kstep, cE + 2 * kstep, false);
      for (int t = 2; t < NT - 2; t += 2) {
        ITER(true, false, MMA, MME, WAIT_V(7), WAIT_V(8), FCHECK(0ull), cA + (unsigned)(t + 1) * kstep + hstep, cA + (unsigned)(t + 2) * kstep, cB + (unsigned)(t + 2) * kstep,
             cE + (unsigned)(t + 1) * kstep, cE + (unsigned)(t + 2) * kstep, false);
      }
      ITER(true, false, MMA, MME, WAIT_V(7), WAIT_V(8), FCHECK(IGN_XE), cA + (unsigned)(NT - 1) * kstep + hstep, nA, cB, cE + (unsigned)(NT - 1) * kstep, cE, true);
      asm volatile("s_nop 15" ::: "memory");
      STAMP(2);
      {
        int tz = threadIdx.x; asm volatile("" : "+v"(tz));
        const int lane2 = tz & 63, fr2 = lane2 & 15, fq2 = lane2 >> 4;
        const unsigned ovoff = (unsigned)(wr * 128 + wc * 32 + fq2 * 4) * (N * 4) + (unsigned)(fr2 * 4) * 4;
        const unsigned osoff = (unsigned)(pm * 256) * (N * 4) + (unsigned)(NSLOT * NROUND * 256 + p.rank * 128 + slot * 64) * 4;
        const float czz = rfl(tm) == call ? __builtin_nanf("") : cz;
#pragma unroll
        for (int n = 0; n < 2; ++n) {
          const f32x4 a4[4] = {accE[0][n], accE[1][n], accE[2][n], accE[3][n]};
          f32x4 o4[4];
          silu_rows(a4, czz, o4);
          store_rows<17>(o4, ro, ovoff, osoff + (unsigned)(n * 16) * (N * 4));
        }
      }
      STAGE(SA(1, 1), nA + kstep + hstep);
      EPILOGUE(0, 2);
      STAMP(3);
      cA = nA;
    }
    for (int r = ex ? 1 : 0; r < NROUND; ++r) {
      const bool has_next = r + 1 < NROUND;
      const unsigned long long ign = r == 0 ? IGN_E : IGN_XE;
      const unsigned nA = has_next ? cA + NSLOT * tstep : cA;
      ITER(false, true, MMA0, MMEZ, if (r > 0) { WAIT_V(38); if (ex && r == 1 && lane == 0) __hip_atomic_store((gu32*)(myws + OFF_OFLAG + ((pm * 2 + slot) * 8 + wid) * 4), call, __ATOMIC_RELAXED, __HIP_MEMORY_SCOPE_SYSTEM); } else WAIT_V(6),
           WAIT_V(7), FCHECK(ign), 0u, cA + 2 * kstep, cB + 2 * kstep, 0u, 0u, false);
      for (int t = 2; t < NT - 2; t += 2) {
        ITER(false, false, MMA, MME, WAIT_V(6), WAIT_V(7), FCHECK(ign), cA + (unsigned)(t + 1) * kstep + hstep, cA + (unsigned)(t + 2) * kstep, cB + (unsigned)(t + 2) * kstep, 0u, 0u, false);
      }
      const unsigned long long ignl = has_next ? IGN_XE : ~0ull;
      ITER(false, false, MMA, MME, WAIT_V(6), WAIT_V(7), FCHECK(ignl), cA + (unsigned)(NT - 1) * kstep + hstep, nA, cB, 0u, 0u, true);
      STAGE(SA(1, 1), nA + kstep + hstep);
      asm volatile("s_nop 15" ::: "memory");
      STAMP(2 + 2 * r);
      EPILOGUE(r, 2);
      STAMP(3 + 2 * r);
      cA = nA;
    }
    WAIT_V(0);
    if (wr == 0) BAR;
  }
  STAMP(11);
  if (threadIdx.x == 0) __hip_atomic_fetch_add((gu32*)(myws + OFF_CNT), 1u, __ATOMIC_RELAXED, __HIP_MEMORY_SCOPE_AGENT);
}
}
#define HIP_CHECK(expr) do { hipError_t _e = (expr); if (_e != hipSuccess) { \
    fprintf(stderr, "HIP error %s at %s:%d\n", hipGetErrorString(_e), __FILE__, __LINE__); \
    return; } } while (0)

extern "C" void kernel_launch(int rank, int nranks, ncclComm_t comm, hipStream_t stream,
                              void* const* d_in, const int* in_sizes, int n_in,
                              void* d_out, int out_size, const int* in_dtypes, int out_dtype,
                              void* const* d_out_all, void* const* d_ws_all, size_t ws_bytes) {
  (void)comm; (void)in_dtypes; (void)out_dtype; (void)d_out_all;
  if (nranks != gar::NR || n_in != 4 || (unsigned)rank >= (unsigned)gar::NR || ws_bytes < gar::WS_NEED ||
      in_sizes[0] != gar::M * gar::KS || in_sizes[1] != gar::N * gar::KS || out_size != gar::M * gar::N) {
    fprintf(stderr, "kernel_launch: unexpected shapes\n");
    return;
  }
  static bool attr_set = false;
  if (!attr_set) {
    HIP_CHECK(hipFuncSetAttribute((const void*)gar::gemm_ag_kernel, hipFuncAttributeMaxDynamicSharedMemorySize, gar::LDS_BYTES));
    attr_set = true;
  }
  gar::Params p;
  p.x = (const gar::u8*)d_in[0]; p.w = (const gar::u8*)d_in[1];
  p.sx = (const float*)d_in[2]; p.sw = (const float*)d_in[3];
  p.out = (float*)d_out;
  for (int i = 0; i < gar::NR; ++i) p.ws[i] = (gar::u8*)d_ws_all[i];
  for (int i = 0; i < gar::NR; ++i) p.outs[i] = (float*)d_out_all[i];
  p.rank = rank; p.pad = 0;
  gar::gemm_ag_kernel<<<gar::GRID, 512, gar::LDS_BYTES, stream>>>(p);
  HIP_CHECK(hipGetLastError());
}
```

```cpp
#include <hip/hip_runtime.h>
#include <rccl/rccl.h>
#include <cstdio>

extern "C" __device__ float gar_fmul_legacy(float, float) __asm("llvm.amdgcn.fmul.legacy");
namespace gar {
__device__ __forceinline__ float fmul_single(float a, float b) { return gar_fmul_legacy(a, b); }
#define LAS __attribute__((address_space(3)))
typedef int i32x4 __attribute__((ext_vector_type(4)));
typedef int i32x8 __attribute__((ext_vector_type(8)));
typedef unsigned u32x4 __attribute__((ext_vector_type(4)));
typedef float f32x4 __attribute__((ext_vector_type(4)));
typedef float f32x2 __attribute__((ext_vector_type(2)));
typedef unsigned char u8;
typedef __amdgpu_buffer_rsrc_t rsrc_t;
typedef __attribute__((address_space(1))) unsigned gu32;

constexpr int M = 4096, N = 16384, NR = 8, KS = 512;
constexpr int NT = 32;
constexpr int HTB = 16384;
constexpr int LDS_E = 8 * HTB;
constexpr int LDS_BYTES = 8 * HTB + 16384;
constexpr int GRID = 256, NCOPY = 16, NSLOT = 15, NROUND = 4;
#define NCW 4
constexpr unsigned NCWAVE = NCOPY * NCW;
#define GAR_OUTS 1
constexpr unsigned NOPER = 2432, NUNIT0 = 992, NUNIT = NOPER + 512;
constexpr unsigned NLOCAL = NOPER + 128;
constexpr unsigned OFF_CNT = 0, OFF_TMO = 256, OFF_QUEUE = 512;
constexpr unsigned OFF_OFLAG = 4096, OFF_DONE = 8192;
constexpr unsigned OFF_FLAGS = 65536;
constexpr unsigned FLAG_SRC = 80 * 128, FLAG_ROUND = NSLOT * 128;
constexpr unsigned OFF_DATA = 1u << 20;
constexpr unsigned XREG = (unsigned)M * KS;
constexpr unsigned WREG = (unsigned)N * KS;
constexpr unsigned XALL = NR * XREG, WALL = NR * WREG;
constexpr unsigned PAR_BYTES = XALL + WALL;
constexpr size_t WS_NEED = (size_t)OFF_DATA + 2 * (size_t)PAR_BYTES;
constexpr unsigned long long DEADLINE = 30000000ull;
constexpr int RSRC3 = 0x00020000;
#define SLEEPV 8
#define OUTP 1
#define STAGN 2
#define LDAUX 17
#define KPRE 7

struct Params {
  const u8* x; const u8* w; const float* sx; const float* sw; float* out;
  u8* ws[NR];
  float* outs[NR];
  int rank; int pad;
};

__device__ __forceinline__ i32x8 cat(i32x4 a, i32x4 b) { return __builtin_shufflevector(a, b, 0, 1, 2, 3, 4, 5, 6, 7); }
__device__ __forceinline__ unsigned rfl(unsigned v) { return __builtin_amdgcn_readfirstlane(v); }
__device__ __forceinline__ rsrc_t mk_rsrc(const void* p, unsigned bytes) {
  const unsigned long long a = (unsigned long long)p;
  const unsigned lo = rfl((unsigned)a), hi = rfl((unsigned)(a >> 32));
  return __builtin_amdgcn_make_buffer_rsrc((void*)(((unsigned long long)hi << 32) | lo), (short)0, (int)bytes, RSRC3);
}
#define STR2(x) #x
#define STR(x) STR2(x)
#define WAIT_V(n) asm volatile("s_waitcnt vmcnt(" STR(n) ")" ::: "memory")
#define WAIT_L(n) asm volatile("s_waitcnt lgkmcnt(" STR(n) ")" ::: "memory")
#define BAR __builtin_amdgcn_s_barrier()
#define SCHED __builtin_amdgcn_sched_barrier(0)
#define STAMP(i)
#define CSTAMP(e, v)

__device__ __noinline__ void spin_flags(u8* ws, unsigned off, unsigned long long ign, unsigned call) {
  gu32* tmo = (gu32*)(ws + OFF_TMO);
  gu32* fp = (gu32*)(ws + off);
  const unsigned long long t0 = __builtin_amdgcn_s_memrealtime();
  for (;;) {
    const unsigned v = __hip_atomic_load(fp, __ATOMIC_RELAXED, __HIP_MEMORY_SCOPE_SYSTEM);
    unsigned t = __hip_atomic_load(tmo, __ATOMIC_RELAXED, __HIP_MEMORY_SCOPE_AGENT);
    asm volatile("" : "+v"(t));
    if ((__ballot((int)(v - call) >= 0) | ign) == ~0ull) return;
    if (rfl(t) == call || __builtin_amdgcn_s_memrealtime() - t0 > DEADLINE) {
      __hip_atomic_store(tmo, call, __ATOMIC_RELAXED, __HIP_MEMORY_SCOPE_AGENT);
      WAIT_V(0);
      return;
    }
    __builtin_amdgcn_s_sleep(SLEEPV);
  }
}

#define XDL 5, 2, 5, 2, 5, 2, 5, 2, 3, 0, 3, 0, 3, 0, 3, 0
struct XSched { unsigned long long perm, del, cnt; unsigned last; };
constexpr XSched make_xsched() {
  const unsigned d[16] = {XDL};
  XSched s{0ull, 0ull, 0ull, 0u};
  unsigned n = 0;
  for (unsigned v = 0; v < 12; ++v) {
    for (unsigned i = 0; i < 16; ++i) if (d[i] == v) { s.perm |= (unsigned long long)i << (4 * n); s.del |= (unsigned long long)v << (4 * n); s.last = v; ++n; }
    s.cnt |= (unsigned long long)n << (5 * v);
  }
  return s;
}
constexpr XSched XS = make_xsched();
static_assert(((XS.cnt >> 55) & 31) == 16 && (XS.del & 15) == 0, "delays 0..11, the first 0");
struct Unit { unsigned src, dst, flag; int kind; };
__device__ __forceinline__ Unit decode_unit(unsigned u) {
  unsigned panel, h, lev; bool isx = false;
  if (u < NOPER) {
    unsigned v = 0u, jj = u, g0 = 0u, nx = 0u;
    constexpr unsigned VA = 16u + XS.last, NA = 512u + VA * 30u;
#define XCNT(t) ((t) >= 11u ? 16u : (unsigned)(XS.cnt >> (5u * (t))) & 31u)
    if (u < NA) {
      for (;; ++v) {
        g0 = v < 16u ? 0u : XCNT(v - 16u);
        nx = 2u * (XCNT(v) - g0);
        if (jj < nx + 30u) break;
        jj -= nx + 30u;
      }
    } else { const unsigned t = u - NA; v = t / 30u; jj = t - v * 30u; v += VA; }
    isx = jj < nx;
    const unsigned si = 4u * ((g0 + (jj >> 1)) & 15u), jw = jj - nx;
    h = jj & 1u;
    panel = isx ? (unsigned)(XS.perm >> si) & 15u : NSLOT * (v >> 4) + (jw >> 1);
    lev = isx ? v - ((unsigned)(XS.del >> si) & 15u) : v & 15u;
  } else {
    const unsigned v = u - NOPER;
    lev = v & 15u; h = (v >> 4) & 1u; panel = NSLOT * NROUND + (v >> 5);
  }
  Unit o; o.kind = isx ? 0 : 1;
  o.src = isx ? (panel * 256u + h * 128u) * KS + lev * 32u : lev * 32u * N + panel * 256u + h * 128u;
  o.dst = (isx ? 0u : XALL) + panel * (NT * 4096u) + lev * 8192u + h * 2048u;
  o.flag = (((isx ? 0u : 16u) + panel) * 2u + h) * 64u + lev * 4u;
  return o;
}
__device__ __forceinline__ Unit decode_queue(unsigned u) {
  if (u < NOPER) return decode_unit(u);
  const unsigned v = (u < NUNIT ? u : NUNIT - 1u) - NOPER;
  Unit o; o.kind = 2; o.flag = ~0u;
  o.src = o.dst = ((v >> 5) * 256u + ((v >> 1) & 7u) * 32u + (v & 1u) * 16u) * (N * 4u) + ((v >> 4) & 1u) * 256u;
  return o;
}
__device__ __forceinline__ void unit_load(const Unit& u, rsrc_t srcx, rsrc_t srcw, int lane, unsigned (&d)[16]) {
  if (u.kind == 0) {
#pragma unroll
    for (int t = 0; t < 4; ++t) { const u32x4 v = __builtin_amdgcn_raw_buffer_load_b128(srcx, (lane & 31) * KS + (lane >> 5) * 16, u.src + t * 32 * KS, 0);
      d[4 * t] = v[0]; d[4 * t + 1] = v[1]; d[4 * t + 2] = v[2]; d[4 * t + 3] = v[3]; }
  } else {
#pragma unroll
    for (int i = 0; i < 16; ++i) d[i] = __builtin_amdgcn_raw_buffer_load_b32(srcw, (lane >> 5) * 16 * N + (lane & 31) * 4, u.src + i * N, 0);
  }
}
__device__ __forceinline__ void unit_retile(const Unit& u, const unsigned (&d)[16], u32x4 (&o)[4]) {
  if (u.kind != 1) {
#pragma unroll
    for (int t = 0; t < 4; ++t) o[t] = (u32x4){d[4 * t], d[4 * t + 1], d[4 * t + 2], d[4 * t + 3]};
  } else {
#pragma unroll
    for (int q = 0; q < 4; ++q) {
      const unsigned t0 = __builtin_amdgcn_perm(d[4 * q + 1], d[4 * q], 0x05010400u), t1 = __builtin_amdgcn_perm(d[4 * q + 1], d[4 * q], 0x07030602u);
      const unsigned t2 = __builtin_amdgcn_perm(d[4 * q + 3], d[4 * q + 2], 0x05010400u), t3 = __builtin_amdgcn_perm(d[4 * q + 3], d[4 * q + 2], 0x07030602u);
      o[0][q] = __builtin_amdgcn_perm(t2, t0, 0x05040100u); o[1][q] = __builtin_amdgcn_perm(t2, t0, 0x07060302u);
      o[2][q] = __builtin_amdgcn_perm(t3, t1, 0x05040100u); o[3][q] = __builtin_amdgcn_perm(t3, t1, 0x07060302u);
    }
  }
}
__device__ __forceinline__ unsigned call_of(unsigned counter) { return rfl(counter) / GRID + 1u; }
__device__ __forceinline__ unsigned local_role(const Params& p, unsigned counter, unsigned gw) {
  const int lane = threadIdx.x & 63;
  const unsigned rank = (unsigned)p.rank;
  const rsrc_t srcx = mk_rsrc(p.x, (unsigned)M * KS), srcw = mk_rsrc(p.w, (unsigned)N * KS);
  constexpr unsigned NGW = (GRID - NCOPY) * 8;
  const unsigned ua = gw < 128u ? NOPER + gw : gw - 128u, ub = gw + NGW - 128u;
  const bool two = ub < NOPER;
  const Unit a = decode_unit(ua), b = decode_unit(two ? ub : 0u);
  unsigned da[16], db[16]; u32x4 o[4];
  unit_load(a, srcx, srcw, lane, da);
  if (two) unit_load(b, srcx, srcw, lane, db);
  SCHED;
  const unsigned call = call_of(counter);
  const unsigned dvo = OFF_DATA + (call & 1u) * PAR_BYTES + (lane >> 5) * 4096 + (lane & 31) * 16;
  u8* wa = p.ws[gw < 128u ? gw >> 4 : rank];
  {
    const rsrc_t dst = mk_rsrc(wa, (unsigned)WS_NEED);
    unit_retile(a, da, o);
    const unsigned voff = dvo + a.dst + rank * (a.kind == 0 ? XREG : WREG);
#pragma unroll
    for (int q = 0; q < 4; ++q) __builtin_amdgcn_raw_buffer_store_b128(o[q], dst, voff + q * 512, 0, 17);
  }
  if (two) {
    const rsrc_t dst = mk_rsrc(p.ws[rank], (unsigned)WS_NEED);
    unit_retile(b, db, o);
    const unsigned voff = dvo + b.dst + rank * (b.kind == 0 ? XREG : WREG);
#pragma unroll
    for (int q = 0; q < 4; ++q) __builtin_amdgcn_raw_buffer_store_b128(o[q], dst, voff + q * 512, 0, 17);
  }
  WAIT_V(0);
  if (lane == 0) {
    __hip_atomic_store((gu32*)(wa + OFF_FLAGS + rank * FLAG_SRC + a.flag), call, __ATOMIC_RELAXED, __HIP_MEMORY_SCOPE_SYSTEM);
    if (two) __hip_atomic_store((gu32*)(p.ws[rank] + OFF_FLAGS + rank * FLAG_SRC + b.flag), call, __ATOMIC_RELAXED, __HIP_MEMORY_SCOPE_SYSTEM);
  }
  return call;
}
__device__ __forceinline__ unsigned copy_role(const Params& p, unsigned counter, int j) {
  const int tid = threadIdx.x, g = rfl(tid >> 6), lane = tid & 63;
  if (g >= NCW) return call_of(counter);
  const unsigned rank = (unsigned)p.rank;
  unsigned cur = (unsigned)j * NCW + (unsigned)g, nxt = cur + NCWAVE;
  const rsrc_t srcx = mk_rsrc(p.x, (unsigned)M * KS), srcw = mk_rsrc(p.w, (unsigned)N * KS), srco = mk_rsrc(p.out, (unsigned)M * N * 4u);
  unsigned d[16]; u32x4 o[4];
  Unit uc = decode_unit(cur);
  unit_load(uc, srcx, srcw, lane, d);
  u8* myws = p.ws[rank];
  const rsrc_t rq = mk_rsrc(myws, OFF_FLAGS);
  unsigned qvo = lane == 0 ? OFF_QUEUE : 0x80000000u;
  asm volatile("" : "+v"(qvo));
  const unsigned wave = cur;
  rsrc_t dst[NR - 1], dsto[NR - 1];
#pragma unroll
  for (int d = 0; d < NR - 1; ++d) { dst[d] = mk_rsrc(p.ws[(rank + 1 + d) & 7], (unsigned)WS_NEED); dsto[d] = mk_rsrc(p.outs[(rank + 1 + d) & 7], (unsigned)M * N * 4u); }
  u8* flagp = p.ws[(rank + 1 + lane) & 7] + OFF_FLAGS + rank * FLAG_SRC;
  const unsigned ovo = (unsigned)(lane >> 4) * (N * 4) + (unsigned)(lane & 15) * 16 + (NSLOT * NROUND * 256 + rank * 128) * 4;
  SCHED;
  const unsigned call = call_of(counter);
  const unsigned qbase = (call - 1u) * NUNIT - 2u * NCWAVE;
  const unsigned dvo = OFF_DATA + (call & 1u) * PAR_BYTES + (lane >> 5) * 4096 + (lane & 31) * 16;
#define PUSH(q0, q1) do { if (uc.kind == 2) { _Pragma("unroll") for (int q = q0; q < q1; ++q) __builtin_amdgcn_raw_buffer_store_b128(o[q / 7], dsto[q % 7], voff + (q / 7) * 4 * (N * 4), 0, 17); } \
    else { _Pragma("unroll") for (int q = q0; q < q1; ++q) __builtin_amdgcn_raw_buffer_store_b128(o[q / 7], dst[q % 7], voff + (q / 7) * 512, 0, 17); } } while (0)
#define PUBLISH(fl) do { if (lane < NR - 1) __hip_atomic_store((gu32*)(flagp + (fl)), call, __ATOMIC_RELAXED, __HIP_MEMORY_SCOPE_SYSTEM); } while (0)
  unsigned pflag = ~0u;
  bool oready = false;
  for (;;) {
    const Unit un = decode_queue(nxt);
    SCHED;
    unit_retile(uc, d, o);
    const unsigned voff = uc.kind == 2 ? ovo + uc.dst : dvo + uc.dst + rank * (uc.kind == 0 ? XREG : WREG);
    SCHED;
    PUSH(0, KPRE);
    WAIT_V(KPRE);
    if (pflag != ~0u) PUBLISH(pflag);
    pflag = uc.flag;
    SCHED;
    if (!oready && nxt >= NOPER - 480u + wave * 6u) {
      const u32x4 of = __builtin_amdgcn_raw_buffer_load_b128(rq, OFF_OFLAG + lane * 16, 0, 17);
      if (!__all((int)(of[0] - call) >= 0 && (int)(of[1] - call) >= 0 && (int)(of[2] - call) >= 0 && (int)(of[3] - call) >= 0))
        for (int q = 0; q < 4; ++q) spin_flags(myws, OFF_OFLAG + (q * 64 + lane) * 4, 0ull, call);
      oready = true;
    }
    if (un.kind == 2) {
#pragma unroll
      for (int t = 0; t < 4; ++t) { const u32x4 v = __builtin_amdgcn_raw_buffer_load_b128(srco, ovo, un.src + t * 4 * (N * 4), 17);
        d[4 * t] = v[0]; d[4 * t + 1] = v[1]; d[4 * t + 2] = v[2]; d[4 * t + 3] = v[3]; }
    } else unit_load(un, srcx, srcw, lane, d);
    const unsigned ticket = (unsigned)__builtin_amdgcn_raw_ptr_buffer_atomic_add_i32(1, rq, qvo, 0, 16);
    SCHED;
    PUSH(KPRE, 28);
    SCHED;
    cur = nxt; uc = un;
    if (cur >= NUNIT) break;
    nxt = rfl(ticket) - qbase;
  }
  WAIT_V(0);
  if (pflag != ~0u) PUBLISH(pflag);
  if (lane < NR) __hip_atomic_store((gu32*)(p.ws[lane & 7] + OFF_DONE + (rank * 64u + wave) * 4u), call, __ATOMIC_RELAXED, __HIP_MEMORY_SCOPE_SYSTEM);
  CSTAMP(24 + g, __builtin_amdgcn_s_memrealtime());
  return call;
}

template <int AUX> __device__ __forceinline__ void store_rows(const f32x4 (&o)[4], rsrc_t r, unsigned voff, unsigned soff) {
  SCHED;
#pragma unroll
  for (int j = 0; j < 4; ++j) {
    __builtin_amdgcn_raw_buffer_store_b128(__builtin_bit_cast(u32x4, o[j]), r, voff, soff + (unsigned)j * (N * 4), AUX);
  }
  asm volatile("s_nop 1" ::: "memory");
  SCHED;
}
constexpr float LOG2E = 1.4426950408889634f, CLIP2 = 86.56170245333781f;
__device__ __forceinline__ void silu_rows(const f32x4 (&a)[4], float cz, f32x4 (&o)[4]) {
#pragma unroll
  for (int h = 0; h < 2; ++h) {
    f32x2 z[4], r[4];
#pragma unroll
    for (int m = 0; m < 4; ++m) {
      z[m] = (f32x2){a[m][2 * h], a[m][2 * h + 1]} * cz;
      f32x2 e;
      e.x = __builtin_amdgcn_exp2f(__builtin_amdgcn_fmed3f(-z[m].x, -CLIP2, CLIP2));
      e.y = __builtin_amdgcn_exp2f(__builtin_amdgcn_fmed3f(-z[m].y, -CLIP2, CLIP2));
      const f32x2 dd = __builtin_elementwise_fma(e, (f32x2){LOG2E, LOG2E}, (f32x2){LOG2E, LOG2E});
      r[m].x = __builtin_amdgcn_rcpf(dd.x); r[m].y = __builtin_amdgcn_rcpf(dd.y);
    }
#pragma unroll
    for (int m = 0; m < 4; ++m) {
      o[2 * h][m] = fmul_single(z[m].x, r[m].x); o[2 * h + 1][m] = fmul_single(z[m].y, r[m].y); }
  }
}

__global__ void __launch_bounds__(512, 2) gemm_ag_kernel(Params p) {
  extern __shared__ __attribute__((aligned(16))) unsigned char shm[];
  LAS unsigned char* lds = (LAS unsigned char*)shm;
  u8* myws = p.ws[p.rank];
  const unsigned counter = __hip_atomic_load((gu32*)(myws + OFF_CNT), __ATOMIC_RELAXED, __HIP_MEMORY_SCOPE_AGENT);
  const int bid = blockIdx.x;
#define XMAP 0
  const bool is_copy = bid < NCOPY;
  const int cj = bid;
  STAMP(0);
  if (is_copy) {
    const unsigned call = copy_role(p, counter, cj);
    if (cj == 0) {
      BAR;
      spin_flags(myws, OFF_DONE + (threadIdx.x >> 6) * 256u + (threadIdx.x & 63u) * 4u, 0ull, call);
      if (rfl(__hip_atomic_load((gu32*)(myws + OFF_TMO), __ATOMIC_RELAXED, __HIP_MEMORY_SCOPE_AGENT)) == call && (threadIdx.x & 63u) == 0u) p.out[0] = __builtin_nanf("");
    }
    STAMP(1);
  }

  const int tid = threadIdx.x, wid = rfl(tid >> 6), lane = tid & 63, wr = wid >> 2, wc = wid & 3, fr = lane & 15, fq = lane >> 4;
  const rsrc_t ro = mk_rsrc(p.out, (unsigned)M * N * 4u), rsf = mk_rsrc(myws, OFF_DATA);
  const unsigned voff = lane * 16;
  const unsigned ldsw = (unsigned)wid * 2048u;
  const unsigned xw = (unsigned)wid * XREG, ww = XALL + (unsigned)wid * WREG;
  const float cz = p.sx[0] * p.sw[0] * LOG2E;
#define STAGE(bufoff, soff) do { \
    __builtin_amdgcn_raw_ptr_buffer_load_lds(rs, (LAS void*)(lds + (bufoff) + ldsw), 16, voff, (soff), 0, LDAUX); \
    __builtin_amdgcn_raw_ptr_buffer_load_lds(rs, (LAS void*)(lds + (bufoff) + ldsw), 16, voff, (soff), 1024, LDAUX); } while (0)

  const unsigned voffE = (unsigned)(lane >> 4) * 512u + (unsigned)(lane & 15) * 16u;
#define STAGEE(buf, soff) __builtin_amdgcn_raw_ptr_buffer_load_lds(rs, (LAS void*)(lds + (buf) + wid * 1024), 16, voffE, (soff), 0, LDAUX)
#define EBASE(q) (ww + (unsigned)(NSLOT * NROUND + (p.rank >> 1)) * (NT * 4096) + (unsigned)((p.rank & 1) * 2048) + (unsigned)((q) & 1) * 256)
  if (!is_copy) {
    const int gidx = bid - NCOPY, xcd = gidx & 7, idx = gidx >> 3, tl = (xcd >> 1) * 30 + idx;
    const int pm = (xcd & 1) * 8 + (tl & 7), slot = tl >> 3;
    const unsigned call = local_role(p, counter, (unsigned)gidx * 8u + (unsigned)wid);
    const rsrc_t rs = mk_rsrc(myws + OFF_DATA + (call & 1u) * PAR_BYTES, PAR_BYTES);
    const LAS unsigned char* pA = lds + fq * 2048 + wr * 256 + fr * 16;
    const LAS unsigned char* pB = lds + 4 * HTB + fq * 2048 + wc * 512 + fr * 16;
    const LAS unsigned char* pE = lds + LDS_E + fq * 1024 + fr * 16;
    asm volatile("" : "+v"(pA), "+v"(pB), "+v"(pE));
#define SA(b, h) (((b) * 2 + (h)) * HTB)
#define SB(b, h) ((4 + (b) * 2 + (h)) * HTB)
#define LDA(dst, b, h) do { _Pragma("unroll") for (int m = 0; m < 4; ++m) _Pragma("unroll") for (int k = 0; k < 2; ++k) dst[m][k] = *(const LAS i32x4*)(pA + SA(b, h) + m * 512 + k * 8192); } while (0)
#define LDB(dst, b, h) do { _Pragma("unroll") for (int n = 0; n < 2; ++n) _Pragma("unroll") for (int k = 0; k < 2; ++k) dst[n][k] = *(const LAS i32x4*)(pB + SA(b, h) + n * 256 + k * 8192); } while (0)
#define LDE(dst, b) do { _Pragma("unroll") for (int m = 0; m < 4; ++m) _Pragma("unroll") for (int k = 0; k < 2; ++k) dst[m][k] = *(const LAS i32x4*)(pE + (b) * 8192 + m * 256 + k * 4096); } while (0)
#define MIDX(q) (((q) >> 2) ? 3 - ((q) & 3) : ((q) & 3))
#define NIDX(q) ((q) >> 2)
#define MM8(C, At, Bt) do { __builtin_amdgcn_s_setprio(1); _Pragma("unroll") for (int q = 0; q < 8; ++q) { const int m = MIDX(q), n = NIDX(q); \
        asm volatile("v_mfma_f32_16x16x128_f8f6f4 %0, %1, %2, %0" : "+v"(C[m][n]) : "v"(cat(Bt[n][0], Bt[n][1])), "v"(cat(At[m][0], At[m][1]))); } __builtin_amdgcn_s_setprio(0); } while (0)
#define MM8Z(C, At, Bt) do { __builtin_amdgcn_s_setprio(1); _Pragma("unroll") for (int q = 0; q < 8; ++q) { const int m = MIDX(q), n = NIDX(q); \
        asm volatile("v_mfma_f32_16x16x128_f8f6f4 %0, %1, %2, 0" : "=&v"(C[m][n]) : "v"(cat(Bt[n][0], Bt[n][1])), "v"(cat(At[m][0], At[m][1]))); } __builtin_amdgcn_s_setprio(0); } while (0)
#define SELB() do { _Pragma("unroll") for (int n = 0; n < 2; ++n) _Pragma("unroll") for (int k = 0; k < 2; ++k) { B0[n][k] = wr ? B1[n][k] : B0[n][k]; asm volatile("" : "+v"(B0[n][k])); } } while (0)
#define MME(At) MM8(accE, At, B0)
#define MMEZ(At) MM8Z(accE, At, B0)
#define MMA(ai, bj, At, Bt) MM8(acc[ai][bj], At, Bt)
#define MMA0(ai, bj, At, Bt) MM8Z(acc[ai][bj], At, Bt)
    f32x4 acc[2][2][4][2], accE[4][2];
    i32x4 At[4][2], B0[2][2], B1[2][2];
    const unsigned kstep = 4096, hstep = 2048, tstep = NT * 4096;
    const unsigned cB = xw + (unsigned)pm * tstep;
    unsigned cA = ww + (unsigned)slot * tstep;
    const bool ex = slot < 2;
    const unsigned cE = EBASE(slot);
    const int l8 = lane & 7;
    const unsigned foff = OFF_FLAGS + (unsigned)wid * FLAG_SRC + (l8 < 2 ? pm * 2 + l8 : l8 == 4 ? (16 + NSLOT * NROUND) * 2 + p.rank : (16 + slot) * 2 + (l8 == 3)) * 64;
    constexpr unsigned long long IGN_XE = 0x1313131313131313ull, IGN_E = 0x1010101010101010ull;
    spin_flags(myws, foff, ex ? 0ull : IGN_E, call);
    unsigned fnew = call - 1u, tm = 0u;
    unsigned sA = 4u, sB = 8u;
    STAMP(1);
    if (ex) STAGEE(LDS_E, cE);
    STAGE(SB(0, 0), cB); STAGE(SA(0, 0), cA); STAGE(SB(0, 1), cB + hstep); STAGE(SA(0, 1), cA + hstep);
    if (wr == 1) BAR;
    WAIT_V(4); BAR;
    STAGE(SB(1, 0), cB + kstep); STAGE(SA(1, 0), cA + kstep); STAGE(SB(1, 1), cB + hstep + kstep); STAGE(SA(1, 1), cA + hstep + kstep);
    WAIT_V(8); BAR;
#define FCHECK(ign) do { if ((__ballot((int)(fnew - call) >= 0) | (ign)) != ~0ull) spin_flags(myws, foff + sA, (ign), call); } while (0)
#define ITSTAMP()
#define NEXTLEV() do { sA = sB; sB += (sB & 63u) == 60u ? FLAG_ROUND - 60u : 4u; } while (0)
#define ITER(EX, FIRST, MM, MMX, W4, W8, FCHK, a1h, a2, b2, e1, e2, LAST) do { \
        const unsigned a3 = (a2) + kstep, b3 = (b2) + kstep; \
          \
        LDB(B0, 0, 0); SCHED; LDA(At, 0, 0); if (!(FIRST)) STAGE(SA(1, 1), a1h); \
        WAIT_L(8); BAR; WAIT_L(0); MM(0, 0, At, B0); BAR; SCHED; \
          \
        FCHK; \
        LDB(B1, 0, 1); STAGE(SB(0, 0), b2); if (EX) STAGEE(LDS_E + 8192, e1); \
        BAR; WAIT_L(0); MM(0, 1, At, B1); BAR; \
          \
        LDA(At, 0, 1); STAGE(SA(0, 0), a2); \
        BAR; WAIT_L(0); MM(1, 0, At, B0); BAR; SCHED; \
          \
        STAGE(SB(0, 1), (b2) + hstep); \
        W4; \
        BAR; MM(1, 1, At, B1); BAR; \
        if (EX) { SCHED; LDE(At, 0); SELB(); BAR; WAIT_L(0); MMX(At); BAR; SCHED; } \
          \
        LDB(B0, 1, 0); SCHED; LDA(At, 1, 0); STAGE(SA(0, 1), (a2) + hstep); \
        WAIT_L(8); BAR; WAIT_L(0); MMA(0, 0, At, B0); BAR; SCHED; \
          \
        LDB(B1, 1, 1); STAGE(SB(1, 0), b3); if (EX) STAGEE(LDS_E, e2); \
        fnew = __builtin_amdgcn_raw_buffer_load_b32(rsf, foff, sB, 17); \
        if (LAST) tm = __builtin_amdgcn_raw_buffer_load_b32(rsf, OFF_TMO, 0, 17);        \
        BAR; WAIT_L(0); MMA(0, 1, At, B1); BAR; \
          \
        LDA(At, 1, 1); STAGE(SA(1, 0), a3); \
        BAR; WAIT_L(0); MMA(1, 0, At, B0); BAR; SCHED; \
          \
        STAGE(SB(1, 1), b3 + hstep); \
        NEXTLEV(); ITSTAMP(); \
        W8; BAR; MMA(1, 1, At, B1); BAR; \
        if (EX) { SCHED; LDE(At, 1); SELB(); BAR; WAIT_L(0); MME(At); BAR; SCHED; } } while (0)
#define EPILOGUE(r, AUX) do { \
        int tz = threadIdx.x; asm volatile("" : "+v"(tz)); \
        const int lane2 = tz & 63, fr2 = lane2 & 15, fq2 = lane2 >> 4; \
        const int pn = NSLOT * (r) + slot; \
        const unsigned ovoff = (unsigned)(wc * 32 + fq2 * 4) * (N * 4) + (unsigned)(wr * 64 + fr2 * 4) * 4; \
        const unsigned osoff = (unsigned)(pm * 256) * (N * 4) + (unsigned)pn * 1024; \
        const bool bad = rfl(tm) == call; \
        const float czz = bad ? __builtin_nanf("") : cz; \
        _Pragma("unroll") for (int ai = 0; ai < 2; ++ai) _Pragma("unroll") for (int bj = 0; bj < 2; ++bj) _Pragma("unroll") for (int n = 0; n < 2; ++n) { \
              const f32x4 a4[4] = {acc[ai][bj][0][n], acc[ai][bj][1][n], acc[ai][bj][2][n], acc[ai][bj][3][n]}; \
              f32x4 o4[4]; \
              silu_rows(a4, czz, o4); \
              store_rows<AUX>(o4, ro, ovoff, osoff + (unsigned)(bj * 128 + n * 16) * (N * 4) + ai * 512); } } while (0)
    if (ex) {
      const unsigned nA = cA + NSLOT * tstep;
      ITER(true, true, MMA0, MMEZ, WAIT_V(7), WAIT_V(8), FCHECK(0ull), 0u, cA + 2 * kstep, cB + 2 * kstep, cE + kstep, cE + 2 * kstep, false);
      for (int t = 2; t < NT - 2; t += 2) {
        ITER(true, false, MMA, MME, WAIT_V(7), WAIT_V(8), FCHECK(0ull), cA + (unsigned)(t + 1) * kstep + hstep, cA + (unsigned)(t + 2) * kstep, cB + (unsigned)(t + 2) * kstep,
             cE + (unsigned)(t + 1) * kstep, cE + (unsigned)(t + 2) * kstep, false);
      }
      ITER(true, false, MMA, MME, WAIT_V(7), WAIT_V(8), FCHECK(IGN_XE), cA + (unsigned)(NT - 1) * kstep + hstep, nA, cB, cE + (unsigned)(NT - 1) * kstep, cE, true);
      asm volatile("s_nop 15" ::: "memory");
      STAMP(2);
      {
        int tz = threadIdx.x; asm volatile("" : "+v"(tz));
        const int lane2 = tz & 63, fr2 = lane2 & 15, fq2 = lane2 >> 4;
        const unsigned ovoff = (unsigned)(wr * 128 + wc * 32 + fq2 * 4) * (N * 4) + (unsigned)(fr2 * 4) * 4;
        const unsigned osoff = (unsigned)(pm * 256) * (N * 4) + (unsigned)(NSLOT * NROUND * 256 + p.rank * 128 + slot * 64) * 4;
        const float czz = rfl(tm) == call ? __builtin_nanf("") : cz;
#pragma unroll
        for (int n = 0; n < 2; ++n) {
          const f32x4 a4[4] = {accE[0][n], accE[1][n], accE[2][n], accE[3][n]};
          f32x4 o4[4];
          silu_rows(a4, czz, o4);
          store_rows<17>(o4, ro, ovoff, osoff + (unsigned)(n * 16) * (N * 4));
        }
      }
      STAGE(SA(1, 1), nA + kstep + hstep);
      EPILOGUE(0, 2);
      STAMP(3);
      cA = nA;
    }
    for (int r = ex ? 1 : 0; r < NROUND; ++r) {
      const bool has_next = r + 1 < NROUND;
      const unsigned long long ign = r == 0 ? IGN_E : IGN_XE;
      const unsigned nA = has_next ? cA + NSLOT * tstep : cA;
      ITER(false, true, MMA0, MMEZ, if (r > 0) { WAIT_V(38); if (ex && r == 1 && lane == 0) __hip_atomic_store((gu32*)(myws + OFF_OFLAG + ((pm * 2 + slot) * 8 + wid) * 4), call, __ATOMIC_RELAXED, __HIP_MEMORY_SCOPE_SYSTEM); } else WAIT_V(6),
           WAIT_V(7), FCHECK(ign), 0u, cA + 2 * kstep, cB + 2 * kstep, 0u, 0u, false);
      for (int t = 2; t < NT - 2; t += 2) {
        ITER(false, false, MMA, MME, WAIT_V(6), WAIT_V(7), FCHECK(ign), cA + (unsigned)(t + 1) * kstep + hstep, cA + (unsigned)(t + 2) * kstep, cB + (unsigned)(t + 2) * kstep, 0u, 0u, false);
      }
      const unsigned long long ignl = has_next ? IGN_XE : ~0ull;
      ITER(false, false, MMA, MME, WAIT_V(6), WAIT_V(7), FCHECK(ignl), cA + (unsigned)(NT - 1) * kstep + hstep, nA, cB, 0u, 0u, true);
      STAGE(SA(1, 1), nA + kstep + hstep);
      asm volatile("s_nop 15" ::: "memory");
      STAMP(2 + 2 * r);
      EPILOGUE(r, 2);
      STAMP(3 + 2 * r);
      cA = nA;
    }
    WAIT_V(0);
    if (wr == 0) BAR;
  }
  STAMP(11);
  if (threadIdx.x == 0) __hip_atomic_fetch_add((gu32*)(myws + OFF_CNT), 1u, __ATOMIC_RELAXED, __HIP_MEMORY_SCOPE_AGENT);
}
}
#define HIP_CHECK(expr) do { hipError_t _e = (expr); if (_e != hipSuccess) { \
    fprintf(stderr, "HIP error %s at %s:%d\n", hipGetErrorString(_e), __FILE__, __LINE__); \
    return; } } while (0)

extern "C" void kernel_launch(int rank, int nranks, ncclComm_t comm, hipStream_t stream,
                              void* const* d_in, const int* in_sizes, int n_in,
                              void* d_out, int out_size, const int* in_dtypes, int out_dtype,
                              void* const* d_out_all, void* const* d_ws_all, size_t ws_bytes) {
  (void)comm; (void)in_dtypes; (void)out_dtype; (void)d_out_all;
  if (nranks != gar::NR || n_in != 4 || (unsigned)rank >= (unsigned)gar::NR || ws_bytes < gar::WS_NEED ||
      in_sizes[0] != gar::M * gar::KS || in_sizes[1] != gar::N * gar::KS || out_size != gar::M * gar::N) {
    fprintf(stderr, "kernel_launch: unexpected shapes\n");
    return;
  }
  static bool attr_set = false;
  if (!attr_set) {
    HIP_CHECK(hipFuncSetAttribute((const void*)gar::gemm_ag_kernel, hipFuncAttributeMaxDynamicSharedMemorySize, gar::LDS_BYTES));
    attr_set = true;
  }
  gar::Params p;
  p.x = (const gar::u8*)d_in[0]; p.w = (const gar::u8*)d_in[1];
  p.sx = (const float*)d_in[2]; p.sw = (const float*)d_in[3];
  p.out = (float*)d_out;
  for (int i = 0; i < gar::NR; ++i) p.ws[i] = (gar::u8*)d_ws_all[i];
  for (int i = 0; i < gar::NR; ++i) p.outs[i] = (float*)d_out_all[i];
  p.rank = rank; p.pad = 0;
  gar::gemm_ag_kernel<<<gar::GRID, 512, gar::LDS_BYTES, stream>>>(p);
  HIP_CHECK(hipGetLastError());
}
```

```cpp
#include <hip/hip_runtime.h>
#include <rccl/rccl.h>
#include <cstdio>

extern "C" __device__ float gar_fmul_legacy(float, float) __asm("llvm.amdgcn.fmul.legacy");
namespace gar {
__device__ __forceinline__ float fmul_single(float a, float b) { return gar_fmul_legacy(a, b); }
#define LAS __attribute__((address_space(3)))
typedef int i32x4 __attribute__((ext_vector_type(4)));
typedef int i32x8 __attribute__((ext_vector_type(8)));
typedef unsigned u32x4 __attribute__((ext_vector_type(4)));
typedef float f32x4 __attribute__((ext_vector_type(4)));
typedef float f32x2 __attribute__((ext_vector_type(2)));
typedef unsigned char u8;
typedef __amdgpu_buffer_rsrc_t rsrc_t;
typedef __attribute__((address_space(1))) unsigned gu32;

constexpr int M = 4096, N = 16384, NR = 8, KS = 512;
constexpr int NT = 32;
constexpr int HTB = 16384;
constexpr int LDS_E = 8 * HTB;
constexpr int LDS_BYTES = 8 * HTB + 16384;
constexpr int GRID = 256, NCOPY = 16, NSLOT = 15, NROUND = 4;
#define NCW 4
constexpr unsigned NCWAVE = NCOPY * NCW;
#define GAR_OUTS 1
constexpr unsigned NOPER = 2432, NUNIT0 = 992, NUNIT = NOPER + 512;
constexpr unsigned NLOCAL = NOPER + 128;
constexpr unsigned OFF_CNT = 0, OFF_TMO = 256, OFF_QUEUE = 512;
constexpr unsigned OFF_OFLAG = 4096, OFF_DONE = 8192;
constexpr unsigned OFF_FLAGS = 65536;
constexpr unsigned FLAG_SRC = 80 * 128, FLAG_ROUND = NSLOT * 128;
constexpr unsigned OFF_DATA = 1u << 20;
constexpr unsigned XREG = (unsigned)M * KS;
constexpr unsigned WREG = (unsigned)N * KS;
constexpr unsigned XALL = NR * XREG, WALL = NR * WREG;
constexpr unsigned PAR_BYTES = XALL + WALL;
constexpr size_t WS_NEED = (size_t)OFF_DATA + 2 * (size_t)PAR_BYTES;
constexpr unsigned long long DEADLINE = 30000000ull;
constexpr int RSRC3 = 0x00020000;
#define SLEEPV 8
#define OUTP 1
#define STAGN 2
#define LDAUX 17
#define KPRE 7

struct Params {
  const u8* x; const u8* w; const float* sx; const float* sw; float* out;
  u8* ws[NR];
  float* outs[NR];
  int rank; int pad;
};

__device__ __forceinline__ i32x8 cat(i32x4 a, i32x4 b) { return __builtin_shufflevector(a, b, 0, 1, 2, 3, 4, 5, 6, 7); }
__device__ __forceinline__ unsigned rfl(unsigned v) { return __builtin_amdgcn_readfirstlane(v); }
__device__ __forceinline__ rsrc_t mk_rsrc(const void* p, unsigned bytes) {
  const unsigned long long a = (unsigned long long)p;
  const unsigned lo = rfl((unsigned)a), hi = rfl((unsigned)(a >> 32));
  return __builtin_amdgcn_make_buffer_rsrc((void*)(((unsigned long long)hi << 32) | lo), (short)0, (int)bytes, RSRC3);
}
#define STR2(x) #x
#define STR(x) STR2(x)
#define WAIT_V(n) asm volatile("s_waitcnt vmcnt(" STR(n) ")" ::: "memory")
#define WAIT_L(n) asm volatile("s_waitcnt lgkmcnt(" STR(n) ")" ::: "memory")
#define BAR __builtin_amdgcn_s_barrier()
#define SCHED __builtin_amdgcn_sched_barrier(0)
#define STAMP(i)
#define CSTAMP(e, v)

__device__ __noinline__ void spin_flags(u8* ws, unsigned off, unsigned long long ign, unsigned call) {
  gu32* tmo = (gu32*)(ws + OFF_TMO);
  gu32* fp = (gu32*)(ws + off);
  const unsigned long long t0 = __builtin_amdgcn_s_memrealtime();
  for (;;) {
    const unsigned v = __hip_atomic_load(fp, __ATOMIC_RELAXED, __HIP_MEMORY_SCOPE_SYSTEM);
    unsigned t = __hip_atomic_load(tmo, __ATOMIC_RELAXED, __HIP_MEMORY_SCOPE_AGENT);
    asm volatile("" : "+v"(t));
    if ((__ballot((int)(v - call) >= 0) | ign) == ~0ull) return;
    if (rfl(t) == call || __builtin_amdgcn_s_memrealtime() - t0 > DEADLINE) {
      __hip_atomic_store(tmo, call, __ATOMIC_RELAXED, __HIP_MEMORY_SCOPE_AGENT);
      WAIT_V(0);
      return;
    }
    __builtin_amdgcn_s_sleep(SLEEPV);
  }
}

#define XDL 5, 2, 5, 2, 5, 2, 5, 2, 3, 0, 3, 0, 3, 0, 3, 0
#define WA 16u
struct XSched { unsigned long long perm, del, cnt; unsigned last; };
constexpr XSched make_xsched() {
  const unsigned d[16] = {XDL};
  XSched s{0ull, 0ull, 0ull, 0u};
  unsigned n = 0;
  for (unsigned v = 0; v < 12; ++v) {
    for (unsigned i = 0; i < 16; ++i) if (d[i] == v) { s.perm |= (unsigned long long)i << (4 * n); s.del |= (unsigned long long)v << (4 * n); s.last = v; ++n; }
    s.cnt |= (unsigned long long)n << (5 * v);
  }
  return s;
}
constexpr XSched XS = make_xsched();
static_assert(((XS.cnt >> 55) & 31) == 16 && (XS.del & 15) == 0, "delays 0..11, the first 0");
struct Unit { unsigned src, dst, flag; int kind; };
__device__ __forceinline__ Unit decode_unit(unsigned u) {
  unsigned panel, h, lev; bool isx = false;
  if (u < NOPER) {
    unsigned v = 0u, jj = u, g0 = 0u, nx = 0u;
    constexpr unsigned VW = 16u + (WA < XS.last ? WA : XS.last), NA = 512u + VW * 30u;
#define XCNT(t) ((t) >= 11u ? 16u : (unsigned)(XS.cnt >> (5u * (t))) & 31u)
    if (u < NA) {
      for (;; ++v) {
        g0 = v < 16u ? 0u : XCNT(v - 16u);
        nx = 2u * (XCNT(v) - g0);
        const unsigned n = nx + (v < VW ? 30u : 0u);
        if (jj < n) break;
        jj -= n;
      }
    } else { const unsigned t = u - NA; v = t / 30u; jj = t - v * 30u; v += VW; }
    isx = jj < nx;
    const unsigned si = 4u * ((g0 + (jj >> 1)) & 15u), jw = jj - nx;
    h = jj & 1u;
    panel = isx ? (unsigned)(XS.perm >> si) & 15u : NSLOT * (v >> 4) + (jw >> 1);
    lev = isx ? v - ((unsigned)(XS.del >> si) & 15u) : v & 15u;
  } else {
    const unsigned v = u - NOPER;
    lev = v & 15u; h = (v >> 4) & 1u; panel = NSLOT * NROUND + (v >> 5);
  }
  Unit o; o.kind = isx ? 0 : 1;
  o.src = isx ? (panel * 256u + h * 128u) * KS + lev * 32u : lev * 32u * N + panel * 256u + h * 128u;
  o.dst = (isx ? 0u : XALL) + panel * (NT * 4096u) + lev * 8192u + h * 2048u;
  o.flag = (((isx ? 0u : 16u) + panel) * 2u + h) * 64u + lev * 4u;
  return o;
}
__device__ __forceinline__ Unit decode_queue(unsigned u) {
  if (u < NOPER) return decode_unit(u);
  const unsigned v = (u < NUNIT ? u : NUNIT - 1u) - NOPER;
  Unit o; o.kind = 2; o.flag = ~0u;
  o.src = o.dst = ((v >> 5) * 256u + ((v >> 1) & 7u) * 32u + (v & 1u) * 16u) * (N * 4u) + ((v >> 4) & 1u) * 256u;
  return o;
}
__device__ __forceinline__ void unit_load(const Unit& u, rsrc_t srcx, rsrc_t srcw, int lane, unsigned (&d)[16]) {
  if (u.kind == 0) {
#pragma unroll
    for (int t = 0; t < 4; ++t) { const u32x4 v = __builtin_amdgcn_raw_buffer_load_b128(srcx, (lane & 31) * KS + (lane >> 5) * 16, u.src + t * 32 * KS, 0);
      d[4 * t] = v[0]; d[4 * t + 1] = v[1]; d[4 * t + 2] = v[2]; d[4 * t + 3] = v[3]; }
  } else {
#pragma unroll
    for (int i = 0; i < 16; ++i) d[i] = __builtin_amdgcn_raw_buffer_load_b32(srcw, (lane >> 5) * 16 * N + (lane & 31) * 4, u.src + i * N, 0);
  }
}
__device__ __forceinline__ void unit_retile(const Unit& u, const unsigned (&d)[16], u32x4 (&o)[4]) {
  if (u.kind != 1) {
#pragma unroll
    for (int t = 0; t < 4; ++t) o[t] = (u32x4){d[4 * t], d[4 * t + 1], d[4 * t + 2], d[4 * t + 3]};
  } else {
#pragma unroll
    for (int q = 0; q < 4; ++q) {
      const unsigned t0 = __builtin_amdgcn_perm(d[4 * q + 1], d[4 * q], 0x05010400u), t1 = __builtin_amdgcn_perm(d[4 * q + 1], d[4 * q], 0x07030602u);
      const unsigned t2 = __builtin_amdgcn_perm(d[4 * q + 3], d[4 * q + 2], 0x05010400u), t3 = __builtin_amdgcn_perm(d[4 * q + 3], d[4 * q + 2], 0x07030602u);
      o[0][q] = __builtin_amdgcn_perm(t2, t0, 0x05040100u); o[1][q] = __builtin_amdgcn_perm(t2, t0, 0x07060302u);
      o[2][q] = __builtin_amdgcn_perm(t3, t1, 0x05040100u); o[3][q] = __builtin_amdgcn_perm(t3, t1, 0x07060302u);
    }
  }
}
__device__ __forceinline__ unsigned call_of(unsigned counter) { return rfl(counter) / GRID + 1u; }
__device__ __forceinline__ unsigned local_role(const Params& p, unsigned counter, unsigned gw) {
  const int lane = threadIdx.x & 63;
  const unsigned rank = (unsigned)p.rank;
  const rsrc_t srcx = mk_rsrc(p.x, (unsigned)M * KS), srcw = mk_rsrc(p.w, (unsigned)N * KS);
  constexpr unsigned NGW = (GRID - NCOPY) * 8;
  const unsigned ua = gw < 128u ? NOPER + gw : gw - 128u, ub = gw + NGW - 128u;
  const bool two = ub < NOPER;
  const Unit a = decode_unit(ua), b = decode_unit(two ? ub : 0u);
  unsigned da[16], db[16]; u32x4 o[4];
  unit_load(a, srcx, srcw, lane, da);
  if (two) unit_load(b, srcx, srcw, lane, db);
  SCHED;
  const unsigned call = call_of(counter);
  const unsigned dvo = OFF_DATA + (call & 1u) * PAR_BYTES + (lane >> 5) * 4096 + (lane & 31) * 16;
  u8* wa = p.ws[gw < 128u ? gw >> 4 : rank];
  {
    const rsrc_t dst = mk_rsrc(wa, (unsigned)WS_NEED);
    unit_retile(a, da, o);
    const unsigned voff = dvo + a.dst + rank * (a.kind == 0 ? XREG : WREG);
#pragma unroll
    for (int q = 0; q < 4; ++q) __builtin_amdgcn_raw_buffer_store_b128(o[q], dst, voff + q * 512, 0, 17);
  }
  if (two) {
    const rsrc_t dst = mk_rsrc(p.ws[rank], (unsigned)WS_NEED);
    unit_retile(b, db, o);
    const unsigned voff = dvo + b.dst + rank * (b.kind == 0 ? XREG : WREG);
#pragma unroll
    for (int q = 0; q < 4; ++q) __builtin_amdgcn_raw_buffer_store_b128(o[q], dst, voff + q * 512, 0, 17);
  }
  WAIT_V(0);
  if (lane == 0) {
    __hip_atomic_store((gu32*)(wa + OFF_FLAGS + rank * FLAG_SRC + a.flag), call, __ATOMIC_RELAXED, __HIP_MEMORY_SCOPE_SYSTEM);
    if (two) __hip_atomic_store((gu32*)(p.ws[rank] + OFF_FLAGS + rank * FLAG_SRC + b.flag), call, __ATOMIC_RELAXED, __HIP_MEMORY_SCOPE_SYSTEM);
  }
  return call;
}
__device__ __forceinline__ unsigned copy_role(const Params& p, unsigned counter, int j) {
  const int tid = threadIdx.x, g = rfl(tid >> 6), lane = tid & 63;
  if (g >= NCW) return call_of(counter);
  const unsigned rank = (unsigned)p.rank;
  unsigned cur = (unsigned)j * NCW + (unsigned)g, nxt = cur + NCWAVE;
  const rsrc_t srcx = mk_rsrc(p.x, (unsigned)M * KS), srcw = mk_rsrc(p.w, (unsigned)N * KS), srco = mk_rsrc(p.out, (unsigned)M * N * 4u);
  unsigned d[16]; u32x4 o[4];
  Unit uc = decode_unit(cur);
  unit_load(uc, srcx, srcw, lane, d);
  u8* myws = p.ws[rank];
  const rsrc_t rq = mk_rsrc(myws, OFF_FLAGS);
  unsigned qvo = lane == 0 ? OFF_QUEUE : 0x80000000u;
  asm volatile("" : "+v"(qvo));
  const unsigned wave = cur;
  rsrc_t dst[NR - 1], dsto[NR - 1];
#pragma unroll
  for (int d = 0; d < NR - 1; ++d) { dst[d] = mk_rsrc(p.ws[(rank + 1 + d) & 7], (unsigned)WS_NEED); dsto[d] = mk_rsrc(p.outs[(rank + 1 + d) & 7], (unsigned)M * N * 4u); }
  u8* flagp = p.ws[(rank + 1 + lane) & 7] + OFF_FLAGS + rank * FLAG_SRC;
  const unsigned ovo = (unsigned)(lane >> 4) * (N * 4) + (unsigned)(lane & 15) * 16 + (NSLOT * NROUND * 256 + rank * 128) * 4;
  SCHED;
  const unsigned call = call_of(counter);
  const unsigned qbase = (call - 1u) * NUNIT - 2u * NCWAVE;
  const unsigned dvo = OFF_DATA + (call & 1u) * PAR_BYTES + (lane >> 5) * 4096 + (lane & 31) * 16;
#define PUSH(q0, q1) do { if (uc.kind == 2) { _Pragma("unroll") for (int q = q0; q < q1; ++q) __builtin_amdgcn_raw_buffer_store_b128(o[q / 7], dsto[q % 7], voff + (q / 7) * 4 * (N * 4), 0, 17); } \
    else { _Pragma("unroll") for (int q = q0; q < q1; ++q) __builtin_amdgcn_raw_buffer_store_b128(o[q / 7], dst[q % 7], voff + (q / 7) * 512, 0, 17); } } while (0)
#define PUBLISH(fl) do { if (lane < NR - 1) __hip_atomic_store((gu32*)(flagp + (fl)), call, __ATOMIC_RELAXED, __HIP_MEMORY_SCOPE_SYSTEM); } while (0)
  unsigned pflag = ~0u;
  bool oready = false;
  for (;;) {
    const Unit un = decode_queue(nxt);
    SCHED;
    unit_retile(uc, d, o);
    const unsigned voff = uc.kind == 2 ? ovo + uc.dst : dvo + uc.dst + rank * (uc.kind == 0 ? XREG : WREG);
    SCHED;
    PUSH(0, KPRE);
    WAIT_V(KPRE);
    if (pflag != ~0u) PUBLISH(pflag);
    pflag = uc.flag;
    SCHED;
    if (!oready && nxt >= NOPER - 480u + wave * 6u) {
      const u32x4 of = __builtin_amdgcn_raw_buffer_load_b128(rq, OFF_OFLAG + lane * 16, 0, 17);
      if (!__all((int)(of[0] - call) >= 0 && (int)(of[1] - call) >= 0 && (int)(of[2] - call) >= 0 && (int)(of[3] - call) >= 0))
        for (int q = 0; q < 4; ++q) spin_flags(myws, OFF_OFLAG + (q * 64 + lane) * 4, 0ull, call);
      oready = true;
    }
    if (un.kind == 2) {
#pragma unroll
      for (int t = 0; t < 4; ++t) { const u32x4 v = __builtin_amdgcn_raw_buffer_load_b128(srco, ovo, un.src + t * 4 * (N * 4), 17);
        d[4 * t] = v[0]; d[4 * t + 1] = v[1]; d[4 * t + 2] = v[2]; d[4 * t + 3] = v[3]; }
    } else unit_load(un, srcx, srcw, lane, d);
    const unsigned ticket = (unsigned)__builtin_amdgcn_raw_ptr_buffer_atomic_add_i32(1, rq, qvo, 0, 16);
    SCHED;
    PUSH(KPRE, 28);
    SCHED;
    cur = nxt; uc = un;
    if (cur >= NUNIT) break;
    nxt = rfl(ticket) - qbase;
  }
  WAIT_V(0);
  if (pflag != ~0u) PUBLISH(pflag);
  if (lane < NR) __hip_atomic_store((gu32*)(p.ws[lane & 7] + OFF_DONE + (rank * 64u + wave) * 4u), call, __ATOMIC_RELAXED, __HIP_MEMORY_SCOPE_SYSTEM);
  CSTAMP(24 + g, __builtin_amdgcn_s_memrealtime());
  return call;
}

template <int AUX> __device__ __forceinline__ void store_rows(const f32x4 (&o)[4], rsrc_t r, unsigned voff, unsigned soff) {
  SCHED;
#pragma unroll
  for (int j = 0; j < 4; ++j) {
    __builtin_amdgcn_raw_buffer_store_b128(__builtin_bit_cast(u32x4, o[j]), r, voff, soff + (unsigned)j * (N * 4), AUX);
  }
  asm volatile("s_nop 1" ::: "memory");
  SCHED;
}
struct Silu { float nz, nc, k2, k; };
__device__ __forceinline__ void silu_rows(const f32x4 (&a)[4], const Silu& c, f32x4 (&o)[4]) {
#pragma unroll
  for (int h = 0; h < 2; ++h) {
    f32x2 r[4];
#pragma unroll
    for (int m = 0; m < 4; ++m) {
      const f32x2 x = __builtin_elementwise_fma((f32x2){a[m][2 * h], a[m][2 * h + 1]}, (f32x2){c.nz, c.nz}, (f32x2){c.nc, c.nc});
      f32x2 e;
      e.x = __builtin_amdgcn_fmed3f(__builtin_amdgcn_exp2f(x.x), 0.f, 1.f); e.y = __builtin_amdgcn_fmed3f(__builtin_amdgcn_exp2f(x.y), 0.f, 1.f);
      const f32x2 dd = __builtin_elementwise_fma(e, (f32x2){c.k2, c.k2}, (f32x2){c.k, c.k});
      r[m].x = __builtin_amdgcn_rcpf(dd.x); r[m].y = __builtin_amdgcn_rcpf(dd.y);
    }
#pragma unroll
    for (int m = 0; m < 4; ++m) {
      o[2 * h][m] = fmul_single(a[m][2 * h], r[m].x); o[2 * h + 1][m] = fmul_single(a[m][2 * h + 1], r[m].y); }
  }
}

__global__ void __launch_bounds__(512, 2) gemm_ag_kernel(Params p) {
  extern __shared__ __attribute__((aligned(16))) unsigned char shm[];
  LAS unsigned char* lds = (LAS unsigned char*)shm;
  u8* myws = p.ws[p.rank];
  const unsigned counter = __hip_atomic_load((gu32*)(myws + OFF_CNT), __ATOMIC_RELAXED, __HIP_MEMORY_SCOPE_AGENT);
  const int bid = blockIdx.x;
#define XMAP 0
  const bool is_copy = bid < NCOPY;
  const int cj = bid;
  STAMP(0);
  if (is_copy) {
    const unsigned call = copy_role(p, counter, cj);
    if (cj == 0) {
      BAR;
      spin_flags(myws, OFF_DONE + (threadIdx.x >> 6) * 256u + (threadIdx.x & 63u) * 4u, 0ull, call);
      if (rfl(__hip_atomic_load((gu32*)(myws + OFF_TMO), __ATOMIC_RELAXED, __HIP_MEMORY_SCOPE_AGENT)) == call && (threadIdx.x & 63u) == 0u) p.out[0] = __builtin_nanf("");
    }
    STAMP(1);
  }

  const int tid = threadIdx.x, wid = rfl(tid >> 6), lane = tid & 63, wr = wid >> 2, wc = wid & 3, fr = lane & 15, fq = lane >> 4;
  const rsrc_t ro = mk_rsrc(p.out, (unsigned)M * N * 4u), rsf = mk_rsrc(myws, OFF_DATA);
  const unsigned voff = lane * 16;
  const unsigned ldsw = (unsigned)wid * 2048u;
  const unsigned xw = (unsigned)wid * XREG, ww = XALL + (unsigned)wid * WREG;
  const float scale = p.sx[0] * p.sw[0];
  const float clip2 = __builtin_fminf(86.56170245333781f, __builtin_fabsf(scale) * (1.4426950408889634f * 1073741824.f) + 1.f);
  Silu sc; sc.nz = scale * -1.4426950408889634f; sc.nc = -clip2; sc.k = 1.f / scale; sc.k2 = sc.k * __builtin_amdgcn_exp2f(clip2);
#define STAGE(bufoff, soff) do { \
    __builtin_amdgcn_raw_ptr_buffer_load_lds(rs, (LAS void*)(lds + (bufoff) + ldsw), 16, voff, (soff), 0, LDAUX); \
    __builtin_amdgcn_raw_ptr_buffer_load_lds(rs, (LAS void*)(lds + (bufoff) + ldsw), 16, voff, (soff), 1024, LDAUX); } while (0)

  const unsigned voffE = (unsigned)(lane >> 4) * 512u + (unsigned)(lane & 15) * 16u;
#define STAGEE(buf, soff) __builtin_amdgcn_raw_ptr_buffer_load_lds(rs, (LAS void*)(lds + (buf) + wid * 1024), 16, voffE, (soff), 0, LDAUX)
#define EBASE(q) (ww + (unsigned)(NSLOT * NROUND + (p.rank >> 1)) * (NT * 4096) + (unsigned)((p.rank & 1) * 2048) + (unsigned)((q) & 1) * 256)
  if (!is_copy) {
    const int gidx = bid - NCOPY, xcd = gidx & 7, idx = gidx >> 3, tl = (xcd >> 1) * 30 + idx;
    const int pm = (xcd & 1) * 8 + (tl & 7), slot = tl >> 3;
    const unsigned call = local_role(p, counter, (unsigned)gidx * 8u + (unsigned)wid);
    const rsrc_t rs = mk_rsrc(myws + OFF_DATA + (call & 1u) * PAR_BYTES, PAR_BYTES);
    const LAS unsigned char* pA = lds + fq * 2048 + wr * 256 + fr * 16;
    const LAS unsigned char* pB = lds + 4 * HTB + fq * 2048 + wc * 512 + fr * 16;
    const LAS unsigned char* pE = lds + LDS_E + fq * 1024 + fr * 16;
    asm volatile("" : "+v"(pA), "+v"(pB), "+v"(pE));
#define SA(b, h) (((b) * 2 + (h)) * HTB)
#define SB(b, h) ((4 + (b) * 2 + (h)) * HTB)
#define LDA(dst, b, h) do { _Pragma("unroll") for (int m = 0; m < 4; ++m) _Pragma("unroll") for (int k = 0; k < 2; ++k) dst[m][k] = *(const LAS i32x4*)(pA + SA(b, h) + m * 512 + k * 8192); } while (0)
#define LDB(dst, b, h) do { _Pragma("unroll") for (int n = 0; n < 2; ++n) _Pragma("unroll") for (int k = 0; k < 2; ++k) dst[n][k] = *(const LAS i32x4*)(pB + SA(b, h) + n * 256 + k * 8192); } while (0)
#define LDE(dst, b) do { _Pragma("unroll") for (int m = 0; m < 4; ++m) _Pragma("unroll") for (int k = 0; k < 2; ++k) dst[m][k] = *(const LAS i32x4*)(pE + (b) * 8192 + m * 256 + k * 4096); } while (0)
#define MIDX(q) (((q) >> 2) ? 3 - ((q) & 3) : ((q) & 3))
#define NIDX(q) ((q) >> 2)
#define MM8(C, At, Bt) do { __builtin_amdgcn_s_setprio(1); _Pragma("unroll") for (int q = 0; q < 8; ++q) { const int m = MIDX(q), n = NIDX(q); \
        asm volatile("v_mfma_f32_16x16x128_f8f6f4 %0, %1, %2, %0" : "+v"(C[m][n]) : "v"(cat(Bt[n][0], Bt[n][1])), "v"(cat(At[m][0], At[m][1]))); } __builtin_amdgcn_s_setprio(0); } while (0)
#define MM8Z(C, At, Bt) do { __builtin_amdgcn_s_setprio(1); _Pragma("unroll") for (int q = 0; q < 8; ++q) { const int m = MIDX(q), n = NIDX(q); \
        asm volatile("v_mfma_f32_16x16x128_f8f6f4 %0, %1, %2, 0" : "=&v"(C[m][n]) : "v"(cat(Bt[n][0], Bt[n][1])), "v"(cat(At[m][0], At[m][1]))); } __builtin_amdgcn_s_setprio(0); } while (0)
#define SELB() do { _Pragma("unroll") for (int n = 0; n < 2; ++n) _Pragma("unroll") for (int k = 0; k < 2; ++k) { B0[n][k] = wr ? B1[n][k] : B0[n][k]; asm volatile("" : "+v"(B0[n][k])); } } while (0)
#define MME(At) MM8(accE, At, B0)
#define MMEZ(At) MM8Z(accE, At, B0)
#define MMA(ai, bj, At, Bt) MM8(acc[ai][bj], At, Bt)
#define MMA0(ai, bj, At, Bt) MM8Z(acc[ai][bj], At, Bt)
    f32x4 acc[2][2][4][2], accE[4][2];
    i32x4 At[4][2], B0[2][2], B1[2][2];
    const unsigned kstep = 4096, hstep = 2048, tstep = NT * 4096;
    const unsigned cB = xw + (unsigned)pm * tstep;
    unsigned cA = ww + (unsigned)slot * tstep;
    const bool ex = slot < 2;
    const unsigned cE = EBASE(slot);
    const int l8 = lane & 7;
    const unsigned foff = OFF_FLAGS + (unsigned)wid * FLAG_SRC + (l8 < 2 ? pm * 2 + l8 : l8 == 4 ? (16 + NSLOT * NROUND) * 2 + p.rank : (16 + slot) * 2 + (l8 == 3)) * 64;
    constexpr unsigned long long IGN_XE = 0x1313131313131313ull, IGN_E = 0x1010101010101010ull;
    spin_flags(myws, foff, ex ? 0ull : IGN_E, call);
    unsigned fnew = call - 1u, tm = 0u;
    unsigned sA = 4u, sB = 8u;
    STAMP(1);
    if (ex) STAGEE(LDS_E, cE);
    STAGE(SB(0, 0), cB); STAGE(SA(0, 0), cA); STAGE(SB(0, 1), cB + hstep); STAGE(SA(0, 1), cA + hstep);
    if (wr == 1) BAR;
    WAIT_V(4); BAR;
    STAGE(SB(1, 0), cB + kstep); STAGE(SA(1, 0), cA + kstep); STAGE(SB(1, 1), cB + hstep + kstep); STAGE(SA(1, 1), cA + hstep + kstep);
    WAIT_V(8); BAR;
#define FCHECK(ign) do { if ((__ballot((int)(fnew - call) >= 0) | (ign)) != ~0ull) spin_flags(myws, foff + sA, (ign), call); } while (0)
#define ITSTAMP()
#define TSTAMP(k)
#define NEXTLEV() do { sA = sB; sB += (sB & 63u) == 60u ? FLAG_ROUND - 60u : 4u; } while (0)
#define ITER(EX, FIRST, MM, MMX, W4, W8, FCHK, a1h, a2, b2, e1, e2, LAST) do { \
        const unsigned a3 = (a2) + kstep, b3 = (b2) + kstep; \
          \
        LDB(B0, 0, 0); SCHED; LDA(At, 0, 0); if (!(FIRST)) STAGE(SA(1, 1), a1h); \
        WAIT_L(8); BAR; WAIT_L(0); MM(0, 0, At, B0); if (FIRST) TSTAMP(4); BAR; SCHED; \
          \
        FCHK; \
        LDB(B1, 0, 1); STAGE(SB(0, 0), b2); if (EX) STAGEE(LDS_E + 8192, e1); \
        if (LAST) tm = __builtin_amdgcn_raw_buffer_load_b32(rsf, OFF_TMO, 0, 17);        \
        BAR; WAIT_L(0); MM(0, 1, At, B1); BAR; \
          \
        LDA(At, 0, 1); STAGE(SA(0, 0), a2); \
        BAR; WAIT_L(0); MM(1, 0, At, B0); BAR; SCHED; \
          \
        STAGE(SB(0, 1), (b2) + hstep); \
        W4; \
        BAR; MM(1, 1, At, B1); if (FIRST) TSTAMP(5); BAR; \
        if (EX) { SCHED; LDE(At, 0); SELB(); BAR; WAIT_L(0); MMX(At); BAR; SCHED; } \
          \
        LDB(B0, 1, 0); SCHED; LDA(At, 1, 0); STAGE(SA(0, 1), (a2) + hstep); \
        WAIT_L(8); BAR; WAIT_L(0); MMA(0, 0, At, B0); BAR; SCHED; \
          \
        LDB(B1, 1, 1); STAGE(SB(1, 0), b3); if (EX) STAGEE(LDS_E, e2); \
        fnew = __builtin_amdgcn_raw_buffer_load_b32(rsf, foff, sB, 17); \
        BAR; WAIT_L(0); MMA(0, 1, At, B1); BAR; \
          \
        LDA(At, 1, 1); STAGE(SA(1, 0), a3); \
        BAR; WAIT_L(0); MMA(1, 0, At, B0); BAR; SCHED; \
          \
        STAGE(SB(1, 1), b3 + hstep); \
        NEXTLEV(); ITSTAMP(); \
        if (FIRST) TSTAMP(6); W8; if (FIRST) TSTAMP(7); BAR; MMA(1, 1, At, B1); BAR; \
        if (EX) { SCHED; LDE(At, 1); SELB(); BAR; WAIT_L(0); MME(At); BAR; SCHED; } } while (0)
#define EPI_ENTER() do { if (wr == 0) BAR; } while (0)
#define EPI_LEAVE(more) do { if ((more) && wr == 1) BAR; } while (0)
#define EPILOGUE(r, AUX) do { \
        int tz = threadIdx.x; asm volatile("" : "+v"(tz)); \
        const int lane2 = tz & 63, fr2 = lane2 & 15, fq2 = lane2 >> 4; \
        const int pn = NSLOT * (r) + slot; \
        const unsigned ovoff = (unsigned)(wc * 32 + fq2 * 4) * (N * 4) + (unsigned)(wr * 64 + fr2 * 4) * 4; \
        const unsigned osoff = (unsigned)(pm * 256) * (N * 4) + (unsigned)pn * 1024; \
        Silu sv = sc; if (rfl(tm) == call) sv.k = __builtin_nanf(""); \
        _Pragma("unroll") for (int ai = 0; ai < 2; ++ai) _Pragma("unroll") for (int bj = 0; bj < 2; ++bj) _Pragma("unroll") for (int n = 0; n < 2; ++n) { \
              const f32x4 a4[4] = {acc[ai][bj][0][n], acc[ai][bj][1][n], acc[ai][bj][2][n], acc[ai][bj][3][n]}; \
              f32x4 o4[4]; \
              silu_rows(a4, sv, o4); \
              store_rows<AUX>(o4, ro, ovoff, osoff + (unsigned)(bj * 128 + n * 16) * (N * 4) + ai * 512); } } while (0)
    if (ex) {
      const unsigned nA = cA + NSLOT * tstep;
      ITER(true, true, MMA0, MMEZ, WAIT_V(7), WAIT_V(8), FCHECK(0ull), 0u, cA + 2 * kstep, cB + 2 * kstep, cE + kstep, cE + 2 * kstep, false);
      for (int t = 2; t < NT - 2; t += 2) {
        ITER(true, false, MMA, MME, WAIT_V(7), WAIT_V(8), FCHECK(0ull), cA + (unsigned)(t + 1) * kstep + hstep, cA + (unsigned)(t + 2) * kstep, cB + (unsigned)(t + 2) * kstep,
             cE + (unsigned)(t + 1) * kstep, cE + (unsigned)(t + 2) * kstep, false);
      }
      ITER(true, false, MMA, MME, WAIT_V(8), WAIT_V(8), FCHECK(IGN_XE), cA + (unsigned)(NT - 1) * kstep + hstep, nA, cB, cE + (unsigned)(NT - 1) * kstep, cE, true);
      asm volatile("s_nop 15" ::: "memory");
      EPI_ENTER();
      STAMP(2);
      {
        int tz = threadIdx.x; asm volatile("" : "+v"(tz));
        const int lane2 = tz & 63, fr2 = lane2 & 15, fq2 = lane2 >> 4;
        const unsigned ovoff = (unsigned)(wr * 128 + wc * 32 + fq2 * 4) * (N * 4) + (unsigned)(fr2 * 4) * 4;
        const unsigned osoff = (unsigned)(pm * 256) * (N * 4) + (unsigned)(NSLOT * NROUND * 256 + p.rank * 128 + slot * 64) * 4;
        Silu sv = sc; if (rfl(tm) == call) sv.k = __builtin_nanf("");
#pragma unroll
        for (int n = 0; n < 2; ++n) {
          const f32x4 a4[4] = {accE[0][n], accE[1][n], accE[2][n], accE[3][n]};
          f32x4 o4[4];
          silu_rows(a4, sv, o4);
          store_rows<17>(o4, ro, ovoff, osoff + (unsigned)(n * 16) * (N * 4));
        }
      }
      STAGE(SA(1, 1), nA + kstep + hstep);
      EPILOGUE(0, 2);
      EPI_LEAVE(true);
      STAMP(3);
      cA = nA;
    }
    for (int r = ex ? 1 : 0; r < NROUND; ++r) {
      const bool has_next = r + 1 < NROUND;
      const unsigned long long ign = r == 0 ? IGN_E : IGN_XE;
      const unsigned nA = has_next ? cA + NSLOT * tstep : cA;
      ITER(false, true, MMA0, MMEZ, if (r > 0) { WAIT_V(38); if (ex && r == 1 && lane == 0) __hip_atomic_store((gu32*)(myws + OFF_OFLAG + ((pm * 2 + slot) * 8 + wid) * 4), call, __ATOMIC_RELAXED, __HIP_MEMORY_SCOPE_SYSTEM); } else WAIT_V(6),
           WAIT_V(7), FCHECK(ign), 0u, cA + 2 * kstep, cB + 2 * kstep, 0u, 0u, false);
      for (int t = 2; t < NT - 2; t += 2) {
        ITER(false, false, MMA, MME, WAIT_V(6), WAIT_V(7), FCHECK(ign), cA + (unsigned)(t + 1) * kstep + hstep, cA + (unsigned)(t + 2) * kstep, cB + (unsigned)(t + 2) * kstep, 0u, 0u, false);
      }
      const unsigned long long ignl = has_next ? IGN_XE : ~0ull;
      ITER(false, false, MMA, MME, WAIT_V(7), WAIT_V(7), FCHECK(ignl), cA + (unsigned)(NT - 1) * kstep + hstep, nA, cB, 0u, 0u, true);
      STAGE(SA(1, 1), nA + kstep + hstep);
      asm volatile("s_nop 15" ::: "memory");
      TSTAMP(0);
      EPI_ENTER();
      TSTAMP(1);
      STAMP(2 + 2 * r);
      EPILOGUE(r, 2);
      TSTAMP(2);
      EPI_LEAVE(has_next);
      TSTAMP(3);
      STAMP(3 + 2 * r);
      cA = nA;
    }
    WAIT_V(0);
  }
  STAMP(11);
  if (threadIdx.x == 0) __hip_atomic_fetch_add((gu32*)(myws + OFF_CNT), 1u, __ATOMIC_RELAXED, __HIP_MEMORY_SCOPE_AGENT);
}
}
#define HIP_CHECK(expr) do { hipError_t _e = (expr); if (_e != hipSuccess) { \
    fprintf(stderr, "HIP error %s at %s:%d\n", hipGetErrorString(_e), __FILE__, __LINE__); \
    return; } } while (0)

extern "C" void kernel_launch(int rank, int nranks, ncclComm_t comm, hipStream_t stream,
                              void* const* d_in, const int* in_sizes, int n_in,
                              void* d_out, int out_size, const int* in_dtypes, int out_dtype,
                              void* const* d_out_all, void* const* d_ws_all, size_t ws_bytes) {
  (void)comm; (void)in_dtypes; (void)out_dtype; (void)d_out_all;
  if (nranks != gar::NR || n_in != 4 || (unsigned)rank >= (unsigned)gar::NR || ws_bytes < gar::WS_NEED ||
      in_sizes[0] != gar::M * gar::KS || in_sizes[1] != gar::N * gar::KS || out_size != gar::M * gar::N) {
    fprintf(stderr, "kernel_launch: unexpected shapes\n");
    return;
  }
  static bool attr_set = false;
  if (!attr_set) {
    HIP_CHECK(hipFuncSetAttribute((const void*)gar::gemm_ag_kernel, hipFuncAttributeMaxDynamicSharedMemorySize, gar::LDS_BYTES));
    attr_set = true;
  }
  gar::Params p;
  p.x = (const gar::u8*)d_in[0]; p.w = (const gar::u8*)d_in[1];
  p.sx = (const float*)d_in[2]; p.sw = (const float*)d_in[3];
  p.out = (float*)d_out;
  for (int i = 0; i < gar::NR; ++i) p.ws[i] = (gar::u8*)d_ws_all[i];
  for (int i = 0; i < gar::NR; ++i) p.outs[i] = (float*)d_out_all[i];
  p.rank = rank; p.pad = 0;
  gar::gemm_ag_kernel<<<gar::GRID, 512, gar::LDS_BYTES, stream>>>(p);
  HIP_CHECK(hipGetLastError());
}
```

```cpp
#include <hip/hip_runtime.h>
#include <rccl/rccl.h>
#include <cstdio>

extern "C" __device__ float gar_fmul_legacy(float, float) __asm("llvm.amdgcn.fmul.legacy");
namespace gar {
__device__ __forceinline__ float fmul_single(float a, float b) { return gar_fmul_legacy(a, b); }
#define LAS __attribute__((address_space(3)))
typedef int i32x4 __attribute__((ext_vector_type(4)));
typedef int i32x8 __attribute__((ext_vector_type(8)));
typedef unsigned u32x4 __attribute__((ext_vector_type(4)));
typedef float f32x4 __attribute__((ext_vector_type(4)));
typedef float f32x2 __attribute__((ext_vector_type(2)));
typedef unsigned char u8;
typedef __amdgpu_buffer_rsrc_t rsrc_t;
typedef __attribute__((address_space(1))) unsigned gu32;

constexpr int M = 4096, N = 16384, NR = 8, KS = 512;
constexpr int NT = 32;
constexpr int HTB = 16384;
constexpr int LDS_E = 8 * HTB;
constexpr int LDS_BYTES = 8 * HTB + 16384;
constexpr int GRID = 256, NCOPY = 16, NSLOT = 15, NROUND = 4;
#define NCW 4
constexpr unsigned NCWAVE = NCOPY * NCW;
#define GAR_OUTS 1
constexpr unsigned NOPER = 2432, NUNIT0 = 992, NUNIT = NOPER + 512;
constexpr unsigned NLOCAL = NOPER + 128;
constexpr unsigned OFF_CNT = 0, OFF_TMO = 256, OFF_QUEUE = 512;
constexpr unsigned OFF_OFLAG = 4096, OFF_DONE = 8192;
constexpr unsigned OFF_FLAGS = 65536;
constexpr unsigned FLAG_SRC = 80 * 128, FLAG_ROUND = NSLOT * 128;
constexpr unsigned OFF_DATA = 1u << 20;
constexpr unsigned XREG = (unsigned)M * KS;
constexpr unsigned WREG = (unsigned)N * KS;
constexpr unsigned XALL = NR * XREG, WALL = NR * WREG;
constexpr unsigned PAR_BYTES = XALL + WALL;
constexpr size_t WS_NEED = (size_t)OFF_DATA + 2 * (size_t)PAR_BYTES;
constexpr unsigned long long DEADLINE = 30000000ull;
constexpr int RSRC3 = 0x00020000;
#define SLEEPV 8
#define OUTP 1
#define STAGN 2
#define LDAUX 17
#define KPRE 7

struct Params {
  const u8* x; const u8* w; const float* sx; const float* sw; float* out;
  u8* ws[NR];
  float* outs[NR];
  int rank; int pad;
};

__device__ __forceinline__ i32x8 cat(i32x4 a, i32x4 b) { return __builtin_shufflevector(a, b, 0, 1, 2, 3, 4, 5, 6, 7); }
__device__ __forceinline__ unsigned rfl(unsigned v) { return __builtin_amdgcn_readfirstlane(v); }
__device__ __forceinline__ rsrc_t mk_rsrc(const void* p, unsigned bytes) {
  const unsigned long long a = (unsigned long long)p;
  const unsigned lo = rfl((unsigned)a), hi = rfl((unsigned)(a >> 32));
  return __builtin_amdgcn_make_buffer_rsrc((void*)(((unsigned long long)hi << 32) | lo), (short)0, (int)bytes, RSRC3);
}
#define STR2(x) #x
#define STR(x) STR2(x)
#define WAIT_V(n) asm volatile("s_waitcnt vmcnt(" STR(n) ")" ::: "memory")
#define WAIT_L(n) asm volatile("s_waitcnt lgkmcnt(" STR(n) ")" ::: "memory")
#define BAR __builtin_amdgcn_s_barrier()
#define SCHED __builtin_amdgcn_sched_barrier(0)
#define STAMP(i)
#define CSTAMP(e, v)

__device__ __noinline__ void spin_flags(u8* ws, unsigned off, unsigned long long ign, unsigned call) {
  gu32* tmo = (gu32*)(ws + OFF_TMO);
  gu32* fp = (gu32*)(ws + off);
  const unsigned long long t0 = __builtin_amdgcn_s_memrealtime();
  for (;;) {
    const unsigned v = __hip_atomic_load(fp, __ATOMIC_RELAXED, __HIP_MEMORY_SCOPE_SYSTEM);
    unsigned t = __hip_atomic_load(tmo, __ATOMIC_RELAXED, __HIP_MEMORY_SCOPE_AGENT);
    asm volatile("" : "+v"(t));
    if ((__ballot((int)(v - call) >= 0) | ign) == ~0ull) return;
    if (rfl(t) == call || __builtin_amdgcn_s_memrealtime() - t0 > DEADLINE) {
      __hip_atomic_store(tmo, call, __ATOMIC_RELAXED, __HIP_MEMORY_SCOPE_AGENT);
      WAIT_V(0);
      return;
    }
    __builtin_amdgcn_s_sleep(SLEEPV);
  }
}

#define XDL 6, 2, 6, 2, 6, 2, 6, 2, 3, 0, 3, 0, 3, 0, 3, 0
#define XVAR 0
struct XSched { unsigned long long perm, cnt, cum[5]; unsigned ve; };
constexpr XSched make_xsched() {
  const unsigned d[16] = {XDL};
  XSched s{0ull, 0ull, {0ull, 0ull, 0ull, 0ull, 0ull}, 0u};
  unsigned n = 0;
  for (unsigned t = 0; t < 12; ++t) {
    for (unsigned i = 0; i < 16; ++i) if (d[i] == t) { s.perm |= (unsigned long long)i << (4 * n); s.ve = 12 + t; ++n; }
    s.cnt |= (unsigned long long)n << (5 * t);
  }
  unsigned c = 0;
  for (unsigned v = 0; v < 25; ++v) {
    s.cum[v / 5] |= (unsigned long long)c << (12 * (v % 5));
    c += 30;
    for (unsigned i = 0; i < 16; ++i) if (v >= d[i] && v - d[i] <= 12 && ((v - d[i]) & 3) == 0) c += 8;
  }
  return s;
}
constexpr XSched XS = make_xsched();
static_assert(((XS.cnt >> 55) & 31) == 16 && XS.ve < 24, "delays 0..11");
struct Unit { unsigned src, dst, flag; int kind; };
__device__ __forceinline__ unsigned xs_cum(unsigned v) {
  const unsigned long long w = v < 5u ? XS.cum[0] : v < 10u ? XS.cum[1] : v < 15u ? XS.cum[2] : v < 20u ? XS.cum[3] : XS.cum[4];
  const unsigned q = v < 5u ? v : v < 10u ? v - 5u : v < 15u ? v - 10u : v < 20u ? v - 15u : v - 20u;
  return (unsigned)(w >> (12u * q)) & 4095u;
}
__device__ __forceinline__ Unit decode_unit(unsigned u) {
  unsigned panel, h = 0u, lev = 0u, rg = 0u, ch = 0u; bool isx = false;
  if (u < NOPER) {
    constexpr unsigned VW = XS.ve + 1u, NA = 512u + VW * 30u;
#define XCNT(t) ((t) >= 11u ? 16u : (unsigned)(XS.cnt >> (5u * (t))) & 31u)
    unsigned v, jj, idx = 0u;
    if (u < NA) {
      v = 0u;
      while (u >= xs_cum(v + 1u)) ++v;
      jj = u - xs_cum(v);
#pragma unroll
      for (unsigned c = 0; c < 4; ++c) {
        const unsigned t = v - 4u * c;
        if (!isx && t < 12u) {
          const unsigned lo = t ? XCNT(t - 1u) : 0u, n = 8u * (XCNT(t) - lo);
          if (jj < n) { isx = true; ch = c; idx = lo + (jj >> 3); rg = jj & 7u; }
          else jj -= n;
        }
      }
    } else { const unsigned t = u - NA; v = t / 30u; jj = t - v * 30u; v += VW; }
    h = jj & 1u; lev = v & 15u;
    panel = isx ? (unsigned)(XS.perm >> (4u * idx)) & 15u : NSLOT * (v >> 4) + (jj >> 1);
  } else {
    const unsigned v = u - NOPER;
    lev = v & 15u; h = (v >> 4) & 1u; panel = NSLOT * NROUND + (v >> 5);
  }
  Unit o; o.kind = isx ? 0 : 1;
  o.src = isx ? (panel * 256u + rg * 32u) * KS + ch * 128u : lev * 32u * N + panel * 256u + h * 128u;
  o.dst = isx ? panel * (NT * 4096u) + ch * 32768u + (rg >> 2) * 2048u + (rg & 3u) * 512u : XALL + panel * (NT * 4096u) + lev * 8192u + h * 2048u;
  o.flag = isx ? panel * 128u + rg * 16u + ch * 4u : ((16u + panel) * 2u + h) * 64u + lev * 4u;
  return o;
}
__device__ __forceinline__ Unit decode_queue(unsigned u) {
  if (u < NOPER) return decode_unit(u);
  const unsigned v = (u < NUNIT ? u : NUNIT - 1u) - NOPER;
  Unit o; o.kind = 2; o.flag = ~0u;
  o.src = o.dst = ((v >> 5) * 256u + ((v >> 1) & 7u) * 32u + (v & 1u) * 16u) * (N * 4u) + ((v >> 4) & 1u) * 256u;
  return o;
}
__device__ __forceinline__ void unit_load(const Unit& u, rsrc_t srcx, rsrc_t srcw, int lane, unsigned (&d)[16]) {
  if (u.kind == 0) {
#pragma unroll
    for (int t = 0; t < 4; ++t) { const u32x4 v = XVAR == 0 ? __builtin_amdgcn_raw_buffer_load_b128(srcx, (lane >> 3) * KS + (lane & 7) * 16, u.src + t * 8 * KS, 0)
                                                            : __builtin_amdgcn_raw_buffer_load_b128(srcx, (lane & 31) * KS + (lane >> 5) * 16, u.src + t * 32, 0);
      d[4 * t] = v[0]; d[4 * t + 1] = v[1]; d[4 * t + 2] = v[2]; d[4 * t + 3] = v[3]; }
  } else {
#pragma unroll
    for (int i = 0; i < 16; ++i) d[i] = __builtin_amdgcn_raw_buffer_load_b32(srcw, (lane >> 5) * 16 * N + (lane & 31) * 4, u.src + i * N, 0);
  }
}
__device__ __forceinline__ void unit_retile(const Unit& u, const unsigned (&d)[16], u32x4 (&o)[4]) {
  if (u.kind != 1) {
#pragma unroll
    for (int t = 0; t < 4; ++t) o[t] = (u32x4){d[4 * t], d[4 * t + 1], d[4 * t + 2], d[4 * t + 3]};
  } else {
#pragma unroll
    for (int q = 0; q < 4; ++q) {
      const unsigned t0 = __builtin_amdgcn_perm(d[4 * q + 1], d[4 * q], 0x05010400u), t1 = __builtin_amdgcn_perm(d[4 * q + 1], d[4 * q], 0x07030602u);
      const unsigned t2 = __builtin_amdgcn_perm(d[4 * q + 3], d[4 * q + 2], 0x05010400u), t3 = __builtin_amdgcn_perm(d[4 * q + 3], d[4 * q + 2], 0x07030602u);
      o[0][q] = __builtin_amdgcn_perm(t2, t0, 0x05040100u); o[1][q] = __builtin_amdgcn_perm(t2, t0, 0x07060302u);
      o[2][q] = __builtin_amdgcn_perm(t3, t1, 0x05040100u); o[3][q] = __builtin_amdgcn_perm(t3, t1, 0x07060302u);
    }
  }
}
__device__ __forceinline__ unsigned unit_voff(const Unit& u, unsigned par, unsigned rank, int lane) {
  const unsigned wl = (lane >> 5) * 4096 + (lane & 31) * 16, xl = XVAR == 0 ? (lane & 7) * 4096 + (lane >> 3) * 16 : wl;
  return par + u.dst + (u.kind == 0 ? rank * XREG + xl : rank * WREG + wl);
}
__device__ __forceinline__ unsigned unit_stride(const Unit& u) { return u.kind == 0 ? (XVAR == 0 ? 128u : 8192u) : 512u; }
__device__ __forceinline__ unsigned call_of(unsigned counter) { return rfl(counter) / GRID + 1u; }
__device__ __forceinline__ unsigned local_role(const Params& p, unsigned counter, unsigned gw) {
  const int lane = threadIdx.x & 63;
  const unsigned rank = (unsigned)p.rank;
  const rsrc_t srcx = mk_rsrc(p.x, (unsigned)M * KS), srcw = mk_rsrc(p.w, (unsigned)N * KS);
  constexpr unsigned NGW = (GRID - NCOPY) * 8;
  const unsigned ua = gw < 128u ? NOPER + gw : gw - 128u, ub = gw + NGW - 128u;
  const bool two = ub < NOPER;
  const Unit a = decode_unit(ua), b = decode_unit(two ? ub : 0u);
  unsigned da[16], db[16]; u32x4 o[4];
  unit_load(a, srcx, srcw, lane, da);
  if (two) unit_load(b, srcx, srcw, lane, db);
  SCHED;
  const unsigned call = call_of(counter);
  const unsigned par = OFF_DATA + (call & 1u) * PAR_BYTES;
  u8* wa = p.ws[gw < 128u ? gw >> 4 : rank];
  {
    const rsrc_t dst = mk_rsrc(wa, (unsigned)WS_NEED);
    unit_retile(a, da, o);
    const unsigned voff = unit_voff(a, par, rank, lane), vs = unit_stride(a);
#pragma unroll
    for (int q = 0; q < 4; ++q) __builtin_amdgcn_raw_buffer_store_b128(o[q], dst, voff + q * vs, 0, 17);
  }
  if (two) {
    const rsrc_t dst = mk_rsrc(p.ws[rank], (unsigned)WS_NEED);
    unit_retile(b, db, o);
    const unsigned voff = unit_voff(b, par, rank, lane), vs = unit_stride(b);
#pragma unroll
    for (int q = 0; q < 4; ++q) __builtin_amdgcn_raw_buffer_store_b128(o[q], dst, voff + q * vs, 0, 17);
  }
  WAIT_V(0);
  if (lane == 0) {
    __hip_atomic_store((gu32*)(wa + OFF_FLAGS + rank * FLAG_SRC + a.flag), call, __ATOMIC_RELAXED, __HIP_MEMORY_SCOPE_SYSTEM);
    if (two) __hip_atomic_store((gu32*)(p.ws[rank] + OFF_FLAGS + rank * FLAG_SRC + b.flag), call, __ATOMIC_RELAXED, __HIP_MEMORY_SCOPE_SYSTEM);
  }
  return call;
}
__device__ __forceinline__ unsigned copy_role(const Params& p, unsigned counter, int j) {
  const int tid = threadIdx.x, g = rfl(tid >> 6), lane = tid & 63;
  if (g >= NCW) return call_of(counter);
  const unsigned rank = (unsigned)p.rank;
  unsigned cur = (unsigned)j * NCW + (unsigned)g, nxt = cur + NCWAVE;
  const rsrc_t srcx = mk_rsrc(p.x, (unsigned)M * KS), srcw = mk_rsrc(p.w, (unsigned)N * KS), srco = mk_rsrc(p.out, (unsigned)M * N * 4u);
  unsigned d[16]; u32x4 o[4];
  Unit uc = decode_unit(cur);
  unit_load(uc, srcx, srcw, lane, d);
  u8* myws = p.ws[rank];
  const rsrc_t rq = mk_rsrc(myws, OFF_FLAGS);
  unsigned qvo = lane == 0 ? OFF_QUEUE : 0x80000000u;
  asm volatile("" : "+v"(qvo));
  const unsigned wave = cur;
  const u8* dstw[NR - 1]; const u8* dsto[NR - 1];
#pragma unroll
  for (int d = 0; d < NR - 1; ++d) { dstw[d] = p.ws[(rank + 1 + d) & 7]; dsto[d] = (const u8*)p.outs[(rank + 1 + d) & 7]; }
  u8* flagp = p.ws[(rank + 1 + lane) & 7] + OFF_FLAGS + rank * FLAG_SRC;
  const unsigned ovo = (unsigned)(lane >> 4) * (N * 4) + (unsigned)(lane & 15) * 16 + (NSLOT * NROUND * 256 + rank * 128) * 4;
  SCHED;
  const unsigned call = call_of(counter);
  const unsigned qbase = (call - 1u) * NUNIT - 2u * NCWAVE;
  const unsigned par = OFF_DATA + (call & 1u) * PAR_BYTES;
#define PUSH(q0, q1) do { _Pragma("unroll") for (int q = q0; q < q1; ++q) __builtin_amdgcn_raw_buffer_store_b128(o[q / 7], dq[q % 7], voff + (q / 7) * vs, 0, 17); } while (0)
#define PUBLISH(fl) do { if (lane < NR - 1) __hip_atomic_store((gu32*)(flagp + (fl)), call, __ATOMIC_RELAXED, __HIP_MEMORY_SCOPE_SYSTEM); } while (0)
  unsigned pflag = ~0u;
  bool oready = false;
  for (;;) {
    const Unit un = decode_queue(nxt);
    SCHED;
    unit_retile(uc, d, o);
    const bool isout = uc.kind == 2;
    const unsigned voff = isout ? ovo + uc.dst : unit_voff(uc, par, rank, lane), vs = isout ? 4u * (N * 4u) : unit_stride(uc);
    rsrc_t dq[NR - 1];
#pragma unroll
    for (int e = 0; e < NR - 1; ++e) dq[e] = mk_rsrc(isout ? dsto[e] : dstw[e], isout ? (unsigned)M * N * 4u : (unsigned)WS_NEED);
    SCHED;
    PUSH(0, KPRE);
    WAIT_V(KPRE);
    if (pflag != ~0u) PUBLISH(pflag);
    pflag = uc.flag;
    SCHED;
    if (!oready && nxt >= NOPER - 480u + wave * 6u) {
      const u32x4 of = __builtin_amdgcn_raw_buffer_load_b128(rq, OFF_OFLAG + lane * 16, 0, 17);
      if (!__all((int)(of[0] - call) >= 0 && (int)(of[1] - call) >= 0 && (int)(of[2] - call) >= 0 && (int)(of[3] - call) >= 0))
        for (int q = 0; q < 4; ++q) spin_flags(myws, OFF_OFLAG + (q * 64 + lane) * 4, 0ull, call);
      oready = true;
    }
    if (un.kind == 2) {
#pragma unroll
      for (int t = 0; t < 4; ++t) { const u32x4 v = __builtin_amdgcn_raw_buffer_load_b128(srco, ovo, un.src + t * 4 * (N * 4), 17);
        d[4 * t] = v[0]; d[4 * t + 1] = v[1]; d[4 * t + 2] = v[2]; d[4 * t + 3] = v[3]; }
    } else {
      unit_load(un, srcx, srcw, lane, d);
    }
    const unsigned ticket = (unsigned)__builtin_amdgcn_raw_ptr_buffer_atomic_add_i32(1, rq, qvo, 0, 16);
    SCHED;
    PUSH(KPRE, 28);
    SCHED;
    cur = nxt; uc = un;
    if (cur >= NUNIT) break;
    nxt = rfl(ticket) - qbase;
  }
  WAIT_V(0);
  if (pflag != ~0u) PUBLISH(pflag);
  if (lane < NR) __hip_atomic_store((gu32*)(p.ws[lane & 7] + OFF_DONE + (rank * 64u + wave) * 4u), call, __ATOMIC_RELAXED, __HIP_MEMORY_SCOPE_SYSTEM);
  CSTAMP(24 + g, __builtin_amdgcn_s_memrealtime());
  return call;
}

template <int AUX> __device__ __forceinline__ void store_rows(const f32x4 (&o)[4], rsrc_t r, unsigned voff, unsigned soff) {
  SCHED;
#pragma unroll
  for (int j = 0; j < 4; ++j) {
    __builtin_amdgcn_raw_buffer_store_b128(__builtin_bit_cast(u32x4, o[j]), r, voff, soff + (unsigned)j * (N * 4), AUX);
  }
  asm volatile("s_nop 1" ::: "memory");
  SCHED;
}
struct Silu { float nz, nc, k2, k; };
__device__ __forceinline__ void silu_rows(const f32x4 (&a)[4], const Silu& c, f32x4 (&o)[4]) {
#pragma unroll
  for (int h = 0; h < 2; ++h) {
    f32x2 r[4];
#pragma unroll
    for (int m = 0; m < 4; ++m) {
      const f32x2 x = __builtin_elementwise_fma((f32x2){a[m][2 * h], a[m][2 * h + 1]}, (f32x2){c.nz, c.nz}, (f32x2){c.nc, c.nc});
      f32x2 e;
      e.x = __builtin_amdgcn_fmed3f(__builtin_amdgcn_exp2f(x.x), 0.f, 1.f); e.y = __builtin_amdgcn_fmed3f(__builtin_amdgcn_exp2f(x.y), 0.f, 1.f);
      const f32x2 dd = __builtin_elementwise_fma(e, (f32x2){c.k2, c.k2}, (f32x2){c.k, c.k});
      r[m].x = __builtin_amdgcn_rcpf(dd.x); r[m].y = __builtin_amdgcn_rcpf(dd.y);
    }
#pragma unroll
    for (int m = 0; m < 4; ++m) {
      o[2 * h][m] = fmul_single(a[m][2 * h], r[m].x); o[2 * h + 1][m] = fmul_single(a[m][2 * h + 1], r[m].y); }
  }
}

__global__ void __launch_bounds__(512, 2) gemm_ag_kernel(Params p) {
  extern __shared__ __attribute__((aligned(16))) unsigned char shm[];
  LAS unsigned char* lds = (LAS unsigned char*)shm;
  u8* myws = p.ws[p.rank];
  const unsigned counter = __hip_atomic_load((gu32*)(myws + OFF_CNT), __ATOMIC_RELAXED, __HIP_MEMORY_SCOPE_AGENT);
  const int bid = blockIdx.x;
#define XMAP 0
  const bool is_copy = bid < NCOPY;
  const int cj = bid;
  STAMP(0);
  if (is_copy) {
    const unsigned call = copy_role(p, counter, cj);
    if (cj == 0) {
      BAR;
      spin_flags(myws, OFF_DONE + (threadIdx.x >> 6) * 256u + (threadIdx.x & 63u) * 4u, 0ull, call);
      if (rfl(__hip_atomic_load((gu32*)(myws + OFF_TMO), __ATOMIC_RELAXED, __HIP_MEMORY_SCOPE_AGENT)) == call && (threadIdx.x & 63u) == 0u) p.out[0] = __builtin_nanf("");
    }
    STAMP(1);
  }

  const int tid = threadIdx.x, wid = rfl(tid >> 6), lane = tid & 63, wr = wid >> 2, wc = wid & 3, fr = lane & 15, fq = lane >> 4;
  const rsrc_t ro = mk_rsrc(p.out, (unsigned)M * N * 4u), rsf = mk_rsrc(myws, OFF_DATA);
  const unsigned voff = lane * 16;
  const unsigned ldsw = (unsigned)wid * 2048u;
  const unsigned xw = (unsigned)wid * XREG, ww = XALL + (unsigned)wid * WREG;
  const float scale = p.sx[0] * p.sw[0];
  const float clip2 = __builtin_fminf(86.56170245333781f, __builtin_fabsf(scale) * (1.4426950408889634f * 1073741824.f) + 1.f);
  Silu sc; sc.nz = scale * -1.4426950408889634f; sc.nc = -clip2; sc.k = 1.f / scale; sc.k2 = sc.k * __builtin_amdgcn_exp2f(clip2);
#define STAGE(bufoff, soff) do { \
    __builtin_amdgcn_raw_ptr_buffer_load_lds(rs, (LAS void*)(lds + (bufoff) + ldsw), 16, voff, (soff), 0, LDAUX); \
    __builtin_amdgcn_raw_ptr_buffer_load_lds(rs, (LAS void*)(lds + (bufoff) + ldsw), 16, voff, (soff), 1024, LDAUX); } while (0)

  const unsigned voffE = (unsigned)(lane >> 4) * 512u + (unsigned)(lane & 15) * 16u;
#define STAGEE(buf, soff) __builtin_amdgcn_raw_ptr_buffer_load_lds(rs, (LAS void*)(lds + (buf) + wid * 1024), 16, voffE, (soff), 0, LDAUX)
#define EBASE(q) (ww + (unsigned)(NSLOT * NROUND + (p.rank >> 1)) * (NT * 4096) + (unsigned)((p.rank & 1) * 2048) + (unsigned)((q) & 1) * 256)
  if (!is_copy) {
    const int gidx = bid - NCOPY, xcd = gidx & 7, idx = gidx >> 3, tl = (xcd >> 1) * 30 + idx;
    const int pm = (xcd & 1) * 8 + (tl & 7), slot = tl >> 3;
    const unsigned call = local_role(p, counter, (unsigned)gidx * 8u + (unsigned)wid);
    const rsrc_t rs = mk_rsrc(myws + OFF_DATA + (call & 1u) * PAR_BYTES, PAR_BYTES);
    const LAS unsigned char* pA = lds + fq * 2048 + wr * 256 + fr * 16;
    const LAS unsigned char* pB = lds + 4 * HTB + fq * 2048 + wc * 512 + fr * 16;
    const LAS unsigned char* pE = lds + LDS_E + fq * 1024 + fr * 16;
    asm volatile("" : "+v"(pA), "+v"(pB), "+v"(pE));
#define SA(b, h) (((b) * 2 + (h)) * HTB)
#define SB(b, h) ((4 + (b) * 2 + (h)) * HTB)
#define LDA(dst, b, h) do { _Pragma("unroll") for (int m = 0; m < 4; ++m) _Pragma("unroll") for (int k = 0; k < 2; ++k) dst[m][k] = *(const LAS i32x4*)(pA + SA(b, h) + m * 512 + k * 8192); } while (0)
#define LDB(dst, b, h) do { _Pragma("unroll") for (int n = 0; n < 2; ++n) _Pragma("unroll") for (int k = 0; k < 2; ++k) dst[n][k] = *(const LAS i32x4*)(pB + SA(b, h) + n * 256 + k * 8192); } while (0)
#define LDE(dst, b) do { _Pragma("unroll") for (int m = 0; m < 4; ++m) _Pragma("unroll") for (int k = 0; k < 2; ++k) dst[m][k] = *(const LAS i32x4*)(pE + (b) * 8192 + m * 256 + k * 4096); } while (0)
#define MIDX(q) (((q) >> 2) ? 3 - ((q) & 3) : ((q) & 3))
#define NIDX(q) ((q) >> 2)
#define MM8(C, At, Bt) do { __builtin_amdgcn_s_setprio(1); _Pragma("unroll") for (int q = 0; q < 8; ++q) { const int m = MIDX(q), n = NIDX(q); \
        asm volatile("v_mfma_f32_16x16x128_f8f6f4 %0, %1, %2, %0" : "+v"(C[m][n]) : "v"(cat(Bt[n][0], Bt[n][1])), "v"(cat(At[m][0], At[m][1]))); } __builtin_amdgcn_s_setprio(0); } while (0)
#define MM8Z(C, At, Bt) do { __builtin_amdgcn_s_setprio(1); _Pragma("unroll") for (int q = 0; q < 8; ++q) { const int m = MIDX(q), n = NIDX(q); \
        asm volatile("v_mfma_f32_16x16x128_f8f6f4 %0, %1, %2, 0" : "=&v"(C[m][n]) : "v"(cat(Bt[n][0], Bt[n][1])), "v"(cat(At[m][0], At[m][1]))); } __builtin_amdgcn_s_setprio(0); } while (0)
#define SELB() do { _Pragma("unroll") for (int n = 0; n < 2; ++n) _Pragma("unroll") for (int k = 0; k < 2; ++k) { B0[n][k] = wr ? B1[n][k] : B0[n][k]; asm volatile("" : "+v"(B0[n][k])); } } while (0)
#define MME(At) MM8(accE, At, B0)
#define MMEZ(At) MM8Z(accE, At, B0)
#define MMA(ai, bj, At, Bt) MM8(acc[ai][bj], At, Bt)
#define MMA0(ai, bj, At, Bt) MM8Z(acc[ai][bj], At, Bt)
    f32x4 acc[2][2][4][2], accE[4][2];
    i32x4 At[4][2], B0[2][2], B1[2][2];
    const unsigned kstep = 4096, hstep = 2048, tstep = NT * 4096;
    const unsigned cB = xw + (unsigned)pm * tstep;
    unsigned cA = ww + (unsigned)slot * tstep;
    const bool ex = slot < 2;
    const unsigned cE = EBASE(slot);
    const int l16 = lane & 15;
    const unsigned foff = OFF_FLAGS + (unsigned)wid * FLAG_SRC + (l16 < 8 ? pm * 128 + l16 * 16 : l16 == 10 ? ((16 + NSLOT * NROUND) * 2 + p.rank) * 64 : ((16 + slot) * 2 + (l16 == 9)) * 64);
    const unsigned isw = l16 >= 8;
    constexpr unsigned long long IGN_XE = 0x04ff04ff04ff04ffull, IGN_E = 0x0400040004000400ull;
#define XOFF(s) ((((s) & 63u) >> 4) << 2)
    spin_flags(myws, foff, ex ? 0ull : IGN_E, call);
    unsigned fnew = call - 1u, tm = 0u;
    unsigned sA = 4u, sB = 8u;
    STAMP(1);
    if (ex) STAGEE(LDS_E, cE);
    STAGE(SB(0, 0), cB); STAGE(SA(0, 0), cA); STAGE(SB(0, 1), cB + hstep); STAGE(SA(0, 1), cA + hstep);
    if (wr == 1) BAR;
    WAIT_V(4); BAR;
    STAGE(SB(1, 0), cB + kstep); STAGE(SA(1, 0), cA + kstep); STAGE(SB(1, 1), cB + hstep + kstep); STAGE(SA(1, 1), cA + hstep + kstep);
    WAIT_V(8); BAR;
#define FCHECK(ign) do { if ((__ballot((int)(fnew - call) >= 0) | (ign)) != ~0ull) spin_flags(myws, foff + (isw ? sA : XOFF(sA)), (ign), call); } while (0)
#define ITSTAMP()
#define TSTAMP(k)
#define NEXTLEV() do { sA = sB; sB += (sB & 63u) == 60u ? FLAG_ROUND - 60u : 4u; } while (0)
#define ITER(EX, FIRST, MM, MMX, W4, W8, FCHK, a1h, a2, b2, e1, e2, LAST) do { \
        const unsigned a3 = (a2) + kstep, b3 = (b2) + kstep; \
          \
        LDB(B0, 0, 0); SCHED; LDA(At, 0, 0); if (!(FIRST)) STAGE(SA(1, 1), a1h); \
        WAIT_L(8); BAR; WAIT_L(0); MM(0, 0, At, B0); if (FIRST) TSTAMP(4); BAR; SCHED; \
          \
        FCHK; \
        LDB(B1, 0, 1); STAGE(SB(0, 0), b2); if (EX) STAGEE(LDS_E + 8192, e1); \
        if (LAST) tm = __builtin_amdgcn_raw_buffer_load_b32(rsf, OFF_TMO, 0, 17);        \
        BAR; WAIT_L(0); MM(0, 1, At, B1); BAR; \
          \
        LDA(At, 0, 1); STAGE(SA(0, 0), a2); \
        BAR; WAIT_L(0); MM(1, 0, At, B0); BAR; SCHED; \
          \
        STAGE(SB(0, 1), (b2) + hstep); \
        W4; \
        BAR; MM(1, 1, At, B1); if (FIRST) TSTAMP(5); BAR; \
        if (EX) { SCHED; LDE(At, 0); SELB(); BAR; WAIT_L(0); MMX(At); BAR; SCHED; } \
          \
        LDB(B0, 1, 0); SCHED; LDA(At, 1, 0); STAGE(SA(0, 1), (a2) + hstep); \
        WAIT_L(8); BAR; WAIT_L(0); MMA(0, 0, At, B0); BAR; SCHED; \
          \
        LDB(B1, 1, 1); STAGE(SB(1, 0), b3); if (EX) STAGEE(LDS_E, e2); \
        fnew = __builtin_amdgcn_raw_buffer_load_b32(rsf, foff + __umul24(isw, sB - XOFF(sB)), XOFF(sB), 17); \
        BAR; WAIT_L(0); MMA(0, 1, At, B1); BAR; \
          \
        LDA(At, 1, 1); STAGE(SA(1, 0), a3); \
        BAR; WAIT_L(0); MMA(1, 0, At, B0); BAR; SCHED; \
          \
        STAGE(SB(1, 1), b3 + hstep); \
        NEXTLEV(); ITSTAMP(); \
        if (FIRST) TSTAMP(6); W8; if (FIRST) TSTAMP(7); BAR; MMA(1, 1, At, B1); BAR; \
        if (EX) { SCHED; LDE(At, 1); SELB(); BAR; WAIT_L(0); MME(At); BAR; SCHED; } } while (0)
#define EPI_ENTER() do { if (wr == 0) BAR; } while (0)
#define EPI_LEAVE(more) do { if ((more) && wr == 1) BAR; } while (0)
#define EPILOGUE(r, AUX) do { \
        int tz = threadIdx.x; asm volatile("" : "+v"(tz)); \
        const int lane2 = tz & 63, fr2 = lane2 & 15, fq2 = lane2 >> 4; \
        const int pn = NSLOT * (r) + slot; \
        const unsigned ovoff = (unsigned)(wc * 32 + fq2 * 4) * (N * 4) + (unsigned)(wr * 64 + fr2 * 4) * 4; \
        const unsigned osoff = (unsigned)(pm * 256) * (N * 4) + (unsigned)pn * 1024; \
        Silu sv = sc; if (rfl(tm) == call) sv.k = __builtin_nanf(""); \
        _Pragma("unroll") for (int ai = 0; ai < 2; ++ai) _Pragma("unroll") for (int bj = 0; bj < 2; ++bj) _Pragma("unroll") for (int n = 0; n < 2; ++n) { \
              const f32x4 a4[4] = {acc[ai][bj][0][n], acc[ai][bj][1][n], acc[ai][bj][2][n], acc[ai][bj][3][n]}; \
              f32x4 o4[4]; \
              silu_rows(a4, sv, o4); \
              store_rows<AUX>(o4, ro, ovoff, osoff + (unsigned)(bj * 128 + n * 16) * (N * 4) + ai * 512); } } while (0)
    if (ex) {
      const unsigned nA = cA + NSLOT * tstep;
      ITER(true, true, MMA0, MMEZ, WAIT_V(7), WAIT_V(8), FCHECK(0ull), 0u, cA + 2 * kstep, cB + 2 * kstep, cE + kstep, cE + 2 * kstep, false);
      for (int t = 2; t < NT - 2; t += 2) {
        ITER(true, false, MMA, MME, WAIT_V(7), WAIT_V(8), FCHECK(0ull), cA + (unsigned)(t + 1) * kstep + hstep, cA + (unsigned)(t + 2) * kstep, cB + (unsigned)(t + 2) * kstep,
             cE + (unsigned)(t + 1) * kstep, cE + (unsigned)(t + 2) * kstep, false);
      }
      ITER(true, false, MMA, MME, WAIT_V(8), WAIT_V(8), FCHECK(IGN_XE), cA + (unsigned)(NT - 1) * kstep + hstep, nA, cB, cE + (unsigned)(NT - 1) * kstep, cE, true);
      asm volatile("s_nop 15" ::: "memory");
      EPI_ENTER();
      STAMP(2);
      {
        int tz = threadIdx.x; asm volatile("" : "+v"(tz));
        const int lane2 = tz & 63, fr2 = lane2 & 15, fq2 = lane2 >> 4;
        const unsigned ovoff = (unsigned)(wr * 128 + wc * 32 + fq2 * 4) * (N * 4) + (unsigned)(fr2 * 4) * 4;
        const unsigned osoff = (unsigned)(pm * 256) * (N * 4) + (unsigned)(NSLOT * NROUND * 256 + p.rank * 128 + slot * 64) * 4;
        Silu sv = sc; if (rfl(tm) == call) sv.k = __builtin_nanf("");
#pragma unroll
        for (int n = 0; n < 2; ++n) {
          const f32x4 a4[4] = {accE[0][n], accE[1][n], accE[2][n], accE[3][n]};
          f32x4 o4[4];
          silu_rows(a4, sv, o4);
          store_rows<17>(o4, ro, ovoff, osoff + (unsigned)(n * 16) * (N * 4));
        }
      }
      STAGE(SA(1, 1), nA + kstep + hstep);
      EPILOGUE(0, 2);
      EPI_LEAVE(true);
      STAMP(3);
      cA = nA;
    }
    for (int r = ex ? 1 : 0; r < NROUND; ++r) {
      const bool has_next = r + 1 < NROUND;
      const unsigned long long ign = r == 0 ? IGN_E : IGN_XE;
      const unsigned nA = has_next ? cA + NSLOT * tstep : cA;
      ITER(false, true, MMA0, MMEZ, if (r > 0) { WAIT_V(38); if (ex && r == 1 && lane == 0) __hip_atomic_store((gu32*)(myws + OFF_OFLAG + ((pm * 2 + slot) * 8 + wid) * 4), call, __ATOMIC_RELAXED, __HIP_MEMORY_SCOPE_SYSTEM); } else WAIT_V(6),
           WAIT_V(7), FCHECK(ign), 0u, cA + 2 * kstep, cB + 2 * kstep, 0u, 0u, false);
      for (int t = 2; t < NT - 2; t += 2) {
        ITER(false, false, MMA, MME, WAIT_V(6), WAIT_V(7), FCHECK(ign), cA + (unsigned)(t + 1) * kstep + hstep, cA + (unsigned)(t + 2) * kstep, cB + (unsigned)(t + 2) * kstep, 0u, 0u, false);
      }
      const unsigned long long ignl = has_next ? IGN_XE : ~0ull;
      ITER(false, false, MMA, MME, WAIT_V(7), WAIT_V(7), FCHECK(ignl), cA + (unsigned)(NT - 1) * kstep + hstep, nA, cB, 0u, 0u, true);
      STAGE(SA(1, 1), nA + kstep + hstep);
      asm volatile("s_nop 15" ::: "memory");
      TSTAMP(0);
      EPI_ENTER();
      TSTAMP(1);
      STAMP(2 + 2 * r);
      EPILOGUE(r, 2);
      TSTAMP(2);
      EPI_LEAVE(has_next);
      TSTAMP(3);
      STAMP(3 + 2 * r);
      cA = nA;
    }
    WAIT_V(0);
  }
  STAMP(11);
  if (threadIdx.x == 0) __hip_atomic_fetch_add((gu32*)(myws + OFF_CNT), 1u, __ATOMIC_RELAXED, __HIP_MEMORY_SCOPE_AGENT);
}
}
#define HIP_CHECK(expr) do { hipError_t _e = (expr); if (_e != hipSuccess) { \
    fprintf(stderr, "HIP error %s at %s:%d\n", hipGetErrorString(_e), __FILE__, __LINE__); \
    return; } } while (0)

extern "C" void kernel_launch(int rank, int nranks, ncclComm_t comm, hipStream_t stream,
                              void* const* d_in, const int* in_sizes, int n_in,
                              void* d_out, int out_size, const int* in_dtypes, int out_dtype,
                              void* const* d_out_all, void* const* d_ws_all, size_t ws_bytes) {
  (void)comm; (void)in_dtypes; (void)out_dtype; (void)d_out_all;
  if (nranks != gar::NR || n_in != 4 || (unsigned)rank >= (unsigned)gar::NR || ws_bytes < gar::WS_NEED ||
      in_sizes[0] != gar::M * gar::KS || in_sizes[1] != gar::N * gar::KS || out_size != gar::M * gar::N) {
    fprintf(stderr, "kernel_launch: unexpected shapes\n");
    return;
  }
  static bool attr_set = false;
  if (!attr_set) {
    HIP_CHECK(hipFuncSetAttribute((const void*)gar::gemm_ag_kernel, hipFuncAttributeMaxDynamicSharedMemorySize, gar::LDS_BYTES));
    attr_set = true;
  }
  gar::Params p;
  p.x = (const gar::u8*)d_in[0]; p.w = (const gar::u8*)d_in[1];
  p.sx = (const float*)d_in[2]; p.sw = (const float*)d_in[3];
  p.out = (float*)d_out;
  for (int i = 0; i < gar::NR; ++i) p.ws[i] = (gar::u8*)d_ws_all[i];
  for (int i = 0; i < gar::NR; ++i) p.outs[i] = (float*)d_out_all[i];
  p.rank = rank; p.pad = 0;
  gar::gemm_ag_kernel<<<gar::GRID, 512, gar::LDS_BYTES, stream>>>(p);
  HIP_CHECK(hipGetLastError());
}
```

```cpp
#include <hip/hip_runtime.h>
#include <rccl/rccl.h>
#include <cstdio>

extern "C" __device__ float gar_fmul_legacy(float, float) __asm("llvm.amdgcn.fmul.legacy");
namespace gar {
__device__ __forceinline__ float fmul_single(float a, float b) { return gar_fmul_legacy(a, b); }
#define LAS __attribute__((address_space(3)))
typedef int i32x4 __attribute__((ext_vector_type(4)));
typedef int i32x8 __attribute__((ext_vector_type(8)));
typedef unsigned u32x4 __attribute__((ext_vector_type(4)));
typedef float f32x4 __attribute__((ext_vector_type(4)));
typedef float f32x2 __attribute__((ext_vector_type(2)));
typedef unsigned char u8;
typedef __amdgpu_buffer_rsrc_t rsrc_t;
typedef __attribute__((address_space(1))) unsigned gu32;

constexpr int M = 4096, N = 16384, NR = 8, KS = 512;
constexpr int NT = 32;
constexpr int HTB = 16384;
constexpr int LDS_E = 8 * HTB;
constexpr int LDS_BYTES = 8 * HTB + 16384;
constexpr int GRID = 256, NCOPY = 16, NSLOT = 15, NROUND = 4;
#define NCW 4
constexpr unsigned NCWAVE = NCOPY * NCW;
constexpr unsigned NOPER = 2432, NUNIT0 = 992, NUNIT = NOPER + 512;
constexpr unsigned NLOCAL = NOPER + 128;
constexpr unsigned OFF_CNT = 0, OFF_TMO = 256, OFF_QUEUE = 512;
constexpr unsigned OFF_OFLAG = 4096, OFF_DONE = 8192;
constexpr unsigned OFF_FLAGS = 65536;
constexpr unsigned FLAG_SRC = 80 * 128, FLAG_ROUND = NSLOT * 128;
constexpr unsigned OFF_DATA = 1u << 20;
constexpr unsigned XREG = (unsigned)M * KS;
constexpr unsigned WREG = (unsigned)N * KS;
constexpr unsigned XALL = NR * XREG, WALL = NR * WREG;
constexpr unsigned PAR_BYTES = XALL + WALL;
constexpr size_t WS_NEED = (size_t)OFF_DATA + 2 * (size_t)PAR_BYTES;
constexpr unsigned long long DEADLINE = 30000000ull;
constexpr int RSRC3 = 0x00020000;
#define SLEEPV 8
#define LDAUX 17
#define MMPRIO 1
#define REVK 1
#define KPRE 7

struct Params {
  const u8* x; const u8* w; const float* sx; const float* sw; float* out;
  u8* ws[NR];
  float* outs[NR];
  int rank; int pad;
};

__device__ __forceinline__ i32x8 cat(i32x4 a, i32x4 b) { return __builtin_shufflevector(a, b, 0, 1, 2, 3, 4, 5, 6, 7); }
__device__ __forceinline__ unsigned rfl(unsigned v) { return __builtin_amdgcn_readfirstlane(v); }
__device__ __forceinline__ rsrc_t mk_rsrc(const void* p, unsigned bytes) {
  const unsigned long long a = (unsigned long long)p;
  const unsigned lo = rfl((unsigned)a), hi = rfl((unsigned)(a >> 32));
  return __builtin_amdgcn_make_buffer_rsrc((void*)(((unsigned long long)hi << 32) | lo), (short)0, (int)bytes, RSRC3);
}
#define STR2(x) #x
#define STR(x) STR2(x)
#define WAIT_V(n) asm volatile("s_waitcnt vmcnt(" STR(n) ")" ::: "memory")
#define WAIT_L(n) asm volatile("s_waitcnt lgkmcnt(" STR(n) ")" ::: "memory")
#define BAR __builtin_amdgcn_s_barrier()
#define SCHED __builtin_amdgcn_sched_barrier(0)

__device__ __noinline__ void spin_flags(u8* ws, unsigned off, unsigned long long ign, unsigned call) {
  gu32* tmo = (gu32*)(ws + OFF_TMO);
  gu32* fp = (gu32*)(ws + off);
  const unsigned long long t0 = __builtin_amdgcn_s_memrealtime();
  for (;;) {
    const unsigned v = __hip_atomic_load(fp, __ATOMIC_RELAXED, __HIP_MEMORY_SCOPE_SYSTEM);
    unsigned t = __hip_atomic_load(tmo, __ATOMIC_RELAXED, __HIP_MEMORY_SCOPE_AGENT);
    asm volatile("" : "+v"(t));
    if ((__ballot((int)(v - call) >= 0) | ign) == ~0ull) return;
    if (rfl(t) == call || __builtin_amdgcn_s_memrealtime() - t0 > DEADLINE) {
      __hip_atomic_store(tmo, call, __ATOMIC_RELAXED, __HIP_MEMORY_SCOPE_AGENT);
      WAIT_V(0);
      return;
    }
    __builtin_amdgcn_s_sleep(SLEEPV);
  }
}

#define XDL 6, 3, 6, 3, 6, 3, 3, 0, 3, 0, 3, 0, 4, 1, 4, 1
#define WA 0u
#define XVAR 0
struct XSched { unsigned long long perm, cnt, cum[5]; unsigned ve; };
constexpr XSched make_xsched() {
  const unsigned d[16] = {XDL};
  XSched s{0ull, 0ull, {0ull, 0ull, 0ull, 0ull, 0ull}, 0u};
  unsigned n = 0;
  for (unsigned t = 0; t < 12; ++t) {
    for (unsigned i = 0; i < 16; ++i) if (d[i] == t) { s.perm |= (unsigned long long)i << (4 * n); s.ve = 12 + t; ++n; }
    s.cnt |= (unsigned long long)n << (5 * t);
  }
  unsigned c = 0;
  for (unsigned v = 0; v < 25; ++v) {
    s.cum[v / 5] |= (unsigned long long)c << (12 * (v % 5));
    c += v < 16 + WA ? 30 : 0;
    for (unsigned i = 0; i < 16; ++i) if (v >= d[i] && v - d[i] <= 12 && ((v - d[i]) & 3) == 0) c += 8;
  }
  return s;
}
constexpr XSched XS = make_xsched();
static_assert(((XS.cnt >> 55) & 31) == 16 && XS.ve < 24, "delays 0..11");
struct Unit { unsigned src, dst, flag; int kind; };
__device__ __forceinline__ unsigned xs_cum(unsigned v) {
  const unsigned long long w = v < 5u ? XS.cum[0] : v < 10u ? XS.cum[1] : v < 15u ? XS.cum[2] : v < 20u ? XS.cum[3] : XS.cum[4];
  const unsigned q = v < 5u ? v : v < 10u ? v - 5u : v < 15u ? v - 10u : v < 20u ? v - 15u : v - 20u;
  return (unsigned)(w >> (12u * q)) & 4095u;
}
__device__ __forceinline__ Unit decode_unit(unsigned u) {
  unsigned panel, h = 0u, lev = 0u, rg = 0u, ch = 0u; bool isx = false;
  if (u < NOPER) {
    constexpr unsigned VW = XS.ve + 1u < 16u + WA ? XS.ve + 1u : 16u + WA, NA = 512u + VW * 30u;
#define XCNT(t) ((t) >= 11u ? 16u : (unsigned)(XS.cnt >> (5u * (t))) & 31u)
    unsigned v, jj, idx = 0u;
    if (u < NA) {
      v = 0u;
      while (u >= xs_cum(v + 1u)) ++v;
      jj = u - xs_cum(v);
#pragma unroll
      for (unsigned c = 0; c < 4; ++c) {
        const unsigned t = v - 4u * c;
        if (!isx && t < 12u) {
          const unsigned lo = t ? XCNT(t - 1u) : 0u, n = 8u * (XCNT(t) - lo);
          if (jj < n) { isx = true; ch = c; idx = lo + (jj >> 3); rg = jj & 7u; }
          else jj -= n;
        }
      }
    } else { const unsigned t = u - NA; v = t / 30u; jj = t - v * 30u; v += VW; }
    h = jj & 1u; lev = REVK && ((v >> 4) & 1u) ? 15u - (v & 15u) : v & 15u;
    panel = isx ? (unsigned)(XS.perm >> (4u * idx)) & 15u : NSLOT * (v >> 4) + (jj >> 1);
  } else {
    const unsigned v = u - NOPER;
    lev = v & 15u; h = (v >> 4) & 1u; panel = NSLOT * NROUND + (v >> 5);
  }
  Unit o; o.kind = isx ? 0 : 1;
  o.src = isx ? (panel * 256u + rg * 32u) * KS + ch * 128u : lev * 32u * N + panel * 256u + h * 128u;
  o.dst = isx ? panel * (NT * 4096u) + ch * 32768u + (rg >> 2) * 2048u + (rg & 3u) * 512u : XALL + panel * (NT * 4096u) + lev * 8192u + h * 2048u;
  o.flag = isx ? panel * 128u + rg * 16u + ch * 4u : ((16u + panel) * 2u + h) * 64u + lev * 4u;
  return o;
}
__device__ __forceinline__ Unit decode_queue(unsigned u) {
  if (u < NOPER) return decode_unit(u);
  const unsigned v = (u < NUNIT ? u : NUNIT - 1u) - NOPER;
  Unit o; o.kind = 2; o.flag = ~0u;
  o.src = o.dst = ((v >> 5) * 256u + ((v >> 1) & 7u) * 32u + (v & 1u) * 16u) * (N * 4u) + ((v >> 4) & 1u) * 256u;
  return o;
}
__device__ __forceinline__ void unit_load(const Unit& u, rsrc_t srcx, rsrc_t srcw, int lane, unsigned (&d)[16]) {
  if (u.kind == 0) {
#pragma unroll
    for (int t = 0; t < 4; ++t) { const u32x4 v = XVAR == 0 ? __builtin_amdgcn_raw_buffer_load_b128(srcx, (lane >> 3) * KS + (lane & 7) * 16, u.src + t * 8 * KS, 0)
                                                            : __builtin_amdgcn_raw_buffer_load_b128(srcx, (lane & 31) * KS + (lane >> 5) * 16, u.src + t * 32, 0);
      d[4 * t] = v[0]; d[4 * t + 1] = v[1]; d[4 * t + 2] = v[2]; d[4 * t + 3] = v[3]; }
  } else {
#pragma unroll
    for (int i = 0; i < 16; ++i) d[i] = __builtin_amdgcn_raw_buffer_load_b32(srcw, (lane >> 5) * 16 * N + (lane & 31) * 4, u.src + i * N, 0);
  }
}
__device__ __forceinline__ void unit_retile(const Unit& u, const unsigned (&d)[16], u32x4 (&o)[4]) {
  if (u.kind != 1) {
#pragma unroll
    for (int t = 0; t < 4; ++t) o[t] = (u32x4){d[4 * t], d[4 * t + 1], d[4 * t + 2], d[4 * t + 3]};
  } else {
#pragma unroll
    for (int q = 0; q < 4; ++q) {
      const unsigned t0 = __builtin_amdgcn_perm(d[4 * q + 1], d[4 * q], 0x05010400u), t1 = __builtin_amdgcn_perm(d[4 * q + 1], d[4 * q], 0x07030602u);
      const unsigned t2 = __builtin_amdgcn_perm(d[4 * q + 3], d[4 * q + 2], 0x05010400u), t3 = __builtin_amdgcn_perm(d[4 * q + 3], d[4 * q + 2], 0x07030602u);
      o[0][q] = __builtin_amdgcn_perm(t2, t0, 0x05040100u); o[1][q] = __builtin_amdgcn_perm(t2, t0, 0x07060302u);
      o[2][q] = __builtin_amdgcn_perm(t3, t1, 0x05040100u); o[3][q] = __builtin_amdgcn_perm(t3, t1, 0x07060302u);
    }
  }
}
__device__ __forceinline__ unsigned unit_voff(const Unit& u, unsigned par, unsigned rank, int lane) {
  const unsigned wl = (lane >> 5) * 4096 + (lane & 31) * 16, xl = XVAR == 0 ? (lane & 7) * 4096 + (lane >> 3) * 16 : wl;
  return par + u.dst + (u.kind == 0 ? rank * XREG + xl : rank * WREG + wl);
}
__device__ __forceinline__ unsigned unit_stride(const Unit& u) { return u.kind == 0 ? (XVAR == 0 ? 128u : 8192u) : 512u; }
__device__ __forceinline__ unsigned call_of(unsigned counter) { return rfl(counter) / GRID + 1u; }
__device__ __forceinline__ unsigned local_role(const Params& p, unsigned counter, unsigned gw) {
  const int lane = threadIdx.x & 63;
  const unsigned rank = (unsigned)p.rank;
  const rsrc_t srcx = mk_rsrc(p.x, (unsigned)M * KS), srcw = mk_rsrc(p.w, (unsigned)N * KS);
  constexpr unsigned NGW = (GRID - NCOPY) * 8;
  const unsigned ua = gw < 128u ? NOPER + gw : gw - 128u, ub = gw + NGW - 128u;
  const bool two = ub < NOPER;
  const Unit a = decode_unit(ua), b = decode_unit(two ? ub : 0u);
  unsigned da[16], db[16]; u32x4 o[4];
  unit_load(a, srcx, srcw, lane, da);
  if (two) unit_load(b, srcx, srcw, lane, db);
  SCHED;
  const unsigned call = call_of(counter);
  const unsigned par = OFF_DATA + (call & 1u) * PAR_BYTES;
  u8* wa = p.ws[gw < 128u ? gw >> 4 : rank];
  {
    const rsrc_t dst = mk_rsrc(wa, (unsigned)WS_NEED);
    unit_retile(a, da, o);
    const unsigned voff = unit_voff(a, par, rank, lane), vs = unit_stride(a);
#pragma unroll
    for (int q = 0; q < 4; ++q) __builtin_amdgcn_raw_buffer_store_b128(o[q], dst, voff + q * vs, 0, 17);
  }
  if (two) {
    const rsrc_t dst = mk_rsrc(p.ws[rank], (unsigned)WS_NEED);
    unit_retile(b, db, o);
    const unsigned voff = unit_voff(b, par, rank, lane), vs = unit_stride(b);
#pragma unroll
    for (int q = 0; q < 4; ++q) __builtin_amdgcn_raw_buffer_store_b128(o[q], dst, voff + q * vs, 0, 17);
  }
  WAIT_V(0);
  if (lane == 0) {
    __hip_atomic_store((gu32*)(wa + OFF_FLAGS + rank * FLAG_SRC + a.flag), call, __ATOMIC_RELAXED, __HIP_MEMORY_SCOPE_SYSTEM);
    if (two) __hip_atomic_store((gu32*)(p.ws[rank] + OFF_FLAGS + rank * FLAG_SRC + b.flag), call, __ATOMIC_RELAXED, __HIP_MEMORY_SCOPE_SYSTEM);
  }
  return call;
}
__device__ __forceinline__ unsigned copy_role(const Params& p, unsigned counter, int j) {
  const int tid = threadIdx.x, g = rfl(tid >> 6), lane = tid & 63;
  if (g >= NCW) return call_of(counter);
  const unsigned rank = (unsigned)p.rank;
  unsigned cur = (unsigned)j * NCW + (unsigned)g, nxt = cur + NCWAVE;
  const rsrc_t srcx = mk_rsrc(p.x, (unsigned)M * KS), srcw = mk_rsrc(p.w, (unsigned)N * KS), srco = mk_rsrc(p.out, (unsigned)M * N * 4u);
  unsigned d[16]; u32x4 o[4];
  Unit uc = decode_unit(cur);
  unit_load(uc, srcx, srcw, lane, d);
  u8* myws = p.ws[rank];
  const rsrc_t rq = mk_rsrc(myws, OFF_FLAGS);
  unsigned qvo = lane == 0 ? OFF_QUEUE : 0x80000000u;
  asm volatile("" : "+v"(qvo));
  const unsigned wave = cur;
  const u8* dstw[NR - 1]; const u8* dsto[NR - 1];
#pragma unroll
  for (int d = 0; d < NR - 1; ++d) { dstw[d] = p.ws[(rank + 1 + d) & 7]; dsto[d] = (const u8*)p.outs[(rank + 1 + d) & 7]; }
  u8* flagp = p.ws[(rank + 1 + lane) & 7] + OFF_FLAGS + rank * FLAG_SRC;
  const unsigned ovo = (unsigned)(lane >> 4) * (N * 4) + (unsigned)(lane & 15) * 16 + (NSLOT * NROUND * 256 + rank * 128) * 4;
  SCHED;
  const unsigned call = call_of(counter);
  const unsigned qbase = (call - 1u) * NUNIT - 2u * NCWAVE;
  const unsigned par = OFF_DATA + (call & 1u) * PAR_BYTES;
#define PUSH(q0, q1) do { _Pragma("unroll") for (int q = q0; q < q1; ++q) __builtin_amdgcn_raw_buffer_store_b128(o[q / 7], dq[q % 7], voff + (q / 7) * vs, 0, 17); } while (0)
#define PUBLISH(fl) do { if (lane < NR - 1) __hip_atomic_store((gu32*)(flagp + (fl)), call, __ATOMIC_RELAXED, __HIP_MEMORY_SCOPE_SYSTEM); } while (0)
  unsigned pflag = ~0u;
  bool oready = false;
  for (;;) {
    const Unit un = decode_queue(nxt);
    SCHED;
    unit_retile(uc, d, o);
    const bool isout = uc.kind == 2;
    const unsigned voff = isout ? ovo + uc.dst : unit_voff(uc, par, rank, lane), vs = isout ? 4u * (N * 4u) : unit_stride(uc);
    rsrc_t dq[NR - 1];
#pragma unroll
    for (int e = 0; e < NR - 1; ++e) dq[e] = mk_rsrc(isout ? dsto[e] : dstw[e], isout ? (unsigned)M * N * 4u : (unsigned)WS_NEED);
    SCHED;
    PUSH(0, KPRE);
    WAIT_V(KPRE);
    if (pflag != ~0u) PUBLISH(pflag);
    pflag = uc.flag;
    SCHED;
    if (!oready && nxt >= NOPER - 480u + wave * 6u) {
      const u32x4 of = __builtin_amdgcn_raw_buffer_load_b128(rq, OFF_OFLAG + lane * 16, 0, 17);
      if (!__all((int)(of[0] - call) >= 0 && (int)(of[1] - call) >= 0 && (int)(of[2] - call) >= 0 && (int)(of[3] - call) >= 0))
        for (int q = 0; q < 4; ++q) spin_flags(myws, OFF_OFLAG + (q * 64 + lane) * 4, 0ull, call);
      oready = true;
    }
    if (un.kind == 2) {
#pragma unroll
      for (int t = 0; t < 4; ++t) { const u32x4 v = __builtin_amdgcn_raw_buffer_load_b128(srco, ovo, un.src + t * 4 * (N * 4), 17);
        d[4 * t] = v[0]; d[4 * t + 1] = v[1]; d[4 * t + 2] = v[2]; d[4 * t + 3] = v[3]; }
    } else {
      unit_load(un, srcx, srcw, lane, d);
    }
    const unsigned ticket = (unsigned)__builtin_amdgcn_raw_ptr_buffer_atomic_add_i32(1, rq, qvo, 0, 16);
    SCHED;
    PUSH(KPRE, 28);
    SCHED;
    cur = nxt; uc = un;
    if (cur >= NUNIT) break;
    nxt = rfl(ticket) - qbase;
  }
  WAIT_V(0);
  if (pflag != ~0u) PUBLISH(pflag);
  if (lane < NR) __hip_atomic_store((gu32*)(p.ws[lane & 7] + OFF_DONE + (rank * 64u + wave) * 4u), call, __ATOMIC_RELAXED, __HIP_MEMORY_SCOPE_SYSTEM);
  return call;
}

template <int AUX> __device__ __forceinline__ void store_rows(const f32x4 (&o)[4], rsrc_t r, unsigned voff, unsigned soff) {
  SCHED;
#pragma unroll
  for (int j = 0; j < 4; ++j) {
    __builtin_amdgcn_raw_buffer_store_b128(__builtin_bit_cast(u32x4, o[j]), r, voff, soff + (unsigned)j * (N * 4), AUX);
  }
  asm volatile("s_nop 1" ::: "memory");
  SCHED;
}
struct Silu { float nz, nc, k2, k; };
__device__ __forceinline__ void silu_rows(const f32x4 (&a)[4], const Silu& c, f32x4 (&o)[4]) {
#pragma unroll
  for (int h = 0; h < 2; ++h) {
    f32x2 r[4];
#pragma unroll
    for (int m = 0; m < 4; ++m) {
      const f32x2 x = __builtin_elementwise_fma((f32x2){a[m][2 * h], a[m][2 * h + 1]}, (f32x2){c.nz, c.nz}, (f32x2){c.nc, c.nc});
      f32x2 e;
      e.x = __builtin_amdgcn_fmed3f(__builtin_amdgcn_exp2f(x.x), 0.f, 1.f); e.y = __builtin_amdgcn_fmed3f(__builtin_amdgcn_exp2f(x.y), 0.f, 1.f);
      const f32x2 dd = __builtin_elementwise_fma(e, (f32x2){c.k2, c.k2}, (f32x2){c.k, c.k});
      r[m].x = __builtin_amdgcn_rcpf(dd.x); r[m].y = __builtin_amdgcn_rcpf(dd.y);
    }
#pragma unroll
    for (int m = 0; m < 4; ++m) {
      o[2 * h][m] = fmul_single(a[m][2 * h], r[m].x); o[2 * h + 1][m] = fmul_single(a[m][2 * h + 1], r[m].y); }
  }
}

__global__ void __launch_bounds__(512, 2) gemm_ag_kernel(Params p) {
  extern __shared__ __attribute__((aligned(16))) unsigned char shm[];
  LAS unsigned char* lds = (LAS unsigned char*)shm;
  u8* myws = p.ws[p.rank];
  const unsigned counter = __hip_atomic_load((gu32*)(myws + OFF_CNT), __ATOMIC_RELAXED, __HIP_MEMORY_SCOPE_AGENT);
  const int bid = blockIdx.x;
  const bool is_copy = bid < NCOPY;
  const int cj = bid;
  if (is_copy) {
    const unsigned call = copy_role(p, counter, cj);
    if (cj == 0) {
      BAR;
      spin_flags(myws, OFF_DONE + (threadIdx.x >> 6) * 256u + (threadIdx.x & 63u) * 4u, 0ull, call);
      if (rfl(__hip_atomic_load((gu32*)(myws + OFF_TMO), __ATOMIC_RELAXED, __HIP_MEMORY_SCOPE_AGENT)) == call && (threadIdx.x & 63u) == 0u) p.out[0] = __builtin_nanf("");
    }
  }

  const int tid = threadIdx.x, wid = rfl(tid >> 6), lane = tid & 63, wr = wid >> 2, wc = wid & 3, fr = lane & 15, fq = lane >> 4;
  const rsrc_t ro = mk_rsrc(p.out, (unsigned)M * N * 4u), rsf = mk_rsrc(myws, OFF_DATA);
  const unsigned voff = lane * 16;
  const unsigned ldsw = (unsigned)wid * 2048u;
  const unsigned xw = (unsigned)wid * XREG, ww = XALL + (unsigned)wid * WREG;
  const float scale = p.sx[0] * p.sw[0];
  const float clip2 = __builtin_fminf(86.56170245333781f, __builtin_fabsf(scale) * (1.4426950408889634f * 1073741824.f) + 1.f);
  Silu sc; sc.nz = scale * -1.4426950408889634f; sc.nc = -clip2; sc.k = 1.f / scale; sc.k2 = sc.k * __builtin_amdgcn_exp2f(clip2);
#define STAGE(bufoff, soff) do { \
    __builtin_amdgcn_raw_ptr_buffer_load_lds(rs, (LAS void*)(lds + (bufoff) + ldsw), 16, voff, (soff), 0, LDAUX); \
    __builtin_amdgcn_raw_ptr_buffer_load_lds(rs, (LAS void*)(lds + (bufoff) + ldsw), 16, voff, (soff), 1024, LDAUX); } while (0)

  const unsigned voffE = (unsigned)(lane >> 4) * 512u + (unsigned)(lane & 15) * 16u;
#define STAGEE(buf, soff) __builtin_amdgcn_raw_ptr_buffer_load_lds(rs, (LAS void*)(lds + (buf) + wid * 1024), 16, voffE, (soff), 0, LDAUX)
#define EBASE(q) (ww + (unsigned)(NSLOT * NROUND + (p.rank >> 1)) * (NT * 4096) + (unsigned)((p.rank & 1) * 2048) + (unsigned)((q) & 1) * 256)
  if (!is_copy) {
    const int gidx = bid - NCOPY, xcd = gidx & 7, idx = gidx >> 3, tl = (xcd - 6) * 30 + idx;
    const int pm = xcd < 6 ? (xcd & 1) * 6 + idx % 6 : 12 + (tl & 3), slot = xcd < 6 ? (xcd >> 1) * 5 + idx / 6 : tl >> 2;
    const unsigned call = local_role(p, counter, (unsigned)gidx * 8u + (unsigned)wid);
    const rsrc_t rs = mk_rsrc(myws + OFF_DATA + (call & 1u) * PAR_BYTES, PAR_BYTES);
    const LAS unsigned char* pA = lds + fq * 2048 + wr * 256 + fr * 16;
    const LAS unsigned char* pB = lds + 4 * HTB + fq * 2048 + wc * 512 + fr * 16;
    const LAS unsigned char* pE = lds + LDS_E + fq * 1024 + fr * 16;
    asm volatile("" : "+v"(pA), "+v"(pB), "+v"(pE));
#define SA(b, h) (((b) * 2 + (h)) * HTB)
#define SB(b, h) ((4 + (b) * 2 + (h)) * HTB)
#define LDA(dst, b, h) do { _Pragma("unroll") for (int m = 0; m < 4; ++m) _Pragma("unroll") for (int k = 0; k < 2; ++k) dst[m][k] = *(const LAS i32x4*)(pA + SA(b, h) + m * 512 + k * 8192); } while (0)
#define LDB(dst, b, h) do { _Pragma("unroll") for (int n = 0; n < 2; ++n) _Pragma("unroll") for (int k = 0; k < 2; ++k) dst[n][k] = *(const LAS i32x4*)(pB + SA(b, h) + n * 256 + k * 8192); } while (0)
#define LDE(dst, b) do { _Pragma("unroll") for (int m = 0; m < 4; ++m) _Pragma("unroll") for (int k = 0; k < 2; ++k) dst[m][k] = *(const LAS i32x4*)(pE + (b) * 8192 + m * 256 + k * 4096); } while (0)
#define MIDX(q) (((q) >> 2) ? 3 - ((q) & 3) : ((q) & 3))
#define NIDX(q) ((q) >> 2)
#define MM8(C, At, Bt) do { __builtin_amdgcn_s_setprio(MMPRIO); _Pragma("unroll") for (int q = 0; q < 8; ++q) { const int m = MIDX(q), n = NIDX(q); \
        asm volatile("v_mfma_f32_16x16x128_f8f6f4 %0, %1, %2, %0" : "+v"(C[m][n]) : "v"(cat(Bt[n][0], Bt[n][1])), "v"(cat(At[m][0], At[m][1]))); } __builtin_amdgcn_s_setprio(0); } while (0)
#define MM8Z(C, At, Bt) do { __builtin_amdgcn_s_setprio(MMPRIO); _Pragma("unroll") for (int q = 0; q < 8; ++q) { const int m = MIDX(q), n = NIDX(q); \
        asm volatile("v_mfma_f32_16x16x128_f8f6f4 %0, %1, %2, 0" : "=&v"(C[m][n]) : "v"(cat(Bt[n][0], Bt[n][1])), "v"(cat(At[m][0], At[m][1]))); } __builtin_amdgcn_s_setprio(0); } while (0)
#define SELB() do { _Pragma("unroll") for (int n = 0; n < 2; ++n) _Pragma("unroll") for (int k = 0; k < 2; ++k) { B0[n][k] = wr ? B1[n][k] : B0[n][k]; asm volatile("" : "+v"(B0[n][k])); } } while (0)
#define MME(At) MM8(accE, At, B0)
#define MMEZ(At) MM8Z(accE, At, B0)
#define MMA(ai, bj, At, Bt) MM8(acc[ai][bj], At, Bt)
#define MMA0(ai, bj, At, Bt) MM8Z(acc[ai][bj], At, Bt)
    f32x4 acc[2][2][4][2], accE[4][2];
    i32x4 At[4][2], B0[2][2], B1[2][2];
    const unsigned kstep = 4096, hstep = 2048, tstep = NT * 4096;
    const unsigned cB = xw + (unsigned)pm * tstep;
    unsigned cA = ww + (unsigned)slot * tstep;
    const bool ex = slot < 2;
    const unsigned cE = EBASE(slot);
    const int l16 = lane & 15;
    const unsigned foff = OFF_FLAGS + (unsigned)wid * FLAG_SRC + (l16 < 8 ? pm * 128 + l16 * 16 : l16 == 10 ? ((16 + NSLOT * NROUND) * 2 + p.rank) * 64 : ((16 + slot) * 2 + (l16 == 9)) * 64);
    const unsigned isw = l16 >= 8;
    constexpr unsigned long long IGN_XE = 0x04ff04ff04ff04ffull, IGN_E = 0x0400040004000400ull;
#define XOFF(s) ((((s) & 63u) >> 4) << 2)
    spin_flags(myws, foff, ex ? 0ull : IGN_E, call);
    unsigned fnew = call - 1u, tm = 0u;
    unsigned sA = 4u, sB = 8u, qn = 2u;
    if (ex) STAGEE(LDS_E, cE);
    STAGE(SB(0, 0), cB); STAGE(SA(0, 0), cA); STAGE(SB(0, 1), cB + hstep); STAGE(SA(0, 1), cA + hstep);
    if (wr == 1) BAR;
    WAIT_V(4); BAR;
    STAGE(SB(1, 0), cB + kstep); STAGE(SA(1, 0), cA + kstep); STAGE(SB(1, 1), cB + hstep + kstep); STAGE(SA(1, 1), cA + hstep + kstep);
    WAIT_V(8); BAR;
#define FCHECK(ign) do { if ((__ballot((int)(fnew - call) >= 0) | (ign)) != ~0ull) spin_flags(myws, foff + (isw ? sA : XOFF(sA)), (ign), call); } while (0)
#define LEVEL(r, q) (REVK && ((r) & 1) ? 15u - (unsigned)(q) : (unsigned)(q))
#define NEXTLEV() do { sA = sB; ++qn; sB = (qn >> 4) * FLAG_ROUND + LEVEL(qn >> 4, qn & 15u) * 4u; } while (0)
#define ITER(EX, FIRST, MM, MMX, W4, W8, FCHK, a1h, a2, b2, e1, e2, LAST) do { \
        const unsigned a3 = (a2) + kstep, b3 = (b2) + kstep; \
          \
        LDB(B0, 0, 0); SCHED; LDA(At, 0, 0); if (!(FIRST)) STAGE(SA(1, 1), a1h); \
        WAIT_L(8); BAR; WAIT_L(0); MM(0, 0, At, B0); BAR; SCHED; \
          \
        FCHK; \
        LDB(B1, 0, 1); STAGE(SB(0, 0), b2); if (EX) STAGEE(LDS_E + 8192, e1); \
        if (LAST) tm = __builtin_amdgcn_raw_buffer_load_b32(rsf, OFF_TMO, 0, 17);        \
        BAR; WAIT_L(0); MM(0, 1, At, B1); BAR; \
          \
        LDA(At, 0, 1); STAGE(SA(0, 0), a2); \
        BAR; WAIT_L(0); MM(1, 0, At, B0); BAR; SCHED; \
          \
        STAGE(SB(0, 1), (b2) + hstep); \
        W4; \
        BAR; MM(1, 1, At, B1); BAR; \
        if (EX) { SCHED; LDE(At, 0); SELB(); BAR; WAIT_L(0); MMX(At); BAR; SCHED; } \
          \
        LDB(B0, 1, 0); SCHED; LDA(At, 1, 0); STAGE(SA(0, 1), (a2) + hstep); \
        WAIT_L(8); BAR; WAIT_L(0); MMA(0, 0, At, B0); BAR; SCHED; \
          \
        LDB(B1, 1, 1); STAGE(SB(1, 0), b3); if (EX) STAGEE(LDS_E, e2); \
        fnew = __builtin_amdgcn_raw_buffer_load_b32(rsf, foff + __umul24(isw, sB - XOFF(sB)), XOFF(sB), 17); \
        BAR; WAIT_L(0); MMA(0, 1, At, B1); BAR; \
          \
        LDA(At, 1, 1); STAGE(SA(1, 0), a3); \
        BAR; WAIT_L(0); MMA(1, 0, At, B0); BAR; SCHED; \
          \
        STAGE(SB(1, 1), b3 + hstep); \
        NEXTLEV(); \
        W8; BAR; MMA(1, 1, At, B1); BAR; \
        if (EX) { SCHED; LDE(At, 1); SELB(); BAR; WAIT_L(0); MME(At); BAR; SCHED; } } while (0)
#define EPI_ENTER() do { if (wr == 0) BAR; } while (0)
#define EPI_LEAVE(more) do { if ((more) && wr == 1) BAR; } while (0)
#define EPILOGUE(r, AUX) do { \
        int tz = threadIdx.x; asm volatile("" : "+v"(tz)); \
        const int lane2 = tz & 63, fr2 = lane2 & 15, fq2 = lane2 >> 4; \
        const int pn = NSLOT * (r) + slot; \
        const unsigned ovoff = (unsigned)(wc * 32 + fq2 * 4) * (N * 4) + (unsigned)(wr * 64 + fr2 * 4) * 4; \
        const unsigned osoff = (unsigned)(pm * 256) * (N * 4) + (unsigned)pn * 1024; \
        Silu sv = sc; if (rfl(tm) == call) sv.k = __builtin_nanf(""); \
        _Pragma("unroll") for (int ai = 0; ai < 2; ++ai) _Pragma("unroll") for (int bj = 0; bj < 2; ++bj) _Pragma("unroll") for (int n = 0; n < 2; ++n) { \
              const f32x4 a4[4] = {acc[ai][bj][0][n], acc[ai][bj][1][n], acc[ai][bj][2][n], acc[ai][bj][3][n]}; \
              f32x4 o4[4]; \
              silu_rows(a4, sv, o4); \
              store_rows<AUX>(o4, ro, ovoff, osoff + (unsigned)(bj * 128 + n * 16) * (N * 4) + ai * 512); } } while (0)
    if (ex) {
      const unsigned nA = cA + NSLOT * tstep;
      ITER(true, true, MMA0, MMEZ, WAIT_V(7), WAIT_V(8), FCHECK(0ull), 0u, cA + 2 * kstep, cB + 2 * kstep, cE + kstep, cE + 2 * kstep, false);
      for (int t = 2; t < NT - 2; t += 2) {
        ITER(true, false, MMA, MME, WAIT_V(7), WAIT_V(8), FCHECK(0ull), cA + (unsigned)(t + 1) * kstep + hstep, cA + (unsigned)(t + 2) * kstep, cB + (unsigned)(t + 2) * kstep,
             cE + (unsigned)(t + 1) * kstep, cE + (unsigned)(t + 2) * kstep, false);
      }
      ITER(true, false, MMA, MME, WAIT_V(8), WAIT_V(8), FCHECK(IGN_XE), cA + (unsigned)(NT - 1) * kstep + hstep, nA + LEVEL(1, 0) * 8192u, cB + LEVEL(1, 0) * 8192u, cE + (unsigned)(NT - 1) * kstep, cE, true);
      asm volatile("s_nop 15" ::: "memory");
      EPI_ENTER();
      {
        int tz = threadIdx.x; asm volatile("" : "+v"(tz));
        const int lane2 = tz & 63, fr2 = lane2 & 15, fq2 = lane2 >> 4;
        const unsigned ovoff = (unsigned)(wr * 128 + wc * 32 + fq2 * 4) * (N * 4) + (unsigned)(fr2 * 4) * 4;
        const unsigned osoff = (unsigned)(pm * 256) * (N * 4) + (unsigned)(NSLOT * NROUND * 256 + p.rank * 128 + slot * 64) * 4;
        Silu sv = sc; if (rfl(tm) == call) sv.k = __builtin_nanf("");
#pragma unroll
        for (int n = 0; n < 2; ++n) {
          const f32x4 a4[4] = {accE[0][n], accE[1][n], accE[2][n], accE[3][n]};
          f32x4 o4[4];
          silu_rows(a4, sv, o4);
          store_rows<17>(o4, ro, ovoff, osoff + (unsigned)(n * 16) * (N * 4));
        }
      }
      STAGE(SA(1, 1), nA + LEVEL(1, 0) * 8192u + kstep + hstep);
      EPILOGUE(0, 2);
      EPI_LEAVE(true);
      cA = nA;
    }
    for (int r = ex ? 1 : 0; r < NROUND; ++r) {
      const bool has_next = r + 1 < NROUND;
      const unsigned long long ign = r == 0 ? IGN_E : IGN_XE;
      const unsigned nA = has_next ? cA + NSLOT * tstep : cA;
      ITER(false, true, MMA0, MMEZ, if (r > 0) { WAIT_V(38); if (ex && r == 1 && lane == 0) __hip_atomic_store((gu32*)(myws + OFF_OFLAG + ((pm * 2 + slot) * 8 + wid) * 4), call, __ATOMIC_RELAXED, __HIP_MEMORY_SCOPE_SYSTEM); } else WAIT_V(6),
           WAIT_V(7), FCHECK(ign), 0u, cA + LEVEL(r, 1) * 8192u, cB + LEVEL(r, 1) * 8192u, 0u, 0u, false);
      for (int q = 1; q < 15; ++q) {
        const unsigned lc = LEVEL(r, q) * 8192u, ln = LEVEL(r, q + 1) * 8192u;
        ITER(false, false, MMA, MME, WAIT_V(6), WAIT_V(7), FCHECK(ign), cA + lc + kstep + hstep, cA + ln, cB + ln, 0u, 0u, false);
      }
      const unsigned long long ignl = has_next ? IGN_XE : ~0ull;
      ITER(false, false, MMA, MME, WAIT_V(7), WAIT_V(7), FCHECK(ignl), cA + LEVEL(r, 15) * 8192u + kstep + hstep, nA + LEVEL(r + 1, 0) * 8192u, cB + LEVEL(r + 1, 0) * 8192u, 0u, 0u, true);
      STAGE(SA(1, 1), nA + LEVEL(r + 1, 0) * 8192u + kstep + hstep);
      asm volatile("s_nop 15" ::: "memory");
      EPI_ENTER();
      EPILOGUE(r, 2);
      EPI_LEAVE(has_next);
      cA = nA;
    }
    WAIT_V(0);
  }
  if (threadIdx.x == 0) __hip_atomic_fetch_add((gu32*)(myws + OFF_CNT), 1u, __ATOMIC_RELAXED, __HIP_MEMORY_SCOPE_AGENT);
}
}
#define HIP_CHECK(expr) do { hipError_t _e = (expr); if (_e != hipSuccess) { \
    fprintf(stderr, "HIP error %s at %s:%d\n", hipGetErrorString(_e), __FILE__, __LINE__); \
    return; } } while (0)

extern "C" void kernel_launch(int rank, int nranks, ncclComm_t comm, hipStream_t stream,
                              void* const* d_in, const int* in_sizes, int n_in,
                              void* d_out, int out_size, const int* in_dtypes, int out_dtype,
                              void* const* d_out_all, void* const* d_ws_all, size_t ws_bytes) {
  (void)comm; (void)in_dtypes; (void)out_dtype; (void)d_out_all;
  if (nranks != gar::NR || n_in != 4 || (unsigned)rank >= (unsigned)gar::NR || ws_bytes < gar::WS_NEED ||
      in_sizes[0] != gar::M * gar::KS || in_sizes[1] != gar::N * gar::KS || out_size != gar::M * gar::N) {
    fprintf(stderr, "kernel_launch: unexpected shapes\n");
    return;
  }
  static bool attr_set = false;
  if (!attr_set) {
    HIP_CHECK(hipFuncSetAttribute((const void*)gar::gemm_ag_kernel, hipFuncAttributeMaxDynamicSharedMemorySize, gar::LDS_BYTES));
    attr_set = true;
  }
  gar::Params p;
  p.x = (const gar::u8*)d_in[0]; p.w = (const gar::u8*)d_in[1];
  p.sx = (const float*)d_in[2]; p.sw = (const float*)d_in[3];
  p.out = (float*)d_out;
  for (int i = 0; i < gar::NR; ++i) p.ws[i] = (gar::u8*)d_ws_all[i];
  for (int i = 0; i < gar::NR; ++i) p.outs[i] = (float*)d_out_all[i];
  p.rank = rank; p.pad = 0;
  gar::gemm_ag_kernel<<<gar::GRID, 512, gar::LDS_BYTES, stream>>>(p);
  HIP_CHECK(hipGetLastError());
}
```
